```python
import numpy as np

kernel_name = "gpu_gemm_ar_m2048_k2048_n2048_bf16_relu"


ABSMAX_RTOL = 0.008

COMPUTE_DTYPE = "bfloat16"
EPILOGUE = "relu"
IN_QUANT = None
OUT_QUANT = None
IN_DTYPES = {"x": "bfloat16", "w_mat": "bfloat16"}
WORLD_SIZE = 8

def _to_bf16(x):
    u = np.ascontiguousarray(x, dtype=np.float32).view(np.uint32)
    return ((u + 0x7FFF + ((u >> 16) & 1)) & 0xFFFF0000).view(np.float32)

def setup_inputs():

    x = _to_bf16(np.random.randn(2048, 2048).astype(np.float32))
    w_mat = _to_bf16((np.random.randn(2048, 2048) / np.sqrt(2048)).astype(np.float32))
    return {"x": x, "w_mat": w_mat}

def _epilogue(y):
    return np.maximum(y, 0.0).astype(np.float32)

def reference(x, w_mat):
    y = (x @ w_mat).astype(np.float32)
    return _epilogue(y)
```

```cpp
#include <hip/hip_runtime.h>
#include <rccl/rccl.h>
#include <cstdio>
#include <cstdlib>

#define GAS __attribute__((address_space(1)))
#define LAS __attribute__((address_space(3)))
typedef unsigned u32;
typedef unsigned long long u64;
typedef u32 u32x4 __attribute__((ext_vector_type(4)));
typedef u32 u32x2 __attribute__((ext_vector_type(2)));
typedef float f32x4 __attribute__((ext_vector_type(4)));
typedef float f32x16 __attribute__((ext_vector_type(16)));
typedef __bf16 bf16x8 __attribute__((ext_vector_type(8)));
typedef GAS u32 gu32; typedef GAS u64 gu64; typedef GAS unsigned char gu8; typedef GAS unsigned short gu16;

constexpr int NR = 8, M_ = 2048, N_ = 2048, KS = 256;
constexpr int NWG = 256, NTHR = 512;
constexpr size_t WS_TMO = 0;
constexpr size_t WS_CNT = 4096;
constexpr size_t WS_XFLAG = WS_CNT + 256 * 128;
constexpr size_t WS_WFLAG = WS_XFLAG + 8 * 32 * 128;
constexpr size_t WS_OFLAG = WS_WFLAG + 8 * 8 * 128;
constexpr size_t WS_XDATA = 1u << 20;
constexpr size_t WS_WDATA = WS_XDATA + (size_t)8 * 32 * 4 * 4096;
constexpr size_t WS_END = WS_WDATA + (size_t)8 * 8 * 8 * 4096;
constexpr size_t DIAG_OFF = 48u << 20;
constexpr size_t DIAG_BYTES = 256 * 8 * 16 * 8;
static_assert(WS_OFLAG + 8 * 256 * 128 <= WS_XDATA, "flags overlap data");
static_assert(WS_END <= DIAG_OFF, "ws overlap");
constexpr u64 DEADLINE = 30000000ull;

struct Params {
    const unsigned short* x; const unsigned short* w;
    unsigned char* ws[NR]; unsigned char* out[NR];
    int rank; int tW, tSlot, tHold;
};

__device__ __forceinline__ u64 rt() { return __builtin_amdgcn_s_memrealtime(); }
__device__ __forceinline__ void st16_sys(gu8* p, u32x4 v) { asm volatile("global_store_dwordx4 %0, %1, off sc0 sc1\n\ts_nop 1" :: "v"(p), "v"(v) : "memory"); }
__device__ __forceinline__ void drain() { asm volatile("s_waitcnt vmcnt(0)" ::: "memory"); }
__device__ __forceinline__ void flag_store(gu8* p, u64 v) { __hip_atomic_store((gu64*)p, v, __ATOMIC_RELAXED, __HIP_MEMORY_SCOPE_SYSTEM); }

__device__ __forceinline__ bool wait_flags(gu8* p, u32 call, gu32* tmo, u64& val) {
    const u64 t0 = rt();
    for (u32 it = 1;; ++it) {
        const u64 v = __hip_atomic_load((gu64*)p, __ATOMIC_RELAXED, __HIP_MEMORY_SCOPE_SYSTEM);
        if (__all((u32)(v >> 32) >= call)) { val = v; asm volatile("" ::: "memory"); return true; }
        if ((it & 31u) == 0u) {
            if (__hip_atomic_load(tmo, __ATOMIC_RELAXED, __HIP_MEMORY_SCOPE_AGENT) == call || rt() - t0 > DEADLINE) {
                __hip_atomic_store(tmo, call, __ATOMIC_RELAXED, __HIP_MEMORY_SCOPE_AGENT);
                val = 0; return false;
            }
        }
        __builtin_amdgcn_s_sleep(2);
    }
}
__device__ __forceinline__ void wait_until(u64 t) { while (rt() < t) __builtin_amdgcn_s_sleep(1); }

#define STAMP(i) do { } while (0)

constexpr int RED_STRIDE = 272;
constexpr int RED_WAVE = 32 * RED_STRIDE;
constexpr int L_RED = 0;
constexpr int L_XS = 69632;
constexpr int L_WS = L_XS + 4096;
constexpr int L_OUT = L_WS + 4096;
constexpr int L_MISC = L_OUT + 4096;
constexpr int LDS_BYTES = 90112;
static_assert(L_MISC + 64 <= LDS_BYTES, "lds");

__global__ void __launch_bounds__(NTHR, 2) fused(Params P) {
    extern __shared__ __attribute__((aligned(16))) unsigned char smem[];
    LAS unsigned char* lds = (LAS unsigned char*)smem;
    const int tid = threadIdx.x, lane = tid & 63, wv = __builtin_amdgcn_readfirstlane(tid >> 6);
    const int g = blockIdx.x, xcd = g & 7, idx = g >> 3, u = 8 * (idx >> 3) + xcd, v = idx & 7;
    const int rank = P.rank;
    gu8* myws = (gu8*)P.ws[rank];
    const u64 tstart = rt();
    STAMP(0);
    gu32* tmo = (gu32*)(myws + WS_TMO);
    gu32* cntp = (gu32*)(myws + WS_CNT + g * 128);
    u32 call0 = 0;
    if (tid == 0) call0 = __hip_atomic_load(cntp, __ATOMIC_RELAXED, __HIP_MEMORY_SCOPE_AGENT);
    const int xU = 32 * (v >> 2) + u, xB = v & 3;
    const int wV = g >> 3, wb = g & 7;
    const int r = lane & 31, h = lane >> 5;
    u32x4 frag;
    if (wv < 4) {
        frag = *(const GAS u32x4*)((const gu16*)P.x + (size_t)(32 * xU + r) * KS + 64 * xB + 16 * wv + 8 * h);
        *(LAS u32x4*)(lds + L_XS + wv * 1024 + lane * 16) = frag;
    } else {
        const int f = wv - 4, cb = f & 1, t = f >> 1;
        const gu16* src = (const gu16*)P.w + (size_t)(32 * wb + 16 * t + 8 * h) * N_ + 64 * wV + 32 * cb + r;
        u32 e[8];
#pragma unroll
        for (int j = 0; j < 8; ++j) e[j] = src[(size_t)j * N_];
        frag = (u32x4){e[0] | (e[1] << 16), e[2] | (e[3] << 16), e[4] | (e[5] << 16), e[6] | (e[7] << 16)};
        *(LAS u32x4*)(lds + L_WS + f * 1024 + lane * 16) = frag;
    }
    if (tid == 0) { call0 += 1u; *(LAS u32*)(lds + L_MISC) = call0; __hip_atomic_store(cntp, call0, __ATOMIC_RELAXED, __HIP_MEMORY_SCOPE_AGENT); }
    STAMP(1);
    __syncthreads();
    const u32 call = __builtin_amdgcn_readfirstlane(*(LAS u32*)(lds + L_MISC));
    const u64 fv = (u64)call << 32;
    if (wv < 2) {
        const int q = (wV >> 3) + 4 * wv, vv = wV & 7;
        gu8* dws = (gu8*)P.ws[q];
        gu8* dst = dws + WS_WDATA + ((size_t)(rank * 8 + vv) * 8 + wb) * 4096 + lane * 16;
        u32x4 d[4];
#pragma unroll
        for (int f = 0; f < 4; ++f) d[f] = *(LAS u32x4*)(lds + L_WS + f * 1024 + lane * 16);
#pragma unroll
        for (int f = 0; f < 4; ++f) st16_sys(dst + f * 1024, d[f]);
        STAMP(2);
        drain();
        if (lane == 0) flag_store(dws + WS_WFLAG + (rank * 8 + vv) * 128 + wb * 8, fv);
        STAMP(3);
    } else if (wv < 6) {
        const int q = 4 * (v >> 2) + (wv - 2);
        gu8* dws = (gu8*)P.ws[q];
        gu8* dst = dws + WS_XDATA + ((size_t)(rank * 32 + u) * 4 + xB) * 4096 + lane * 16;
        u32x4 d[4];
#pragma unroll
        for (int f = 0; f < 4; ++f) d[f] = *(LAS u32x4*)(lds + L_XS + f * 1024 + lane * 16);
        wait_until(tstart + (u64)(P.tW + ((u * P.tSlot) >> 4)));
#pragma unroll
        for (int f = 0; f < 4; ++f) st16_sys(dst + f * 1024, d[f]);
        STAMP(2);
        drain();
        if (lane == 0) flag_store(dws + WS_XFLAG + (rank * 32 + u) * 128 + xB * 8, fv);
        STAMP(3);
    }
    bool ok = true; u64 fval;
    const int p = wv;
    ok &= wait_flags(myws + WS_WFLAG + (p * 8 + v) * 128 + (lane & 7) * 8, call, tmo, fval);
    STAMP(4);
    bf16x8 wf[16][2];
    {
        const __amdgpu_buffer_rsrc_t rs = __builtin_amdgcn_make_buffer_rsrc((void*)(myws + WS_WDATA + (size_t)(p * 8 + v) * 8 * 4096), 0, 8 * 4096, 0x00020000);
#pragma unroll
        for (int T = 0; T < 16; ++T)
#pragma unroll
            for (int cb = 0; cb < 2; ++cb)
                wf[T][cb] = __builtin_bit_cast(bf16x8, __builtin_amdgcn_raw_buffer_load_b128(rs, lane * 16, (T >> 1) * 4096 + (2 * (T & 1) + cb) * 1024, 17));
    }
    STAMP(5);
    ok &= wait_flags(myws + WS_XFLAG + (p * 32 + u) * 128 + (lane & 3) * 8, call, tmo, fval);
    STAMP(6);
    f32x16 acc[2];
#pragma unroll
    for (int i = 0; i < 16; ++i) { acc[0][i] = 0.f; acc[1][i] = 0.f; }
    {
        const __amdgpu_buffer_rsrc_t rs = __builtin_amdgcn_make_buffer_rsrc((void*)(myws + WS_XDATA + (size_t)(p * 32 + u) * 4 * 4096), 0, 4 * 4096, 0x00020000);
        bf16x8 xf[16];
#pragma unroll
        for (int T = 0; T < 16; ++T) xf[T] = __builtin_bit_cast(bf16x8, __builtin_amdgcn_raw_buffer_load_b128(rs, lane * 16, T * 1024, 17));
        __builtin_amdgcn_sched_barrier(0);
#pragma unroll
        for (int T = 0; T < 16; ++T) {
            acc[0] = __builtin_amdgcn_mfma_f32_32x32x16_bf16(wf[T][0], xf[T], acc[0], 0, 0, 0);
            acc[1] = __builtin_amdgcn_mfma_f32_32x32x16_bf16(wf[T][1], xf[T], acc[1], 0, 0, 0);
        }
    }
#pragma unroll
    for (int cb = 0; cb < 2; ++cb)
#pragma unroll
        for (int g4 = 0; g4 < 4; ++g4)
            *(LAS f32x4*)(lds + L_RED + p * RED_WAVE + r * RED_STRIDE + (32 * cb + 8 * g4 + 4 * h) * 4) =
                (f32x4){acc[cb][4 * g4], acc[cb][4 * g4 + 1], acc[cb][4 * g4 + 2], acc[cb][4 * g4 + 3]};
    if (!ok && lane == 0) *(LAS u32*)(lds + L_MISC + 4) = 1u;
    STAMP(7);
    __syncthreads();
    {
        const int m = tid >> 4, n0 = 4 * (tid & 15);
        f32x4 s = *(LAS f32x4*)(lds + L_RED + m * RED_STRIDE + n0 * 4);
#pragma unroll
        for (int pp = 1; pp < 8; ++pp) s += *(LAS f32x4*)(lds + L_RED + pp * RED_WAVE + m * RED_STRIDE + n0 * 4);
        const u32 tm = __hip_atomic_load(tmo, __ATOMIC_RELAXED, __HIP_MEMORY_SCOPE_AGENT);
        u32 o[4];
#pragma unroll
        for (int j = 0; j < 4; ++j) {
            float y = s[j] < 0.f ? 0.f : s[j];
            u32 b = __float_as_uint(y); b = (b + 0x7FFFu + ((b >> 16) & 1u)) >> 16;
            if (y != y) b = 0x7FC0u;
            if (tm == call) b = 0x7FC0u;
            o[j] = b;
        }
        *(LAS u32x2*)(lds + L_OUT + m * 128 + n0 * 2) = (u32x2){o[0] | (o[1] << 16), o[2] | (o[3] << 16)};
    }
    STAMP(8);
    __syncthreads();
    const int tile = u * 8 + v;
    {
        u32x4 d[4];
#pragma unroll
        for (int s = 0; s < 4; ++s) d[s] = *(LAS u32x4*)(lds + L_OUT + s * 1024 + lane * 16);
        const int row0 = 1024 * (rank >> 2) + 32 * u, col0 = 512 * (rank & 3) + 64 * v;
        gu8* dst = (gu8*)P.out[wv] + ((size_t)(row0 + (lane >> 3)) * N_ + col0) * 2 + (lane & 7) * 16;
        wait_until(tstart + (u64)P.tHold);
#pragma unroll
        for (int s = 0; s < 4; ++s) st16_sys(dst + (size_t)s * 8 * N_ * 2, d[s]);
        STAMP(9);
        drain();
        if (wv != rank) {
            if (lane == 0) flag_store((gu8*)P.ws[wv] + WS_OFLAG + (rank * 256 + tile) * 128, fv);
            STAMP(10);
            const bool ok2 = wait_flags(myws + WS_OFLAG + (wv * 256 + tile) * 128, call, tmo, fval);
            if (!ok2 && lane == 0) *(gu16*)((gu8*)P.out[rank] + ((size_t)row0 * N_ + col0) * 2) = 0x7FC0;
        }
        STAMP(11);
    }
}

extern "C" void kernel_launch(int rank, int nranks, ncclComm_t comm, hipStream_t stream,
                              void* const* d_in, const int* in_sizes, int n_in,
                              void* d_out, int out_size, const int* in_dtypes, int out_dtype,
                              void* const* d_out_all, void* const* d_ws_all, size_t ws_bytes) {
    if (nranks != NR || ws_bytes < DIAG_OFF + DIAG_BYTES) { fprintf(stderr, "bad config\n"); return; }
    static bool init = false;
    static int tW, tSlot, tHold;
    if (!init) {
        (void)hipFuncSetAttribute((const void*)fused, hipFuncAttributeMaxDynamicSharedMemorySize, LDS_BYTES);
        tW = 430; tSlot = 600; tHold = 1800;
        init = true;
    }
    Params P;
    P.x = (const unsigned short*)d_in[0]; P.w = (const unsigned short*)d_in[1];
    for (int i = 0; i < NR; ++i) { P.ws[i] = (unsigned char*)d_ws_all[i]; P.out[i] = (unsigned char*)d_out_all[i]; }
    P.rank = rank; P.tW = tW; P.tSlot = tSlot; P.tHold = tHold;
    hipLaunchKernelGGL(fused, dim3(NWG), dim3(NTHR), LDS_BYTES, stream, P);
}
```

```cpp
#include <hip/hip_runtime.h>
#include <rccl/rccl.h>
#include <cstdio>
#include <cstdlib>

#define GAS __attribute__((address_space(1)))
#define LAS __attribute__((address_space(3)))
typedef unsigned u32;
typedef unsigned long long u64;
typedef u32 u32x4 __attribute__((ext_vector_type(4)));
typedef u32 u32x2 __attribute__((ext_vector_type(2)));
typedef float f32x4 __attribute__((ext_vector_type(4)));
typedef float f32x16 __attribute__((ext_vector_type(16)));
typedef __bf16 bf16x8 __attribute__((ext_vector_type(8)));
typedef GAS u32 gu32; typedef GAS u64 gu64; typedef GAS unsigned char gu8; typedef GAS unsigned short gu16;
typedef LAS unsigned char lu8; typedef LAS u32 lu32;

constexpr int NR = 8, N_ = 2048, KS = 256;
constexpr int NWG = 256, NTHR = 512;
constexpr size_t WS_TMO = 0;
constexpr size_t WS_CNT = 4096;
constexpr size_t WS_XFLAG = WS_CNT + 256 * 128;
constexpr size_t WS_WFLAG = WS_XFLAG + 8 * 32 * 128;
constexpr size_t WS_OFLAG = WS_WFLAG + 8 * 8 * 128;
constexpr size_t WS_XEXC = WS_OFLAG + 8 * 256 * 128;
constexpr size_t WS_WEXC = WS_XEXC + 8 * 32 * 128;
constexpr size_t WS_FEND = WS_WEXC + 8 * 8 * 256;
#define SLOT 4224
#define SKEW 20608
constexpr size_t XSRC = (size_t)32 * 4 * SLOT + SKEW, WSRC = (size_t)8 * 8 * SLOT + SKEW, OSRC = (size_t)256 * SLOT + SKEW;
constexpr size_t WS_XDATA = 1u << 20;
constexpr size_t WS_WDATA = WS_XDATA + 8 * XSRC;
constexpr size_t WS_ODATA = WS_WDATA + 8 * WSRC;
constexpr size_t WS_END = WS_ODATA + 8 * OSRC;
static_assert(SLOT % 128 == 0 && SLOT >= 4096 && SKEW % 128 == 0, "slot");
constexpr size_t DIAG_OFF = 48u << 20;
constexpr size_t DIAG_BYTES = 256 * 8 * 16 * 8;
static_assert(WS_FEND <= WS_XDATA, "flags overlap data");
static_assert(WS_END <= DIAG_OFF, "ws overlap");
constexpr u64 DEADLINE = 30000000ull;

struct Params {
    const unsigned short* x; const unsigned short* w;
    unsigned char* ws[NR]; unsigned char* out[NR];
    int rank; int tX0, dX, tO0, dO, tW0, dW;
    int tPoll, tWj, tSelf, tWd, pad;
};

__device__ __forceinline__ u64 rt() { return __builtin_amdgcn_s_memrealtime(); }
__device__ __forceinline__ void st16_sys(gu8* p, u32x4 v) { asm volatile("global_store_dwordx4 %0, %1, off sc0 sc1\n\ts_nop 1" :: "v"(p), "v"(v) : "memory"); }
__device__ __forceinline__ void st4_sys(gu8* p, u32 v) { asm volatile("global_store_dword %0, %1, off sc0 sc1" :: "v"(p), "v"(v) : "memory"); }
__device__ __forceinline__ void drain() { asm volatile("s_waitcnt vmcnt(0)" ::: "memory"); }
__device__ __forceinline__ void flag_store(gu8* p, u64 v) { __hip_atomic_store((gu64*)p, v, __ATOMIC_RELAXED, __HIP_MEMORY_SCOPE_SYSTEM); }
__device__ __forceinline__ u32 rfl(u32 v) { return __builtin_amdgcn_readfirstlane(v); }
__device__ __forceinline__ u32 perm(u32 hi, u32 lo, u32 sel) { return __builtin_amdgcn_perm(hi, lo, sel); }
__device__ __forceinline__ u32 rotr(u32 x, u32 s) { return __builtin_amdgcn_alignbit(x, x, s); }
__device__ __forceinline__ u32 mbcnt64(u64 m, u32 acc) { return __builtin_amdgcn_mbcnt_hi((u32)(m >> 32), __builtin_amdgcn_mbcnt_lo((u32)m, acc)); }
template <int CTRL, int RM> __device__ __forceinline__ u32 dpp0(u32 v) { return (u32)__builtin_amdgcn_update_dpp(0, (int)v, CTRL, RM, 0xf, false); }
__device__ __forceinline__ u32 wave_max(u32 v) {
    v = max(v, dpp0<0x111, 0xf>(v)); v = max(v, dpp0<0x112, 0xf>(v)); v = max(v, dpp0<0x114, 0xf>(v)); v = max(v, dpp0<0x118, 0xf>(v));
    v = max(v, dpp0<0x142, 0xa>(v)); v = max(v, dpp0<0x143, 0xc>(v));
    return (u32)__builtin_amdgcn_readlane((int)v, 63);
}
__device__ __forceinline__ u32 wave_scan_add(u32 v) {
    v += dpp0<0x111, 0xf>(v); v += dpp0<0x112, 0xf>(v); v += dpp0<0x114, 0xf>(v); v += dpp0<0x118, 0xf>(v);
    v += dpp0<0x142, 0xa>(v); v += dpp0<0x143, 0xc>(v);
    return v;
}

__device__ __forceinline__ bool wait_flags(gu8* p, u32 call, gu32* tmo, u64& val) {
    const u64 t0 = rt();
    for (u32 it = 1;; ++it) {
        const u64 v = __hip_atomic_load((gu64*)p, __ATOMIC_RELAXED, __HIP_MEMORY_SCOPE_SYSTEM);
        if (__all((u32)(v >> 32) >= call)) { val = v; asm volatile("" ::: "memory"); return true; }
        if ((it & 31u) == 0u) {
            if (__hip_atomic_load(tmo, __ATOMIC_RELAXED, __HIP_MEMORY_SCOPE_AGENT) == call || rt() - t0 > DEADLINE) {
                __hip_atomic_store(tmo, call, __ATOMIC_RELAXED, __HIP_MEMORY_SCOPE_AGENT);
                val = 0; asm volatile("" ::: "memory"); return false;
            }
        }
        __builtin_amdgcn_s_sleep(2);
    }
}
__device__ __forceinline__ void wait_until(u64 t) { while (rt() < t) __builtin_amdgcn_s_sleep(1); }

#define STAMP(i) do { } while (0)

constexpr int RED_STRIDE = 272;
constexpr int RED_WAVE = 32 * RED_STRIDE;
constexpr int L_RED = 0;
constexpr int L_XP = 69632;
constexpr int L_WP = L_XP + 3072;
constexpr int L_XR = L_WP + 3072;
constexpr int L_WR = L_XR + 4096;
constexpr int L_XH = L_WR + 4096;
constexpr int L_WH = L_XH + 64;
constexpr int L_OUT = L_WH + 64;
constexpr int L_IMG = L_OUT + 4096;
constexpr int L_LUT = L_IMG + 3200;
constexpr int L_MISC = L_LUT + 2048;
constexpr int LDS_BYTES = L_MISC + 128;
static_assert(LDS_BYTES > 81920 && LDS_BYTES <= 163840 && (L_XP % 16) == 0 && (L_IMG % 16) == 0 && (L_LUT % 16) == 0, "lds");
constexpr u32 F_RAW = 1u << 31, F_EXC = 1u << 30;
constexpr u32 EXC_NONE = 0xFFFFFFFFu;

__device__ __forceinline__ void pack_frag(u32x4 e, int f, int lane, lu8* P, lu8* R, lu8* H) {
    *(LAS u32x4*)(R + f * 1024 + lane * 16) = e;
    typedef unsigned short u16x2 __attribute__((ext_vector_type(2)));
    const u16x2 m01 = __builtin_elementwise_max(__builtin_bit_cast(u16x2, e.x & 0x7FFF7FFFu), __builtin_bit_cast(u16x2, e.y & 0x7FFF7FFFu));
    const u16x2 m23 = __builtin_elementwise_max(__builtin_bit_cast(u16x2, e.z & 0x7FFF7FFFu), __builtin_bit_cast(u16x2, e.w & 0x7FFF7FFFu));
    const u16x2 mm = __builtin_elementwise_max(m01, m23);
    const u32 mx = wave_max(max((u32)mm.x, (u32)mm.y));
    const u32 h7 = mx >> 8, base = h7 > 7u ? h7 - 7u : 0u;
    const u32 lo0 = perm(e.y, e.x, 0x06040200u), lo1 = perm(e.w, e.z, 0x06040200u);
    const u32 hi0 = perm(e.y, e.x, 0x07050301u), hi1 = perm(e.w, e.z, 0x07050301u);
    const u32 K = (128u - base) * 0x01010101u;
    const u32 d0 = (hi0 & 0x7F7F7F7Fu) + K, d1 = (hi1 & 0x7F7F7F7Fu) + K;
    const u32 cA = (hi0 & 0x80808080u) | (d0 & 0x07070707u), cB = (hi1 & 0x80808080u) | (d1 & 0x07070707u);
    const u32 nw = cA | rotr(cB, 28);
    *(LAS u32x2*)(P + (f >> 1) * 1024 + lane * 16 + (f & 1) * 8) = (u32x2){lo0, lo1};
    *(LAS u32*)(P + 2048 + lane * 16 + f * 4) = nw;
    const u32 x0 = ~d0 & 0x80808080u, x1 = ~d1 & 0x80808080u;
    u32 hdr = base;
    if (lane < 2) *(LAS u32*)(H + 16 + f * 8 + lane * 4) = EXC_NONE;
    asm volatile("" ::: "memory");
    if (__any((x0 | x1) != 0u)) {
        u32 cnt = 0;
#pragma unroll
        for (int j = 0; j < 8; ++j) {
            const bool ex = (((j < 4 ? x0 : x1) >> (8 * (j & 3) + 7)) & 1u) != 0u;
            const u64 b = __ballot(ex);
            const u32 rk = mbcnt64(b, cnt);
            const u32 val = (e[j >> 1] >> (16 * (j & 1))) & 0xFFFFu;
            if (ex && rk < 2u) *(LAS u32*)(H + 16 + f * 8 + rk * 4) = ((u32)(f * 512 + lane * 8 + j) << 16) | val;
            cnt += (u32)__popcll(b);
        }
        hdr |= cnt > 2u ? 0x300u : 0x100u;
    }
    if (lane == 0) *(LAS u32*)(H + f * 4) = hdr;
}

__device__ __forceinline__ void send_block(const u32x4 (&d)[4], bool raw, bool exc, u32 excw, int lane, gu8* dst, gu8* dexc) {
    st16_sys(dst + lane * 16, d[0]); st16_sys(dst + 1024 + lane * 16, d[1]); st16_sys(dst + 2048 + lane * 16, d[2]);
    if (raw) st16_sys(dst + 3072 + lane * 16, d[3]);
    if (exc && lane < 8) st4_sys(dexc + lane * 4, excw);
}
__device__ __forceinline__ void load_block(u32x4 (&d)[4], u32& fl, u32& excw, int lane, lu8* P, lu8* R, lu8* H) {
    const u32x4 h = *(LAS u32x4*)H;
    const u32 any = rfl(h.x | h.y | h.z | h.w);
    const bool raw = (any & 0x200u) != 0u;
    fl = (rfl(h.x) & 127u) | ((rfl(h.y) & 127u) << 7) | ((rfl(h.z) & 127u) << 14) | ((rfl(h.w) & 127u) << 21) | (raw ? F_RAW : ((any & 0x100u) ? F_EXC : 0u));
    lu8* S = raw ? R : P;
    d[0] = *(LAS u32x4*)(S + lane * 16); d[1] = *(LAS u32x4*)(S + 1024 + lane * 16); d[2] = *(LAS u32x4*)(S + 2048 + lane * 16);
    d[3] = *(LAS u32x4*)(R + 3072 + lane * 16);
    excw = *(LAS u32*)(H + 16 + (lane & 7) * 4);
}

__device__ __forceinline__ u32x4 unpack_frag(u32 lo0, u32 lo1, u32 nw, u32 b4) {
    const u32 hA = (nw & 0x87878787u) + b4, hB = (rotr(nw, 4) & 0x87878787u) + b4;
    return (u32x4){perm(hA, lo0, 0x05010400u), perm(hA, lo0, 0x07030602u), perm(hB, lo1, 0x05010400u), perm(hB, lo1, 0x07030602u)};
}
__device__ __forceinline__ void unpack_block(u32x4 (&fr)[4], u32x4 L0, u32x4 L1, u32x4 Nw, u32 fl, u32 excw, int blk, int lane, lu8* scratch,
                                             __amdgpu_buffer_rsrc_t rs, int soff) {
    if (fl & F_RAW) {
        fr[0] = L0; fr[1] = L1; fr[2] = Nw;
        fr[3] = __builtin_amdgcn_raw_buffer_load_b128(rs, lane * 16, soff + 3072, 17);
        asm volatile("" : "+v"(fr[3]));
        return;
    }
    fr[0] = unpack_frag(L0.x, L0.y, Nw.x, (fl & 127u) * 0x01010101u);
    fr[1] = unpack_frag(L0.z, L0.w, Nw.y, ((fl >> 7) & 127u) * 0x01010101u);
    fr[2] = unpack_frag(L1.x, L1.y, Nw.z, ((fl >> 14) & 127u) * 0x01010101u);
    fr[3] = unpack_frag(L1.z, L1.w, Nw.w, ((fl >> 21) & 127u) * 0x01010101u);
    if (fl & F_EXC) {
#pragma unroll
        for (int f = 0; f < 4; ++f) *(LAS u32x4*)(scratch + f * 1024 + lane * 16) = fr[f];
        asm volatile("" ::: "memory");
        if ((lane >> 3) == blk && excw != EXC_NONE) *(LAS unsigned short*)(scratch + ((excw >> 16) & 2047u) * 2) = (unsigned short)excw;
        asm volatile("" ::: "memory");
#pragma unroll
        for (int f = 0; f < 4; ++f) fr[f] = *(LAS u32x4*)(scratch + f * 1024 + lane * 16);
        asm volatile("" : "+v"(fr[0]), "+v"(fr[1]), "+v"(fr[2]), "+v"(fr[3]));
    }
}

__global__ void __launch_bounds__(NTHR, 2) fused(Params P) {
    extern __shared__ __attribute__((aligned(16))) unsigned char smem[];
    lu8* lds = (lu8*)smem;
    const int tid = threadIdx.x, lane = tid & 63, wv = __builtin_amdgcn_readfirstlane(tid >> 6);
    const int g = blockIdx.x, xcd = g & 7, idx = g >> 3, u = 2 * (idx >> 1) + (xcd >> 2), v = 2 * (xcd & 3) + (idx & 1);
    const int rank = P.rank;
    gu8* myws = (gu8*)P.ws[rank];
    const u64 tstart = rt();
    STAMP(0);
    gu32* tmo = (gu32*)(myws + WS_TMO);
    gu32* cntp = (gu32*)(myws + WS_CNT + g * 128);
    u32 call0 = 0;
    if (tid == 0) call0 = __hip_atomic_load(cntp, __ATOMIC_RELAXED, __HIP_MEMORY_SCOPE_AGENT);
    const int xi = v >> 2, xB = v & 3;
    const int wV = g >> 3, wb = g & 7;
    const int r = lane & 31, h = lane >> 5;
    {
        u32x4 frag;
        if (wv < 4) {
            frag = *(const GAS u32x4*)((const gu16*)P.x + (size_t)(1024 * xi + 32 * u + r) * KS + 64 * xB + 16 * wv + 8 * h);
        } else {
            const int f = wv - 4;
            const gu16* src = (const gu16*)P.w + (size_t)(32 * wb + 16 * (f >> 1) + 8 * h) * N_ + 64 * wV + 32 * (f & 1) + r;
            u32 e[8];
#pragma unroll
            for (int j = 0; j < 8; ++j) e[j] = src[(size_t)j * N_];
            frag = (u32x4){e[0] | (e[1] << 16), e[2] | (e[3] << 16), e[4] | (e[5] << 16), e[6] | (e[7] << 16)};
        }
        if (tid < 256) {
            u32 s0 = 0, s1 = 0, c = 0;
#pragma unroll
            for (int j = 0; j < 8; ++j) { const u32 b = (tid >> j) & 1; const u32 sv = b ? c : 0x0Cu; c += b; if (j < 4) s0 |= sv << (8 * j); else s1 |= sv << (8 * (j - 4)); }
            *(LAS u32x2*)(lds + L_LUT + tid * 8) = (u32x2){s0, s1};
        }
        if (wv < 4) pack_frag(frag, wv, lane, lds + L_XP, lds + L_XR, lds + L_XH);
        else pack_frag(frag, wv - 4, lane, lds + L_WP, lds + L_WR, lds + L_WH);
    }
    if (tid == 0) { call0 += 1u; *(LAS u32*)(lds + L_MISC) = call0; __hip_atomic_store(cntp, call0, __ATOMIC_RELAXED, __HIP_MEMORY_SCOPE_AGENT); *(LAS u32*)(lds + L_MISC + 80) = 0u; *(LAS u32*)(lds + L_MISC + 84) = 0u; }
    STAMP(1);
    __syncthreads();
    const u32 call = rfl(*(LAS u32*)(lds + L_MISC));
    const u64 fv = (u64)call << 32;
    const int wa = (wv - rank) & 7;
    if (wa == 1 || wa == 2) {
        const int q = (wV >> 3) + 4 * (wa - 1), vv = wV & 7;
        gu8* dws = (gu8*)P.ws[q];
        u32x4 d[4]; u32 fl, excw;
        load_block(d, fl, excw, lane, lds + L_WP, lds + L_WR, lds + L_WH);
        if (q != rank) wait_until(tstart + (u64)(P.tW0 + ((((g & 63) * 4 + (g >> 6)) * P.dW) >> 8)));
        send_block(d, fl & F_RAW, fl & F_EXC, excw, lane, dws + WS_WDATA + rank * WSRC + (size_t)(vv * 8 + wb) * SLOT, dws + WS_WEXC + (rank * 8 + vv) * 256 + wb * 32);
        STAMP(2);
        drain();
        if (lane == 0) flag_store(dws + WS_WFLAG + (rank * 8 + vv) * 128 + wb * 8, fv | fl);
        STAMP(3);
    } else if (wa == 3) {
        if (xi == (rank >> 2)) {
            u32x4 d[4]; u32 fl, excw;
            load_block(d, fl, excw, lane, lds + L_XP, lds + L_XR, lds + L_XH);
            send_block(d, fl & F_RAW, fl & F_EXC, excw, lane, myws + WS_XDATA + rank * XSRC + (size_t)(u * 4 + xB) * SLOT, myws + WS_XEXC + (rank * 32 + u) * 128 + xB * 32);
            drain();
            if (lane == 0) flag_store(myws + WS_XFLAG + (rank * 32 + u) * 128 + xB * 8, fv | fl);
        }
    }
    bool ok = true; u64 fvalW, fvalX;
    const int p = wv;
    const bool self = wa == 0;
    lu8* scratch = lds + L_RED + p * RED_WAVE;
    const __amdgpu_buffer_rsrc_t rsW = __builtin_amdgcn_make_buffer_rsrc((void*)(myws + WS_WDATA + p * WSRC + (size_t)v * 8 * SLOT), 0, 8 * SLOT, 0x00020000);
    const __amdgpu_buffer_rsrc_t rsX = __builtin_amdgcn_make_buffer_rsrc((void*)(myws + WS_XDATA + p * XSRC + (size_t)u * 4 * SLOT), 0, 4 * SLOT, 0x00020000);
    const int tau = P.tX0 + (((u * 8 + xB * 2 + xi) * P.dX) >> 8);
    const bool early = tau <= P.tSelf;
    u32x4 xd[4]; u32 xfl = 0, xexc = 0;
    auto xstores = [&]() {
        load_block(xd, xfl, xexc, lane, lds + L_XP, lds + L_XR, lds + L_XH);
        wait_until(tstart + (u64)tau);
        __builtin_amdgcn_s_setprio(3);
#pragma unroll
        for (int k = 0; k < 4; ++k) {
            const int q = 4 * xi + ((k + g) & 3);
            gu8* dws = (gu8*)P.ws[q];
            if (q != rank) send_block(xd, xfl & F_RAW, xfl & F_EXC, xexc, lane, dws + WS_XDATA + rank * XSRC + (size_t)(u * 4 + xB) * SLOT, dws + WS_XEXC + (rank * 32 + u) * 128 + xB * 32);
        }
        __builtin_amdgcn_s_setprio(0);
        STAMP(2);
    };
    auto xflags = [&]() {
        unsigned char* const q0 = P.ws[4 * xi], * const q1 = P.ws[4 * xi + 1], * const q2 = P.ws[4 * xi + 2], * const q3 = P.ws[4 * xi + 3];
        gu8* fq = (gu8*)(lane == 0 ? q0 : lane == 1 ? q1 : lane == 2 ? q2 : q3);
        if (lane < 4 && 4 * xi + lane != rank) flag_store(fq + WS_XFLAG + (rank * 32 + u) * 128 + xB * 8, fv | xfl);
        STAMP(3);
    };
    if (self) {
        u64 fv2;
        gu8* fp = (lane & 8) ? myws + WS_XFLAG + (p * 32 + u) * 128 + (lane & 3) * 8 : myws + WS_WFLAG + (p * 8 + v) * 128 + (lane & 7) * 8;
        ok &= wait_flags(fp, call, tmo, fv2);
        fvalW = fv2; fvalX = (u64)(u32)__builtin_amdgcn_ds_bpermute((8 + (lane & 3)) * 4, (int)(u32)fv2);
        if (early) xstores();
    } else {
        const int tA = P.tW0 + (((32 * v + 31) * P.dW) >> 8) + P.tPoll;
        wait_until(tstart + (u64)(tA + P.tWd <= tau ? tA : max(tA, tau + P.tWj)));
        ok &= wait_flags(myws + WS_WFLAG + (p * 8 + v) * 128 + (lane & 7) * 8, call, tmo, fvalW);
    }
    STAMP(4);
    const int rot = ((u >> 1) + 2 * (v & 1)) & 3;
    u32x4 wf[8][4];
    u32x4 rawX[4][3]; u32 excX, tm;
    auto xloads = [&]() {
        excX = __hip_atomic_load((gu32*)(myws + WS_XEXC + (p * 32 + u) * 128 + (lane & 31) * 4), __ATOMIC_RELAXED, __HIP_MEMORY_SCOPE_SYSTEM);
        tm = __hip_atomic_load(tmo, __ATOMIC_RELAXED, __HIP_MEMORY_SCOPE_AGENT);
#pragma unroll
        for (int b = 0; b < 4; ++b)
#pragma unroll
            for (int k = 0; k < 3; ++k) rawX[b][k] = __builtin_amdgcn_raw_buffer_load_b128(rsX, lane * 16, ((b + rot) & 3) * SLOT + k * 1024, 17);
        __builtin_amdgcn_sched_barrier(0);
    };
    gu8* const xfp = myws + WS_XFLAG + (p * 32 + u) * 128 + (lane & 3) * 8;
#define SB() __builtin_amdgcn_sched_barrier(0)
    {
        const u32 excW = __hip_atomic_load((gu32*)(myws + WS_WEXC + (p * 8 + v) * 256 + lane * 4), __ATOMIC_RELAXED, __HIP_MEMORY_SCOPE_SYSTEM);
        u32x4 rawW[8][3];
        auto wl = [&](int b) {
            const int so = ((b + 2 * rot) & 7) * SLOT;
#pragma unroll
            for (int k = 0; k < 3; ++k) rawW[b][k] = __builtin_amdgcn_raw_buffer_load_b128(rsW, lane * 16, so + k * 1024, 17);
        };
        auto wu = [&](int b) {
            const int bm = (b + 2 * rot) & 7;
            unpack_block(wf[b], rawW[b][0], rawW[b][1], rawW[b][2], (u32)__builtin_amdgcn_readlane((int)(u32)fvalW, bm), excW, bm, lane, scratch, rsW, bm * SLOT);
        };
        bool xl = false; u64 fx = 0;
        auto xlook = [&]() { if (!xl) fx = __hip_atomic_load((gu64*)xfp, __ATOMIC_RELAXED, __HIP_MEMORY_SCOPE_SYSTEM); };
        auto xtry = [&]() { if (!xl && __all((u32)(fx >> 32) >= call)) { asm volatile("" ::: "memory"); fvalX = fx; xloads(); xl = true; STAMP(6); } };
        wl(0); wl(1); wl(2); SB();
        if (self && early) { asm volatile("s_waitcnt vmcnt(9)" ::: "memory"); xflags(); }
        wu(0); SB(); wl(3); SB();
        wu(1); SB(); wl(4); SB();
        wu(2); SB(); wl(5); SB();
        wu(3); SB(); wl(6); SB();
        if (self) fx = (u64)call << 32 | (u32)fvalX; else xlook();
        SB(); wu(4); SB(); wl(7); SB();
        xtry(); SB();
        wu(5); SB(); xlook(); SB();
        wu(6); SB(); xtry(); SB();
        wu(7);
        STAMP(5);
        if (!xl) {
            wait_until(tstart + (u64)(P.tX0 + (((u * 8 + 7) * P.dX) >> 8) + P.tPoll));
            ok &= wait_flags(xfp, call, tmo, fvalX);
            STAMP(6);
            xloads();
        }
    }
    f32x16 acc[2];
#pragma unroll
    for (int i = 0; i < 16; ++i) { acc[0][i] = 0.f; acc[1][i] = 0.f; }
#pragma unroll
    for (int B = 0; B < 4; ++B) {
        u32x4 xf[4];
        const int bm = (B + rot) & 3;
        unpack_block(xf, rawX[B][0], rawX[B][1], rawX[B][2], (u32)__builtin_amdgcn_readlane((int)(u32)fvalX, bm), lane < 32 ? excX : EXC_NONE, bm, lane, scratch, rsX, bm * SLOT);
#pragma unroll
        for (int f = 0; f < 4; ++f) {
            const int T = 4 * B + f;
            acc[0] = __builtin_amdgcn_mfma_f32_32x32x16_bf16(__builtin_bit_cast(bf16x8, wf[T >> 1][2 * (T & 1)]), __builtin_bit_cast(bf16x8, xf[f]), acc[0], 0, 0, 0);
            acc[1] = __builtin_amdgcn_mfma_f32_32x32x16_bf16(__builtin_bit_cast(bf16x8, wf[T >> 1][2 * (T & 1) + 1]), __builtin_bit_cast(bf16x8, xf[f]), acc[1], 0, 0, 0);
        }
    }
    if ((!ok || tm == call) && lane == 0) *(LAS u32*)(lds + L_MISC + 84) = 1u;
#pragma unroll
    for (int cb = 0; cb < 2; ++cb)
#pragma unroll
        for (int g4 = 0; g4 < 4; ++g4)
            *(LAS f32x4*)(lds + L_RED + p * RED_WAVE + r * RED_STRIDE + (32 * cb + 8 * g4 + 4 * h) * 4) =
                (f32x4){acc[cb][4 * g4], acc[cb][4 * g4 + 1], acc[cb][4 * g4 + 2], acc[cb][4 * g4 + 3]};
    if (self && !early) { xstores(); drain(); xflags(); }
    if (tid < 198) *(LAS u32x4*)(lds + L_IMG + 32 + tid * 16) = (u32x4){0, 0, 0, 0};
    if (tid < 8) *(LAS u32*)(lds + L_IMG + tid * 4) = EXC_NONE;
    STAMP(7);
    __syncthreads();
    STAMP(13);
    u32 o[4]; u32 pre, nzbits;
    {
        const int m = tid >> 4, n0 = 4 * (tid & 15);
        const bool bad = *(LAS u32*)(lds + L_MISC + 84) != 0u;
        f32x4 s = *(LAS f32x4*)(lds + L_RED + m * RED_STRIDE + n0 * 4);
#pragma unroll
        for (int pp = 1; pp < 8; ++pp) s += *(LAS f32x4*)(lds + L_RED + pp * RED_WAVE + m * RED_STRIDE + n0 * 4);
        u32 mx = 0;
#pragma unroll
        for (int j = 0; j < 4; ++j) {
            const float y = s[j] < 0.f ? 0.f : s[j];
            u32 b = __float_as_uint(y); b = ((b + 0x7FFFu + ((b >> 16) & 1u)) >> 16) & 0x7FFFu;
            if (y != y || bad) b = 0x7FC0u;
            o[j] = b; mx = max(mx, b);
        }
        *(LAS u32x2*)(lds + L_OUT + tid * 8) = (u32x2){o[0] | (o[1] << 16), o[2] | (o[3] << 16)};
        const u32 wmx = wave_max(mx);
        u32 cnt = 0; pre = 0; nzbits = 0;
#pragma unroll
        for (int j = 0; j < 4; ++j) { const bool nz = o[j] != 0u; const u64 b = __ballot(nz); pre = mbcnt64(b, pre); cnt += (u32)__popcll(b); nzbits |= nz ? (1u << j) : 0u; }
        if (lane == 0) { *(LAS u32*)(lds + L_MISC + 16 + wv * 4) = cnt; *(LAS u32*)(lds + L_MISC + 48 + wv * 4) = wmx; }
    }
    STAMP(14);
    __syncthreads();
    STAMP(15);
    u32 ntot, obase;
    {
        const u32x4 c0 = *(LAS u32x4*)(lds + L_MISC + 16), c1 = *(LAS u32x4*)(lds + L_MISC + 32);
        const u32x4 m0 = *(LAS u32x4*)(lds + L_MISC + 48), m1 = *(LAS u32x4*)(lds + L_MISC + 64);
        u32 wbase = 0;
        ntot = 0;
#pragma unroll
        for (int k = 0; k < 4; ++k) { if (k < wv) wbase += c0[k]; ntot += c0[k]; }
#pragma unroll
        for (int k = 0; k < 4; ++k) { if (k + 4 < wv) wbase += c1[k]; ntot += c1[k]; }
        ntot = rfl(ntot);
        const u32 mx = max(max(max(m0.x, m0.y), max(m0.z, m0.w)), max(max(m1.x, m1.y), max(m1.z, m1.w)));
        const u32 h7 = rfl(mx) >> 8; obase = h7 > 7u ? h7 - 7u : 0u;
        u32 pos = wbase + pre;
        const u32 coff = 288u + ((ntot + 15u) & ~15u);
        u32 cw = 0, k = 0;
#pragma unroll
        for (int j = 0; j < 4; ++j) {
            if (o[j] != 0u) {
                *(LAS unsigned char*)(lds + L_IMG + 288 + pos + k) = (unsigned char)o[j];
                const u32 hh = o[j] >> 8;
                if (hh < obase) {
                    const u32 e = __hip_atomic_fetch_add((lu32*)(lds + L_MISC + 80), 1u, __ATOMIC_RELAXED, __HIP_MEMORY_SCOPE_WORKGROUP);
                    if (e < 8u) *(LAS u32*)(lds + L_IMG + e * 4) = ((u32)(tid * 4 + j) << 16) | o[j];
                } else cw |= (hh - obase) << (3u * k);
                k++;
            }
        }
        if (cw) {
            const u32 bit = 3u * pos, sh = bit & 31u;
            lu32* cp = (lu32*)(lds + L_IMG + coff + (bit >> 5) * 4);
            __hip_atomic_fetch_or(cp, cw << sh, __ATOMIC_RELAXED, __HIP_MEMORY_SCOPE_WORKGROUP);
            if (sh > 20u) { const u32 hi = cw >> (32u - sh); if (hi) __hip_atomic_fetch_or(cp + 1, hi, __ATOMIC_RELAXED, __HIP_MEMORY_SCOPE_WORKGROUP); }
        }
        const u32 other = (u32)__builtin_amdgcn_update_dpp(0, (int)nzbits, 0xB1  , 0xf, 0xf, false);
        if (!(lane & 1)) *(LAS unsigned char*)(lds + L_IMG + 32 + (tid >> 1)) = (unsigned char)(nzbits | (other << 4));
    }
    STAMP(8);
    __syncthreads();
    const int tile = u * 8 + v;
    const size_t PASS = (size_t)8 * N_ * 2;
    if (wa == 0) {
        u32x4 d[4];
#pragma unroll
        for (int s = 0; s < 4; ++s) d[s] = *(LAS u32x4*)(lds + L_OUT + s * 1024 + lane * 16);
        const int row0 = 1024 * (rank >> 2) + 32 * u, col0 = 512 * (rank & 3) + 64 * v;
        gu8* dst = (gu8*)P.out[rank] + ((size_t)(row0 + (lane >> 3)) * N_ + col0) * 2 + (lane & 7) * 16;
#pragma unroll
        for (int s = 0; s < 4; ++s) st16_sys(dst + s * PASS, d[s]);
    } else {
        const u32 nexc = rfl(*(LAS u32*)(lds + L_MISC + 80));
        const bool raw = nexc > 8u;
        const u32 nch = raw ? 256u : (288u + ((ntot + 15u) & ~15u) + (((3u * ntot + 7u) / 8u + 15u) & ~15u)) / 16u;
        lu8* S = lds + (raw ? L_OUT : L_IMG);
        gu8* dws = (gu8*)P.ws[wv];
        gu8* dst = dws + WS_ODATA + rank * OSRC + (size_t)tile * SLOT + lane * 16;
        u32x4 d[4];
#pragma unroll
        for (int s = 0; s < 4; ++s) d[s] = *(LAS u32x4*)(S + s * 1024 + lane * 16);
        wait_until(tstart + (u64)(P.tO0 + (((u * 8 + v) * P.dO) >> 8)));
        if ((raw || nexc || lane >= 2) && (u32)lane < nch) st16_sys(dst, d[0]);
#pragma unroll
        for (int s = 1; s < 4; ++s) if ((u32)(lane + 64 * s) < nch) st16_sys(dst + s * 1024, d[s]);
        STAMP(9);
        drain();
        if (lane == 0) flag_store(dws + WS_OFLAG + (rank * 256 + tile) * 128, fv | (raw ? F_RAW : 0u) | (nexc ? F_EXC : 0u) | (obase << 12) | ntot);
        STAMP(10);
        u64 fval;
        const bool ok2 = wait_flags(myws + WS_OFLAG + (wv * 256 + tile) * 128, call, tmo, fval);
        STAMP(11);
        const u32 fl = rfl((u32)fval);
        const u32 n = fl & 4095u, rb = (fl >> 12) & 127u;
        const bool rraw = (fl & F_RAW) != 0u;
        const u32 rch = rraw ? 256u : (288u + ((n + 15u) & ~15u) + (((3u * n + 7u) / 8u + 15u) & ~15u)) / 16u;
        const __amdgpu_buffer_rsrc_t rs = __builtin_amdgcn_make_buffer_rsrc((void*)(myws + WS_ODATA + wv * OSRC + (size_t)tile * SLOT), 0, rch * 16, 0x00020000);
        u32x4 in[4];
#pragma unroll
        for (int s = 0; s < 4; ++s) in[s] = __builtin_amdgcn_raw_buffer_load_b128(rs, lane * 16 + s * 1024, 0, 17);
        const u32 tm2 = __hip_atomic_load(tmo, __ATOMIC_RELAXED, __HIP_MEMORY_SCOPE_AGENT);
        const int rrow0 = 1024 * (wv >> 2) + 32 * u, rcol0 = 512 * (wv & 3) + 64 * v;
        gu8* odst = (gu8*)P.out[rank] + ((size_t)(rrow0 + (lane >> 3)) * N_ + rcol0) * 2 + (lane & 7) * 16;
        if (!ok2 || tm2 == call) {
#pragma unroll
            for (int s = 0; s < 4; ++s) st16_sys(odst + s * PASS, (u32x4){0x7FC07FC0u, 0x7FC07FC0u, 0x7FC07FC0u, 0x7FC07FC0u});
        } else if (rraw) {
#pragma unroll
            for (int s = 0; s < 4; ++s) st16_sys(odst + s * PASS, in[s]);
        } else {
            lu8* sc = lds + L_RED + wv * RED_WAVE;
#pragma unroll
            for (int s = 0; s < 4; ++s) *(LAS u32x4*)(sc + s * 1024 + lane * 16) = in[s];
            asm volatile("" ::: "memory");
            const u32 coff = 288u + ((n + 15u) & ~15u);
            u32 mb[4];
#pragma unroll
            for (int i = 0; i < 4; ++i) mb[i] = *(LAS unsigned char*)(sc + 32 + 64 * i + lane);
            const u32 c01 = (u32)__popc(mb[0]) | ((u32)__popc(mb[1]) << 16), c23 = (u32)__popc(mb[2]) | ((u32)__popc(mb[3]) << 16);
            const u32 s01 = wave_scan_add(c01), s23 = wave_scan_add(c23);
            const u32 t01 = (u32)__builtin_amdgcn_readlane((int)s01, 63), t23 = (u32)__builtin_amdgcn_readlane((int)s23, 63);
            const u32 e01 = s01 - c01, e23 = s23 - c23;
            u32 off[4];
            off[0] = e01 & 0xFFFFu; off[1] = (t01 & 0xFFFFu) + (e01 >> 16);
            const u32 b2 = (t01 & 0xFFFFu) + (t01 >> 16);
            off[2] = b2 + (e23 & 0xFFFFu); off[3] = b2 + (t23 & 0xFFFFu) + (e23 >> 16);
            const u32 b4 = rb * 0x01010101u;
#pragma unroll
            for (int i = 0; i < 4; ++i) {
                const u32 oo = off[i];
                lu32* lp = (lu32*)(sc + 288 + (oo & ~3u));
                const u32 a0 = lp[0], a1 = lp[1], a2 = lp[2];
                const u32 bit = 3u * oo;
                lu32* cp = (lu32*)(sc + coff + (bit >> 5) * 4);
                const u32 k0 = cp[0], k1 = cp[1];
                const u32x2 sel = *(LAS u32x2*)(lds + L_LUT + mb[i] * 8);
                const u32 lo0 = __builtin_amdgcn_alignbyte(a1, a0, oo & 3u), lo1 = __builtin_amdgcn_alignbyte(a2, a1, oo & 3u);
                const u32 c = __builtin_amdgcn_alignbit(k1, k0, bit & 31u);
                u32 x0 = c & 0xFFFu, x1 = (c >> 12) & 0xFFFu;
                x0 = (x0 | (x0 << 10)) & 0x003F003Fu; x1 = (x1 | (x1 << 10)) & 0x003F003Fu;
                x0 = (x0 | (x0 << 5)) & 0x07070707u; x1 = (x1 | (x1 << 5)) & 0x07070707u;
                const u32 h0 = x0 + b4, h1 = x1 + b4;
                const u32 l0 = perm(lo1, lo0, sel.x), l1 = perm(lo1, lo0, sel.y), g0 = perm(h1, h0, sel.x), g1 = perm(h1, h0, sel.y);
                st16_sys(odst + i * PASS, (u32x4){perm(g0, l0, 0x05010400u), perm(g0, l0, 0x07030602u), perm(g1, l1, 0x05010400u), perm(g1, l1, 0x07030602u)});
            }
            if (fl & F_EXC) {
                drain();
                if (lane < 8) { const u32 e = *(LAS u32*)(sc + lane * 4);
                    if (e != EXC_NONE) { const u32 ix = (e >> 16) & 2047u; *(gu16*)((gu8*)P.out[rank] + ((size_t)(rrow0 + (ix >> 6)) * N_ + rcol0 + (ix & 63u)) * 2) = (unsigned short)e; } }
            }
        }
        STAMP(12);
    }
}

#define T_W0 125
#define D_W 345
#define T_X0 470
#define D_X 690
#define T_O0 1160
#define D_O 780
#define T_POLL 100
#define T_WJ 20
#define T_SELF 800
#define T_WD 350
extern "C" void kernel_launch(int rank, int nranks, ncclComm_t comm, hipStream_t stream,
                              void* const* d_in, const int* in_sizes, int n_in,
                              void* d_out, int out_size, const int* in_dtypes, int out_dtype,
                              void* const* d_out_all, void* const* d_ws_all, size_t ws_bytes) {
    if (nranks != NR || ws_bytes < DIAG_OFF + DIAG_BYTES) { fprintf(stderr, "bad config\n"); return; }
    static bool init = false;
    static int tp[10];
    if (!init) {
        (void)hipFuncSetAttribute((const void*)fused, hipFuncAttributeMaxDynamicSharedMemorySize, LDS_BYTES);
        const int df[10] = {T_X0, D_X, T_O0, D_O, T_W0, D_W, T_POLL, T_WJ, T_SELF, T_WD};
        for (int i = 0; i < 10; ++i) tp[i] = df[i];
        init = true;
    }
    Params P;
    P.x = (const unsigned short*)d_in[0]; P.w = (const unsigned short*)d_in[1];
    for (int i = 0; i < NR; ++i) { P.ws[i] = (unsigned char*)d_ws_all[i]; P.out[i] = (unsigned char*)d_out_all[i]; }
    P.rank = rank; P.tX0 = tp[0]; P.dX = tp[1]; P.tO0 = tp[2]; P.dO = tp[3]; P.tW0 = tp[4]; P.dW = tp[5]; P.tPoll = tp[6]; P.tWj = tp[7]; P.tSelf = tp[8]; P.tWd = tp[9]; P.pad = 0;
    hipLaunchKernelGGL(fused, dim3(NWG), dim3(NTHR), LDS_BYTES, stream, P);
}
```

```cpp
#define PADO
#define FLAG64
#include <hip/hip_runtime.h>
#include <rccl/rccl.h>
#include <cstdio>
#include <cstdlib>

#define GAS __attribute__((address_space(1)))
#define LAS __attribute__((address_space(3)))
typedef unsigned u32;
typedef unsigned long long u64;
typedef u32 u32x4 __attribute__((ext_vector_type(4)));
typedef u32 u32x2 __attribute__((ext_vector_type(2)));
typedef float f32x4 __attribute__((ext_vector_type(4)));
typedef float f32x16 __attribute__((ext_vector_type(16)));
typedef __bf16 bf16x8 __attribute__((ext_vector_type(8)));
typedef __bf16 bf16x2 __attribute__((ext_vector_type(2)));
typedef float f32x2 __attribute__((ext_vector_type(2)));
typedef unsigned short u16x2 __attribute__((ext_vector_type(2)));
typedef GAS u32 gu32; typedef GAS u64 gu64; typedef GAS unsigned char gu8; typedef GAS unsigned short gu16;
typedef LAS unsigned char lu8; typedef LAS u32 lu32;

constexpr int NR = 8, N_ = 2048, KS = 256;
constexpr int NWG = 256, NTHR = 512;
constexpr size_t WS_TMO = 0;
constexpr size_t WS_CNT = 4096;
constexpr size_t WS_XFLAG = WS_CNT + 256 * 128;
constexpr size_t WS_WFLAG = WS_XFLAG + 8 * 32 * 128;
constexpr size_t WS_OFLAG = WS_WFLAG + 8 * 8 * 128;
constexpr size_t WS_XEXC = WS_OFLAG + 8 * 256 * 128;
constexpr size_t WS_WEXC = WS_XEXC + 8 * 32 * 128;
constexpr size_t WS_FEND = WS_WEXC + 8 * 8 * 256;
#define SLOT 4224
#define SKEW 20608
constexpr size_t XSRC = (size_t)32 * 4 * SLOT + SKEW, WSRC = (size_t)8 * 8 * SLOT + SKEW, OSRC = (size_t)256 * SLOT + SKEW;
constexpr size_t WS_XDATA = 1u << 20;
constexpr size_t WS_WDATA = WS_XDATA + 8 * XSRC;
constexpr size_t WS_ODATA = WS_WDATA + 8 * WSRC;
constexpr size_t WS_END = WS_ODATA + 8 * OSRC;
static_assert(SLOT % 128 == 0 && SLOT >= 4096 && SKEW % 128 == 0, "slot");
constexpr size_t DIAG_OFF = 48u << 20;
constexpr size_t DIAG_BYTES = 256 * 8 * 16 * 8;
constexpr size_t DUR_OFF = 50u << 20;
static_assert(WS_FEND <= WS_XDATA, "flags overlap data");
static_assert(WS_END <= DIAG_OFF, "ws overlap");
constexpr u64 DEADLINE = 30000000ull;

struct Params {
    const unsigned short* x; const unsigned short* w;
    unsigned char* ws[NR]; unsigned char* out[NR];
    int rank; int tX0, dX, tO0, dO, tW0, dW;
    int tPoll, tDir, tSelf, tWd, delay, tO1, dO1, pad;
};

__device__ __forceinline__ u64 rt() { return __builtin_amdgcn_s_memrealtime(); }
__device__ __forceinline__ void st16_sys(gu8* p, u32x4 v) { asm volatile("global_store_dwordx4 %0, %1, off sc0 sc1\n\ts_nop 1" :: "v"(p), "v"(v) : "memory"); }
#define OUTST 0
__device__ __forceinline__ void st16_out(gu8* p, u32x4 v) {
    asm volatile("global_store_dwordx4 %0, %1, off sc0 sc1\n\ts_nop 1" :: "v"(p), "v"(v) : "memory");
}
__device__ __forceinline__ void st4_sys(gu8* p, u32 v) { asm volatile("global_store_dword %0, %1, off sc0 sc1" :: "v"(p), "v"(v) : "memory"); }
__device__ __forceinline__ void drain() { asm volatile("s_waitcnt vmcnt(0)" ::: "memory"); }
__device__ __forceinline__ void flag_store(gu8* p, u64 v) { __hip_atomic_store((gu64*)p, v, __ATOMIC_RELAXED, __HIP_MEMORY_SCOPE_SYSTEM); }
__device__ __forceinline__ u32 rfl(u32 v) { return __builtin_amdgcn_readfirstlane(v); }
__device__ __forceinline__ u32 perm(u32 hi, u32 lo, u32 sel) { return __builtin_amdgcn_perm(hi, lo, sel); }
__device__ __forceinline__ u32 rotr(u32 x, u32 s) { return __builtin_amdgcn_alignbit(x, x, s); }
__device__ __forceinline__ u32 mbcnt64(u64 m, u32 acc) { return __builtin_amdgcn_mbcnt_hi((u32)(m >> 32), __builtin_amdgcn_mbcnt_lo((u32)m, acc)); }
template <int CTRL, int RM> __device__ __forceinline__ u32 dpp0(u32 v) { return (u32)__builtin_amdgcn_update_dpp(0, (int)v, CTRL, RM, 0xf, false); }
__device__ __forceinline__ u32 wave_max(u32 v) {
    v = max(v, dpp0<0x111, 0xf>(v)); v = max(v, dpp0<0x112, 0xf>(v)); v = max(v, dpp0<0x114, 0xf>(v)); v = max(v, dpp0<0x118, 0xf>(v));
    v = max(v, dpp0<0x142, 0xa>(v)); v = max(v, dpp0<0x143, 0xc>(v));
    return (u32)__builtin_amdgcn_readlane((int)v, 63);
}
__device__ __forceinline__ u32 wave_scan_add(u32 v) {
    v += dpp0<0x111, 0xf>(v); v += dpp0<0x112, 0xf>(v); v += dpp0<0x114, 0xf>(v); v += dpp0<0x118, 0xf>(v);
    v += dpp0<0x142, 0xa>(v); v += dpp0<0x143, 0xc>(v);
    return v;
}

__device__ __forceinline__ bool wait_flags(gu8* p, u32 call, gu32* tmo, u64& val) {
    const u64 t0 = rt();
    for (u32 it = 1;; ++it) {
        const u64 v = __hip_atomic_load((gu64*)p, __ATOMIC_RELAXED, __HIP_MEMORY_SCOPE_SYSTEM);
        if (__all((u32)(v >> 32) >= call)) { val = v; asm volatile("" ::: "memory"); return true; }
        if ((it & 31u) == 0u) {
            if (__hip_atomic_load(tmo, __ATOMIC_RELAXED, __HIP_MEMORY_SCOPE_AGENT) == call || rt() - t0 > DEADLINE) {
                __hip_atomic_store(tmo, call, __ATOMIC_RELAXED, __HIP_MEMORY_SCOPE_AGENT);
                val = 0; asm volatile("" ::: "memory"); return false;
            }
        }
        __builtin_amdgcn_s_sleep(2);
    }
}
__device__ __forceinline__ void wait_until(u64 t) { while (rt() < t) __builtin_amdgcn_s_sleep(1); }

#define STAMP(i) do { } while (0)

constexpr int RED_STRIDE = 272;
constexpr int RED_WAVE = 32 * RED_STRIDE;
constexpr int L_RED = 0;
constexpr int L_XP = 69632;
constexpr int L_WP = L_XP + 3072;
constexpr int L_XR = L_WP + 3072;
constexpr int L_WR = L_XR + 4096;
constexpr int L_XH = L_WR + 4096;
constexpr int L_WH = L_XH + 64;
constexpr int L_OUT = L_WH + 64;
constexpr int L_IMG = L_OUT + 4096;
constexpr int L_LUT = L_IMG + 3200;
constexpr int L_LUT4 = L_LUT + 2048;
constexpr int L_MISC = L_LUT4 + 64;
constexpr int LDS_BYTES = L_MISC + 128;
static_assert(LDS_BYTES > 81920 && LDS_BYTES <= 163840 && (L_XP % 16) == 0 && (L_IMG % 16) == 0 && (L_LUT % 16) == 0, "lds");
constexpr u32 F_RAW = 1u << 31, F_EXC = 1u << 30, F_DIRECT = 1u << 29;
__device__ __forceinline__ u32 img_coff(u32 n) { return 256u + ((n + 3u) & ~3u); }
__device__ __forceinline__ u32 img_eoff(u32 n) { return img_coff(n) + ((((3u * n + 7u) >> 3) + 3u) & ~3u); }
__device__ __forceinline__ u32 img_chunks(u32 n, bool exc) { return (img_eoff(n) + (exc ? 32u : 0u) + 15u) >> 4; }
constexpr u32 EXC_NONE = 0xFFFFFFFFu;

__device__ __forceinline__ void pack_frag(u32x4 e, int f, int lane, lu8* P, lu8* R, lu8* H) {
    *(LAS u32x4*)(R + f * 1024 + lane * 16) = e;
    const u16x2 m01 = __builtin_elementwise_max(__builtin_bit_cast(u16x2, e.x & 0x7FFF7FFFu), __builtin_bit_cast(u16x2, e.y & 0x7FFF7FFFu));
    const u16x2 m23 = __builtin_elementwise_max(__builtin_bit_cast(u16x2, e.z & 0x7FFF7FFFu), __builtin_bit_cast(u16x2, e.w & 0x7FFF7FFFu));
    const u16x2 mm = __builtin_elementwise_max(m01, m23);
    const u32 mx = wave_max(max((u32)mm.x, (u32)mm.y));
    const u32 h7 = mx >> 8, base = h7 > 7u ? h7 - 7u : 0u;
    const u32 lo0 = perm(e.y, e.x, 0x06040200u), lo1 = perm(e.w, e.z, 0x06040200u);
    const u32 hi0 = perm(e.y, e.x, 0x07050301u), hi1 = perm(e.w, e.z, 0x07050301u);
    const u32 K = (128u - base) * 0x01010101u;
    const u32 d0 = (hi0 & 0x7F7F7F7Fu) + K, d1 = (hi1 & 0x7F7F7F7Fu) + K;
    const u32 cA = (hi0 & 0x80808080u) | (d0 & 0x07070707u), cB = (hi1 & 0x80808080u) | (d1 & 0x07070707u);
    const u32 nw = cA | rotr(cB, 28);
    *(LAS u32x2*)(P + (f >> 1) * 1024 + lane * 16 + (f & 1) * 8) = (u32x2){lo0, lo1};
    *(LAS u32*)(P + 2048 + lane * 16 + f * 4) = nw;
    const u32 x0 = ~d0 & 0x80808080u, x1 = ~d1 & 0x80808080u;
    u32 hdr = base;
    if (lane < 2) *(LAS u32*)(H + 16 + f * 8 + lane * 4) = EXC_NONE;
    asm volatile("" ::: "memory");
    if (__any((x0 | x1) != 0u)) {
        u32 cnt = 0;
#pragma unroll
        for (int j = 0; j < 8; ++j) {
            const bool ex = (((j < 4 ? x0 : x1) >> (8 * (j & 3) + 7)) & 1u) != 0u;
            const u64 b = __ballot(ex);
            const u32 rk = mbcnt64(b, cnt);
            const u32 val = (e[j >> 1] >> (16 * (j & 1))) & 0xFFFFu;
            if (ex && rk < 2u) *(LAS u32*)(H + 16 + f * 8 + rk * 4) = ((u32)(f * 512 + lane * 8 + j) << 16) | val;
            cnt += (u32)__popcll(b);
        }
        hdr |= cnt > 2u ? 0x300u : 0x100u;
    }
    if (lane == 0) *(LAS u32*)(H + f * 4) = hdr;
}

__device__ __forceinline__ void send_block(const u32x4 (&d)[4], bool raw, bool exc, u32 excw, int lane, gu8* dst, gu8* dexc) {
    st16_sys(dst + lane * 16, d[0]); st16_sys(dst + 1024 + lane * 16, d[1]); st16_sys(dst + 2048 + lane * 16, d[2]);
    if (raw) st16_sys(dst + 3072 + lane * 16, d[3]);
    if (exc && lane < 8) st4_sys(dexc + lane * 4, excw);
}
__device__ __forceinline__ void load_block(u32x4 (&d)[4], u32& fl, u32& excw, int lane, lu8* P, lu8* R, lu8* H) {
    const u32x4 h = *(LAS u32x4*)H;
    const u32 any = rfl(h.x | h.y | h.z | h.w);
    const bool raw = (any & 0x200u) != 0u;
    fl = (rfl(h.x) & 127u) | ((rfl(h.y) & 127u) << 7) | ((rfl(h.z) & 127u) << 14) | ((rfl(h.w) & 127u) << 21) | (raw ? F_RAW : ((any & 0x100u) ? F_EXC : 0u));
    lu8* S = raw ? R : P;
    d[0] = *(LAS u32x4*)(S + lane * 16); d[1] = *(LAS u32x4*)(S + 1024 + lane * 16); d[2] = *(LAS u32x4*)(S + 2048 + lane * 16);
    d[3] = *(LAS u32x4*)(R + 3072 + lane * 16);
    excw = *(LAS u32*)(H + 16 + (lane & 7) * 4);
}

__device__ __forceinline__ u32x4 unpack_frag(u32 lo0, u32 lo1, u32 nw, u32 b4) {
    const u32 hA = (nw & 0x87878787u) + b4, hB = (rotr(nw, 4) & 0x87878787u) + b4;
    return (u32x4){perm(hA, lo0, 0x05010400u), perm(hA, lo0, 0x07030602u), perm(hB, lo1, 0x05010400u), perm(hB, lo1, 0x07030602u)};
}
__device__ __forceinline__ void unpack_block(u32x4 (&fr)[4], u32x4 L0, u32x4 L1, u32x4 Nw, u32 fl, u32 excw, int blk, int lane, lu8* scratch,
                                             __amdgpu_buffer_rsrc_t rs, int soff) {
    if (fl & F_RAW) {
        fr[0] = L0; fr[1] = L1; fr[2] = Nw;
        fr[3] = __builtin_amdgcn_raw_buffer_load_b128(rs, lane * 16, soff + 3072, 17);
        asm volatile("" : "+v"(fr[3]));
        return;
    }
    fr[0] = unpack_frag(L0.x, L0.y, Nw.x, (fl & 127u) * 0x01010101u);
    fr[1] = unpack_frag(L0.z, L0.w, Nw.y, ((fl >> 7) & 127u) * 0x01010101u);
    fr[2] = unpack_frag(L1.x, L1.y, Nw.z, ((fl >> 14) & 127u) * 0x01010101u);
    fr[3] = unpack_frag(L1.z, L1.w, Nw.w, ((fl >> 21) & 127u) * 0x01010101u);
    if (fl & F_EXC) {
#pragma unroll
        for (int f = 0; f < 4; ++f) *(LAS u32x4*)(scratch + f * 1024 + lane * 16) = fr[f];
        asm volatile("" ::: "memory");
        if ((lane >> 3) == blk && excw != EXC_NONE) *(LAS unsigned short*)(scratch + ((excw >> 16) & 2047u) * 2) = (unsigned short)excw;
        asm volatile("" ::: "memory");
#pragma unroll
        for (int f = 0; f < 4; ++f) fr[f] = *(LAS u32x4*)(scratch + f * 1024 + lane * 16);
        asm volatile("" : "+v"(fr[0]), "+v"(fr[1]), "+v"(fr[2]), "+v"(fr[3]));
    }
}

__global__ void __launch_bounds__(NTHR, 2) fused(Params P) {
    extern __shared__ __attribute__((aligned(16))) unsigned char smem[];
    lu8* lds = (lu8*)smem;
    const int tid = threadIdx.x, lane = tid & 63, wv = __builtin_amdgcn_readfirstlane(tid >> 6);
    const int g = blockIdx.x, xcd = g & 7, idx = g >> 3, u = 2 * (idx >> 1) + (xcd >> 2), v = 2 * (xcd & 3) + (idx & 1);
    const int rank = P.rank;
    gu8* myws = (gu8*)P.ws[rank];
    if (P.delay) wait_until(rt() + (u64)P.delay);
    const u64 tstart = rt();
    STAMP(0);
    gu32* tmo = (gu32*)(myws + WS_TMO);
    gu32* cntp = (gu32*)(myws + WS_CNT + g * 128);
    u32 call0 = 0;
    if (tid == 0) call0 = __hip_atomic_load(cntp, __ATOMIC_RELAXED, __HIP_MEMORY_SCOPE_AGENT);
    const int xi = v >> 2, xB = v & 3;
    const int wV = g >> 3, wb = g & 7;
    const int r = lane & 31, h = lane >> 5;
    {
        u32x4 frag;
        if (wv < 4) {
            frag = *(const GAS u32x4*)((const gu16*)P.x + (size_t)(1024 * xi + 32 * u + r) * KS + 64 * xB + 16 * wv + 8 * h);
        } else {
            const int f = wv - 4;
            const gu16* src = (const gu16*)P.w + (size_t)(32 * wb + 16 * (f >> 1) + 8 * h) * N_ + 64 * wV + 32 * (f & 1) + r;
            u32 e[8];
#pragma unroll
            for (int j = 0; j < 8; ++j) e[j] = src[(size_t)j * N_];
            frag = (u32x4){e[0] | (e[1] << 16), e[2] | (e[3] << 16), e[4] | (e[5] << 16), e[6] | (e[7] << 16)};
        }
        if (wv < 4) pack_frag(frag, wv, lane, lds + L_XP, lds + L_XR, lds + L_XH);
        else pack_frag(frag, wv - 4, lane, lds + L_WP, lds + L_WR, lds + L_WH);
    }
    if (tid == 0) { call0 += 1u; *(LAS u32*)(lds + L_MISC) = call0; __hip_atomic_store(cntp, call0, __ATOMIC_RELAXED, __HIP_MEMORY_SCOPE_AGENT); }
    if (tid >= 64 && tid < 64 + 12) *(LAS u32*)(lds + L_MISC + 48 + (tid - 64) * 4) = 0u;
    STAMP(1);
    __syncthreads();
    const u32 call = rfl(*(LAS u32*)(lds + L_MISC));
    const u64 fv = (u64)call << 32;
    const int wa = (wv - rank) & 7;
    if (wa == 1 || wa == 2) {
        const int q = (wV >> 3) + 4 * (wa - 1), vv = wV & 7;
        gu8* dws = (gu8*)P.ws[q];
        u32x4 d[4]; u32 fl, excw;
        load_block(d, fl, excw, lane, lds + L_WP, lds + L_WR, lds + L_WH);
        if (q != rank) wait_until(tstart + (u64)(P.tW0 + ((((g & 63) * 4 + (g >> 6)) * P.dW) >> 8)));
        send_block(d, fl & F_RAW, fl & F_EXC, excw, lane, dws + WS_WDATA + rank * WSRC + (size_t)(vv * 8 + wb) * SLOT, dws + WS_WEXC + (rank * 8 + vv) * 256 + wb * 32);
        STAMP(2);
        drain();
        if (lane == 0) flag_store(dws + WS_WFLAG + (rank * 8 + vv) * 128 + wb * 8, fv | fl);
        STAMP(3);
    } else if (wa >= 4) {
        const u32 m = (u32)(wa - 4) * 64u + (u32)lane;
        u32 s0 = 0, s1 = 0, c = 0;
#pragma unroll
        for (int j = 0; j < 8; ++j) { const u32 b = (m >> j) & 1u; const u32 sv = b ? c : 0x0Cu; c += b; if (j < 4) s0 |= sv << (8 * j); else s1 |= sv << (8 * (j - 4)); }
        *(LAS u32x2*)(lds + L_LUT + m * 8) = (u32x2){s0, s1};
        if (m < 16u) {
            u32 sel = 0x0C0C0C0Cu, k = 0;
#pragma unroll
            for (int j = 0; j < 4; ++j) if ((m >> j) & 1u) { sel = (sel & ~(0xFFu << (8 * k))) | ((u32)j << (8 * k)); k++; }
            *(LAS u32*)(lds + L_LUT4 + m * 4) = sel;
        }
    } else if (wa == 3) {
        if (xi == (rank >> 2)) {
            u32x4 d[4]; u32 fl, excw;
            load_block(d, fl, excw, lane, lds + L_XP, lds + L_XR, lds + L_XH);
            send_block(d, fl & F_RAW, fl & F_EXC, excw, lane, myws + WS_XDATA + rank * XSRC + (size_t)(u * 4 + xB) * SLOT, myws + WS_XEXC + (rank * 32 + u) * 128 + xB * 32);
            drain();
            if (lane == 0) flag_store(myws + WS_XFLAG + (rank * 32 + u) * 128 + xB * 8, fv | fl);
        }
    }
    bool ok = true; u64 fvalW, fvalX;
    const int p = wv;
    const bool self = wa == 0;
    lu8* scratch = lds + L_RED + p * RED_WAVE;
    const __amdgpu_buffer_rsrc_t rsW = __builtin_amdgcn_make_buffer_rsrc((void*)(myws + WS_WDATA + p * WSRC + (size_t)v * 8 * SLOT), 0, 8 * SLOT, 0x00020000);
    const __amdgpu_buffer_rsrc_t rsX = __builtin_amdgcn_make_buffer_rsrc((void*)(myws + WS_XDATA + p * XSRC + (size_t)u * 4 * SLOT), 0, 4 * SLOT, 0x00020000);
    const int tau = P.tX0 + (((u * 8 + xB * 2 + xi) * P.dX) >> 8);
    const bool early = tau <= P.tSelf;
    u32x4 xd[4]; u32 xfl = 0, xexc = 0;
    auto xstores = [&]() {
        load_block(xd, xfl, xexc, lane, lds + L_XP, lds + L_XR, lds + L_XH);
        wait_until(tstart + (u64)tau);
        __builtin_amdgcn_s_setprio(3);
#pragma unroll
        for (int k = 0; k < 4; ++k) {
            const int q = 4 * xi + ((k + g) & 3);
            gu8* dws = (gu8*)P.ws[q];
            if (q != rank) send_block(xd, xfl & F_RAW, xfl & F_EXC, xexc, lane, dws + WS_XDATA + rank * XSRC + (size_t)(u * 4 + xB) * SLOT, dws + WS_XEXC + (rank * 32 + u) * 128 + xB * 32);
        }
        __builtin_amdgcn_s_setprio(0);
        __hip_atomic_store((lu32*)(lds + L_MISC + 88), 1u, __ATOMIC_RELAXED, __HIP_MEMORY_SCOPE_WORKGROUP);
        STAMP(2);
    };
    auto xflags = [&]() {
        unsigned char* const q0 = P.ws[4 * xi], * const q1 = P.ws[4 * xi + 1], * const q2 = P.ws[4 * xi + 2], * const q3 = P.ws[4 * xi + 3];
        gu8* fq = (gu8*)(lane == 0 ? q0 : lane == 1 ? q1 : lane == 2 ? q2 : q3);
        if (lane < 4 && 4 * xi + lane != rank) flag_store(fq + WS_XFLAG + (rank * 32 + u) * 128 + xB * 8, fv | xfl);
        STAMP(3);
    };
    if (self) {
        u64 fv2;
        gu8* fp = (lane & 8) ? myws + WS_XFLAG + (p * 32 + u) * 128 + (lane & 3) * 8 : myws + WS_WFLAG + (p * 8 + v) * 128 + (lane & 7) * 8;
        ok &= wait_flags(fp, call, tmo, fv2);
        fvalW = fv2; fvalX = (u64)(u32)__builtin_amdgcn_ds_bpermute((8 + (lane & 3)) * 4, (int)(u32)fv2);
        if (early) xstores();
    } else {
        const int tA = P.tW0 + (((32 * v + 31) * P.dW) >> 8) + P.tPoll;
        wait_until(tstart + (u64)tA);
        ok &= wait_flags(myws + WS_WFLAG + (p * 8 + v) * 128 + (lane & 7) * 8, call, tmo, fvalW);
        if (tA + P.tWd > tau) {
            const u64 t0 = rt();
            while (__hip_atomic_load((lu32*)(lds + L_MISC + 88), __ATOMIC_RELAXED, __HIP_MEMORY_SCOPE_WORKGROUP) == 0u && rt() - t0 < DEADLINE) __builtin_amdgcn_s_sleep(1);
        }
    }
    STAMP(4);
    const int rot = ((u >> 1) + 2 * (v & 1)) & 3;
    u32x4 wf[8][4];
    u32x4 rawX[4][3]; u32 excX, tm;
    auto xloads = [&]() {
        excX = __hip_atomic_load((gu32*)(myws + WS_XEXC + (p * 32 + u) * 128 + (lane & 31) * 4), __ATOMIC_RELAXED, __HIP_MEMORY_SCOPE_SYSTEM);
        tm = __hip_atomic_load(tmo, __ATOMIC_RELAXED, __HIP_MEMORY_SCOPE_AGENT);
#pragma unroll
        for (int b = 0; b < 4; ++b)
#pragma unroll
            for (int k = 0; k < 3; ++k) rawX[b][k] = __builtin_amdgcn_raw_buffer_load_b128(rsX, lane * 16, ((b + rot) & 3) * SLOT + k * 1024, 17);
        __builtin_amdgcn_sched_barrier(0);
    };
    gu8* const xfp = myws + WS_XFLAG + (p * 32 + u) * 128 + (lane & 3) * 8;
#define SB() __builtin_amdgcn_sched_barrier(0)
    {
        const u32 excW = __hip_atomic_load((gu32*)(myws + WS_WEXC + (p * 8 + v) * 256 + lane * 4), __ATOMIC_RELAXED, __HIP_MEMORY_SCOPE_SYSTEM);
        u32x4 rawW[8][3];
        auto wl = [&](int b) {
            const int so = ((b + 2 * rot) & 7) * SLOT;
#pragma unroll
            for (int k = 0; k < 3; ++k) rawW[b][k] = __builtin_amdgcn_raw_buffer_load_b128(rsW, lane * 16, so + k * 1024, 17);
        };
        auto wu = [&](int b) {
            const int bm = (b + 2 * rot) & 7;
            unpack_block(wf[b], rawW[b][0], rawW[b][1], rawW[b][2], (u32)__builtin_amdgcn_readlane((int)(u32)fvalW, bm), excW, bm, lane, scratch, rsW, bm * SLOT);
        };
        bool xl = false; u64 fx = 0;
        auto xlook = [&]() { if (!xl) fx = __hip_atomic_load((gu64*)xfp, __ATOMIC_RELAXED, __HIP_MEMORY_SCOPE_SYSTEM); };
        auto xtry = [&]() { if (!xl && __all((u32)(fx >> 32) >= call)) { asm volatile("" ::: "memory"); fvalX = fx; xloads(); xl = true; STAMP(6); } };
        wl(0); wl(1); wl(2); SB();
        if (self && early) { asm volatile("s_waitcnt vmcnt(9)" ::: "memory"); xflags(); }
        wu(0); SB(); wl(3); SB();
        wu(1); SB(); wl(4); SB();
        wu(2); SB(); wl(5); SB();
        wu(3); SB(); wl(6); SB();
        if (self) fx = (u64)call << 32 | (u32)fvalX; else xlook();
        SB(); wu(4); SB(); wl(7); SB();
        xtry(); SB();
        wu(5); SB(); xlook(); SB();
        wu(6); SB(); xtry(); SB();
        wu(7);
        STAMP(5);
        if (!xl) {
            wait_until(tstart + (u64)(P.tX0 + (((u * 8 + 7) * P.dX) >> 8) + P.tPoll));
            ok &= wait_flags(xfp, call, tmo, fvalX);
            STAMP(6);
            xloads();
        }
    }
    f32x16 acc[2];
#pragma unroll
    for (int i = 0; i < 16; ++i) { acc[0][i] = 0.f; acc[1][i] = 0.f; }
#pragma unroll
    for (int B = 0; B < 4; ++B) {
        u32x4 xf[4];
        const int bm = (B + rot) & 3;
        unpack_block(xf, rawX[B][0], rawX[B][1], rawX[B][2], (u32)__builtin_amdgcn_readlane((int)(u32)fvalX, bm), lane < 32 ? excX : EXC_NONE, bm, lane, scratch, rsX, bm * SLOT);
#pragma unroll
        for (int f = 0; f < 4; ++f) {
            const int T = 4 * B + f;
            acc[0] = __builtin_amdgcn_mfma_f32_32x32x16_bf16(__builtin_bit_cast(bf16x8, wf[T >> 1][2 * (T & 1)]), __builtin_bit_cast(bf16x8, xf[f]), acc[0], 0, 0, 0);
            acc[1] = __builtin_amdgcn_mfma_f32_32x32x16_bf16(__builtin_bit_cast(bf16x8, wf[T >> 1][2 * (T & 1) + 1]), __builtin_bit_cast(bf16x8, xf[f]), acc[1], 0, 0, 0);
        }
    }
    if ((!ok || tm == call) && lane == 0) *(LAS u32*)(lds + L_MISC + 84) = 1u;
#pragma unroll
    for (int cb = 0; cb < 2; ++cb)
#pragma unroll
        for (int g4 = 0; g4 < 4; ++g4)
            *(LAS f32x4*)(lds + L_RED + p * RED_WAVE + r * RED_STRIDE + (32 * cb + 8 * g4 + 4 * h) * 4) =
                (f32x4){acc[cb][4 * g4], acc[cb][4 * g4 + 1], acc[cb][4 * g4 + 2], acc[cb][4 * g4 + 3]};
    if (self && !early) { xstores(); drain(); xflags(); }
    if (tid < 184) *(LAS u32x4*)(lds + L_IMG + 256 + tid * 16) = (u32x4){0, 0, 0, 0};
    STAMP(7);
    __syncthreads();
    STAMP(13);
    u32 o01, o23, pre, m4;
    {
        const int m = tid >> 4, n0 = 4 * (tid & 15);
        const bool bad = rfl(*(LAS u32*)(lds + L_MISC + 84)) != 0u;
        f32x4 s = *(LAS f32x4*)(lds + L_RED + m * RED_STRIDE + n0 * 4);
#pragma unroll
        for (int pp = 1; pp < 8; ++pp) s += *(LAS f32x4*)(lds + L_RED + pp * RED_WAVE + m * RED_STRIDE + n0 * 4);
        const f32x4 y = __builtin_elementwise_maximum(s, (f32x4){0.f, 0.f, 0.f, 0.f});
        o01 = __builtin_bit_cast(u32, __builtin_convertvector((f32x2){y.x, y.y}, bf16x2)) & 0x7FFF7FFFu;
        o23 = __builtin_bit_cast(u32, __builtin_convertvector((f32x2){y.z, y.w}, bf16x2)) & 0x7FFF7FFFu;
        if (bad) { o01 = 0x7FC07FC0u; o23 = 0x7FC07FC0u; }
        *(LAS u32x2*)(lds + L_OUT + tid * 8) = (u32x2){o01, o23};
        const u16x2 mm = __builtin_elementwise_max(__builtin_bit_cast(u16x2, o01), __builtin_bit_cast(u16x2, o23));
        const u32 wmx = wave_max(max((u32)mm.x, (u32)mm.y));
        const bool z0 = (o01 & 0xFFFFu) != 0u, z1 = o01 > 0xFFFFu, z2 = (o23 & 0xFFFFu) != 0u, z3 = o23 > 0xFFFFu;
        const u64 b0 = __ballot(z0), b1 = __ballot(z1), b2 = __ballot(z2), b3 = __ballot(z3);
        pre = mbcnt64(b3, mbcnt64(b2, mbcnt64(b1, mbcnt64(b0, 0u))));
        m4 = (z0 ? 1u : 0u) | (z1 ? 2u : 0u) | (z2 ? 4u : 0u) | (z3 ? 8u : 0u);
        if (lane == 0) *(LAS u32*)(lds + L_MISC + 16 + wv * 4) = ((wmx >> 8) << 16) | (u32)(__popcll(b0) + __popcll(b1) + __popcll(b2) + __popcll(b3));
    }
    STAMP(14);
    __syncthreads();
    STAMP(15);
    u32 ntot, obase;
    {
        const u32 w = *(LAS u32*)(lds + L_MISC + 16 + (lane & 7) * 4);
        const u32 sel = *(LAS u32*)(lds + L_LUT4 + m4 * 4);
        u32 cs = (w & 0xFFFFu) * ((lane & 7) < wv ? 0x10001u : 1u), mx = w >> 16;
        cs += dpp0<0x111, 0xf>(cs); mx = max(mx, dpp0<0x111, 0xf>(mx));
        cs += dpp0<0x112, 0xf>(cs); mx = max(mx, dpp0<0x112, 0xf>(mx));
        cs += dpp0<0x114, 0xf>(cs); mx = max(mx, dpp0<0x114, 0xf>(mx));
        const u32 cst = (u32)__builtin_amdgcn_readlane((int)cs, 7), h7 = (u32)__builtin_amdgcn_readlane((int)mx, 7);
        ntot = cst & 0xFFFFu; obase = h7 > 7u ? h7 - 7u : 0u;
        const u32 pos = (cst >> 16) + pre;
        const u32 coff = img_coff(ntot);
        const u32 L = perm(o23, o01, 0x06040200u), H = perm(o23, o01, 0x07050301u);
        const u32 d = H + (128u - obase) * 0x01010101u;
        const u32 Lc = perm(0u, L, sel), Cc = perm(0u, d & 0x07070707u, sel);
        const u32 c12 = __builtin_amdgcn_udot4(Cc, 0x00400801u, (Cc >> 15) & 0xE00u, false);
        {
            const u32 sh = (pos & 3u) * 8u;
            lu32* lp = (lu32*)(lds + L_IMG + 256 + (pos & ~3u));
            if (Lc) { __hip_atomic_fetch_or(lp, Lc << sh, __ATOMIC_RELAXED, __HIP_MEMORY_SCOPE_WORKGROUP);
                const u32 hi = (u32)(((u64)Lc << sh) >> 32); if (hi) __hip_atomic_fetch_or(lp + 1, hi, __ATOMIC_RELAXED, __HIP_MEMORY_SCOPE_WORKGROUP); }
        }
        if (c12) {
            const u32 bit = 3u * pos, sh = bit & 31u;
            lu32* cp = (lu32*)(lds + L_IMG + coff + (bit >> 5) * 4);
            __hip_atomic_fetch_or(cp, c12 << sh, __ATOMIC_RELAXED, __HIP_MEMORY_SCOPE_WORKGROUP);
            const u32 hi = (u32)(((u64)c12 << sh) >> 32); if (hi) __hip_atomic_fetch_or(cp + 1, hi, __ATOMIC_RELAXED, __HIP_MEMORY_SCOPE_WORKGROUP);
        }
        if (~perm(0u, d, sel | ((sel & 0x08080808u) >> 3)) & 0x80808080u) {
#pragma unroll
            for (int j = 0; j < 4; ++j) {
                const u32 oj = ((j < 2 ? o01 : o23) >> (16 * (j & 1))) & 0xFFFFu;
                if (oj != 0u && (oj >> 8) < obase) {
                    const u32 e = __hip_atomic_fetch_add((lu32*)(lds + L_MISC + 80), 1u, __ATOMIC_RELAXED, __HIP_MEMORY_SCOPE_WORKGROUP);
                    if (e < 8u) *(LAS u32*)(lds + L_IMG + img_eoff(ntot) + e * 4) = ((u32)(tid * 4 + j) << 16) | oj;
                }
            }
        }
        const u32 other = (u32)__builtin_amdgcn_update_dpp(0, (int)m4, 0xB1  , 0xf, 0xf, false);
        const u32 gi = (u32)tid >> 1;
        if (!(lane & 1)) *(LAS unsigned char*)(lds + L_IMG + 4 * (gi & 63u) + (gi >> 6)) = (unsigned char)(m4 | (other << 4));
    }
    STAMP(8);
    __syncthreads();
    const int tile = u * 8 + v;
    const size_t PASS = (size_t)8 * N_ * 2;
    const u32 nexc = rfl(*(LAS u32*)(lds + L_MISC + 80));
    const bool raw = nexc > 8u;
    const bool direct = tile >= P.tDir;
    const int row0 = 1024 * (rank >> 2) + 32 * u, col0 = 512 * (rank & 3) + 64 * v;
    if (self) {
        u32x4 d[4];
#pragma unroll
        for (int s = 0; s < 4; ++s) d[s] = *(LAS u32x4*)(lds + L_OUT + s * 1024 + lane * 16);
        gu8* dst = (gu8*)P.out[rank] + ((size_t)(row0 + (lane >> 3)) * N_ + col0) * 2 + (lane & 7) * 16;
#pragma unroll
        for (int s = 0; s < 4; ++s) st16_out(dst + s * PASS, d[s]);
        const int fd = (lane >> 2) & 7;
        unsigned char* q = P.ws[0];
#pragma unroll
        for (int i = 1; i < 8; ++i) q = fd == i ? P.ws[i] : q;
        gu8* fp = (gu8*)q + WS_OFLAG + (rank * 256 + tile) * 128;
        const u64 fo = fv | (direct ? F_DIRECT : 0u) | (raw ? F_RAW : 0u) | (nexc ? F_EXC : 0u) | (obase << 12) | ntot;
        bool todo = lane < 32 && fd != rank;
        const u64 t0 = rt();
        while (__any(todo)) {
            const u32 sw = __hip_atomic_load((lu32*)(lds + L_MISC + 48 + fd * 4), __ATOMIC_ACQUIRE, __HIP_MEMORY_SCOPE_WORKGROUP);
            if (todo && sw) { st16_sys(fp + (lane & 3) * 16, (lane & 3) ? (u32x4){0, 0, 0, 0} : (u32x4){(u32)fo, (u32)(fo >> 32), 0u, 0u}); todo = false; }
            if (rt() - t0 > DEADLINE) break;
        }
        STAMP(10);
    } else {
        const u32 nch = (raw || direct) ? 256u : ((img_chunks(ntot, nexc != 0u) + 3u) & ~3u);
        lu8* S = lds + ((raw || direct) ? L_OUT : L_IMG);
        gu8* dst = direct ? (gu8*)P.out[wv] + ((size_t)(row0 + (lane >> 3)) * N_ + col0) * 2 + (lane & 7) * 16
                          : (gu8*)P.ws[wv] + WS_ODATA + rank * OSRC + (size_t)tile * SLOT + lane * 16;
        const size_t dstep = direct ? PASS : (size_t)1024;
        u32x4 d[4];
#pragma unroll
        for (int s = 0; s < 4; ++s) d[s] = *(LAS u32x4*)(S + s * 1024 + lane * 16);
        wait_until(tstart + (u64)(direct ? P.tO1 + (((tile - P.tDir) * P.dO1) >> 8) : P.tO0 + ((tile * P.dO) >> 8)));
#pragma unroll
        for (int s = 0; s < 4; ++s) if ((u32)(lane + 64 * s) < nch) st16_sys(dst + s * dstep, d[s]);
        STAMP(9);
        drain();
        if (lane == 0) __hip_atomic_store((lu32*)(lds + L_MISC + 48 + wv * 4), 1u, __ATOMIC_RELEASE, __HIP_MEMORY_SCOPE_WORKGROUP);
        STAMP(10);
        u64 fval;
        const bool ok2 = wait_flags(myws + WS_OFLAG + (wv * 256 + tile) * 128, call, tmo, fval);
        STAMP(11);
        const u32 fl = rfl((u32)fval);
        const u32 n = fl & 4095u, rb = (fl >> 12) & 127u;
        const bool rraw = (fl & F_RAW) != 0u;
        const u32 rch = rraw ? 256u : img_chunks(n, (fl & F_EXC) != 0u);
        const __amdgpu_buffer_rsrc_t rs = __builtin_amdgcn_make_buffer_rsrc((void*)(myws + WS_ODATA + wv * OSRC + (size_t)tile * SLOT), 0, (fl & F_DIRECT) ? 0u : rch * 16, 0x00020000);
        const u32 mbw = __builtin_amdgcn_raw_buffer_load_b32(rs, lane * 4, 0, 17);
        u32x4 in[4];
#pragma unroll
        for (int s = 0; s < 4; ++s) in[s] = __builtin_amdgcn_raw_buffer_load_b128(rs, lane * 16 + s * 1024, 0, 17);
        const u32 tm2 = __hip_atomic_load(tmo, __ATOMIC_RELAXED, __HIP_MEMORY_SCOPE_AGENT);
        const int rrow0 = 1024 * (wv >> 2) + 32 * u, rcol0 = 512 * (wv & 3) + 64 * v;
        gu8* odst = (gu8*)P.out[rank] + ((size_t)(rrow0 + (lane >> 3)) * N_ + rcol0) * 2 + (lane & 7) * 16;
        if (!ok2 || tm2 == call) {
#pragma unroll
            for (int s = 0; s < 4; ++s) st16_sys(odst + s * PASS, (u32x4){0x7FC07FC0u, 0x7FC07FC0u, 0x7FC07FC0u, 0x7FC07FC0u});
        } else if (fl & F_DIRECT) {
        } else if (rraw) {
#pragma unroll
            for (int s = 0; s < 4; ++s) st16_out(odst + s * PASS, in[s]);
        } else {
            lu8* sc = lds + L_RED + wv * RED_WAVE;
            const u32 mb0 = mbw & 255u, mb1 = (mbw >> 8) & 255u, mb2 = (mbw >> 16) & 255u, mb3 = mbw >> 24;
            u32x2 sel[4];
            sel[0] = *(LAS u32x2*)(lds + L_LUT + mb0 * 8); sel[1] = *(LAS u32x2*)(lds + L_LUT + mb1 * 8);
            sel[2] = *(LAS u32x2*)(lds + L_LUT + mb2 * 8); sel[3] = *(LAS u32x2*)(lds + L_LUT + mb3 * 8);
            const u32 c01 = (u32)__popc(mb0) | ((u32)__popc(mb1) << 16), c23 = (u32)__popc(mb2) | ((u32)__popc(mb3) << 16);
            const u32 s01 = wave_scan_add(c01), s23 = wave_scan_add(c23);
#pragma unroll
            for (int s = 0; s < 4; ++s) *(LAS u32x4*)(sc + s * 1024 + lane * 16) = in[s];
            asm volatile("" ::: "memory");
            const u32 coff = img_coff(n);
            const u32 t01 = (u32)__builtin_amdgcn_readlane((int)s01, 63), t23 = (u32)__builtin_amdgcn_readlane((int)s23, 63);
            const u32 e01 = s01 - c01, e23 = s23 - c23;
            u32 off[4];
            off[0] = e01 & 0xFFFFu; off[1] = (t01 & 0xFFFFu) + (e01 >> 16);
            const u32 b2 = (t01 & 0xFFFFu) + (t01 >> 16);
            off[2] = b2 + (e23 & 0xFFFFu); off[3] = b2 + (t23 & 0xFFFFu) + (e23 >> 16);
            const u32 b4 = rb * 0x01010101u;
            u32 a0[4], a1[4], a2[4], k0[4], k1[4];
#pragma unroll
            for (int i = 0; i < 4; ++i) {
                lu32* lp = (lu32*)(sc + 256 + (off[i] & ~3u));
                a0[i] = lp[0]; a1[i] = lp[1]; a2[i] = lp[2];
                lu32* cp = (lu32*)(sc + coff + ((3u * off[i]) >> 5) * 4);
                k0[i] = cp[0]; k1[i] = cp[1];
            }
#pragma unroll
            for (int i = 0; i < 4; ++i) {
                const u32 lo0 = __builtin_amdgcn_alignbyte(a1[i], a0[i], off[i] & 3u), lo1 = __builtin_amdgcn_alignbyte(a2[i], a1[i], off[i] & 3u);
                const u32 c = __builtin_amdgcn_alignbit(k1[i], k0[i], (3u * off[i]) & 31u);
                u32 x0 = c & 0xFFFu, x1 = (c >> 12) & 0xFFFu;
                x0 = (x0 | (x0 << 10)) & 0x003F003Fu; x1 = (x1 | (x1 << 10)) & 0x003F003Fu;
                x0 = (x0 | (x0 << 5)) & 0x07070707u; x1 = (x1 | (x1 << 5)) & 0x07070707u;
                const u32 h0 = x0 + b4, h1 = x1 + b4;
                const u32 l0 = perm(lo1, lo0, sel[i].x), l1 = perm(lo1, lo0, sel[i].y), g0 = perm(h1, h0, sel[i].x), g1 = perm(h1, h0, sel[i].y);
                st16_out(odst + i * PASS, (u32x4){perm(g0, l0, 0x05010400u), perm(g0, l0, 0x07030602u), perm(g1, l1, 0x05010400u), perm(g1, l1, 0x07030602u)});
            }
            if (fl & F_EXC) {
                drain();
                if (lane < 8) { const u32 e = *(LAS u32*)(sc + img_eoff(n) + lane * 4);
                    if (e != 0u) { const u32 ix = (e >> 16) & 2047u; *(gu16*)((gu8*)P.out[rank] + ((size_t)(rrow0 + (ix >> 6)) * N_ + rcol0 + (ix & 63u)) * 2) = (unsigned short)e; } }
            }
        }
        STAMP(12);
    }
}

#define T_W0 125
#define D_W 345
#define T_X0 470
#define D_X 690
#define T_O0 1195
#define D_O 785
#define T_POLL 130
#define T_DIR 256
#define D_O1 1850
#define T_SELF 800
#define T_WD 380
extern "C" void kernel_launch(int rank, int nranks, ncclComm_t comm, hipStream_t stream,
                              void* const* d_in, const int* in_sizes, int n_in,
                              void* d_out, int out_size, const int* in_dtypes, int out_dtype,
                              void* const* d_out_all, void* const* d_ws_all, size_t ws_bytes) {
    if (nranks != NR || ws_bytes < DIAG_OFF + DIAG_BYTES) { fprintf(stderr, "bad config\n"); return; }
    static bool init = false;
    static int tp[14];
    if (!init) {
        (void)hipFuncSetAttribute((const void*)fused, hipFuncAttributeMaxDynamicSharedMemorySize, LDS_BYTES);
        const int df[14] = {T_X0, D_X, T_O0, D_O, T_W0, D_W, T_POLL, T_DIR, T_SELF, T_WD, 0, -1, 0, D_O1};
        for (int i = 0; i < 14; ++i) tp[i] = df[i];
        init = true;
    }
    Params P;
    P.x = (const unsigned short*)d_in[0]; P.w = (const unsigned short*)d_in[1];
    for (int i = 0; i < NR; ++i) { P.ws[i] = (unsigned char*)d_ws_all[i]; P.out[i] = (unsigned char*)d_out_all[i]; }
    P.rank = rank; P.tX0 = tp[0]; P.dX = tp[1]; P.tO0 = tp[2]; P.dO = tp[3]; P.tW0 = tp[4]; P.dW = tp[5]; P.tPoll = tp[6]; P.tDir = tp[7]; P.tSelf = tp[8]; P.tWd = tp[9]; P.tO1 = tp[10] ? tp[10] : tp[2] + ((tp[7] * tp[3]) >> 8); P.dO1 = tp[13]; P.pad = 0; P.delay = (rank == tp[11]) ? tp[12] : 0;
    hipLaunchKernelGGL(fused, dim3(NWG), dim3(NTHR), LDS_BYTES, stream, P);
}
```

```cpp
#include <hip/hip_runtime.h>
#include <rccl/rccl.h>
#include <cstdio>

#define GAS __attribute__((address_space(1)))
#define LAS __attribute__((address_space(3)))
typedef unsigned u32;
typedef unsigned long long u64;
typedef u32 u32x4 __attribute__((ext_vector_type(4)));
typedef u32 u32x2 __attribute__((ext_vector_type(2)));
typedef float f32x2 __attribute__((ext_vector_type(2)));
typedef float f32x4 __attribute__((ext_vector_type(4)));
typedef float f32x16 __attribute__((ext_vector_type(16)));
typedef __bf16 bf16x8 __attribute__((ext_vector_type(8)));
typedef __bf16 bf16x2 __attribute__((ext_vector_type(2)));
typedef unsigned short u16x2 __attribute__((ext_vector_type(2)));
typedef GAS u32 gu32; typedef GAS u64 gu64; typedef GAS unsigned char gu8; typedef GAS unsigned short gu16;
typedef LAS unsigned char lu8; typedef LAS u32 lu32;

constexpr int NR = 8, M_ = 2048, N_ = 2048, KS = 256;
constexpr int NWG = 256, NTHR = 512;

constexpr size_t WS_TMO = 0;
constexpr size_t WS_CNT = 4096;
constexpr size_t WS_XFLAG = WS_CNT + 256 * 128;
constexpr size_t WS_WFLAG = WS_XFLAG + 8 * 32 * 128;
constexpr size_t WS_OFLAG = WS_WFLAG + 8 * 8 * 128;
constexpr size_t WS_XEXC = WS_OFLAG + 8 * 256 * 128;
constexpr size_t WS_WEXC = WS_XEXC + 8 * 32 * 128;
constexpr size_t WS_FEND = WS_WEXC + 8 * 8 * 256;
constexpr size_t SLOT = 4224, SKEW = 20608;
constexpr size_t XSRC = 32 * 4 * SLOT + SKEW, WSRC = 8 * 8 * SLOT + SKEW, OSRC = 256 * SLOT + SKEW;
constexpr size_t WS_XDATA = 1u << 20;
constexpr size_t WS_WDATA = WS_XDATA + 8 * XSRC;
constexpr size_t WS_ODATA = WS_WDATA + 8 * WSRC;
constexpr size_t WS_END = WS_ODATA + 8 * OSRC;
static_assert(WS_FEND <= WS_XDATA && SLOT % 128 == 0 && SKEW % 128 == 0, "workspace layout");
constexpr u64 DEADLINE = 30000000ull;

constexpr int T_W0 = 100, D_W = 345;
constexpr int T_X0 = 445, D_X = 690;
constexpr int T_O0 = 1160, D_O = 795;
constexpr int T_POLL = 130;
constexpr int T_SELF = 800;
constexpr int T_WD = 380;

struct Params {
    const unsigned short* x; const unsigned short* w;
    unsigned char* ws[NR]; unsigned char* out;
    int rank, pad;
};

__device__ __forceinline__ u64 rt() { return __builtin_amdgcn_s_memrealtime(); }
__device__ __forceinline__ void st16_sys(gu8* p, u32x4 v) { asm volatile("global_store_dwordx4 %0, %1, off sc0 sc1\n\ts_nop 1" :: "v"(p), "v"(v) : "memory"); }
__device__ __forceinline__ void st4_sys(gu8* p, u32 v) { asm volatile("global_store_dword %0, %1, off sc0 sc1\n\ts_nop 1" :: "v"(p), "v"(v) : "memory"); }
__device__ __forceinline__ void drain() { asm volatile("s_waitcnt vmcnt(0)" ::: "memory"); }
__device__ __forceinline__ void flag_store(gu8* p, u64 v) { __hip_atomic_store((gu64*)p, v, __ATOMIC_RELAXED, __HIP_MEMORY_SCOPE_SYSTEM); }
__device__ __forceinline__ u64 flag_load(gu8* p) { return __hip_atomic_load((gu64*)p, __ATOMIC_RELAXED, __HIP_MEMORY_SCOPE_SYSTEM); }
__device__ __forceinline__ u32 rfl(u32 v) { return __builtin_amdgcn_readfirstlane(v); }
__device__ __forceinline__ u32 rdl(u32 v, int l) { return (u32)__builtin_amdgcn_readlane((int)v, l); }
__device__ __forceinline__ u32 perm(u32 hi, u32 lo, u32 sel) { return __builtin_amdgcn_perm(hi, lo, sel); }
__device__ __forceinline__ u32 rotr(u32 x, u32 s) { return __builtin_amdgcn_alignbit(x, x, s); }
__device__ __forceinline__ u32 mbcnt64(u64 m, u32 acc) { return __builtin_amdgcn_mbcnt_hi((u32)(m >> 32), __builtin_amdgcn_mbcnt_lo((u32)m, acc)); }
template <int CTRL, int RM> __device__ __forceinline__ u32 dpp0(u32 v) { return (u32)__builtin_amdgcn_update_dpp(0, (int)v, CTRL, RM, 0xf, false); }
__device__ __forceinline__ u32 wave_max(u32 v) {
    v = max(v, dpp0<0x111, 0xf>(v)); v = max(v, dpp0<0x112, 0xf>(v)); v = max(v, dpp0<0x114, 0xf>(v)); v = max(v, dpp0<0x118, 0xf>(v));
    v = max(v, dpp0<0x142, 0xa>(v)); v = max(v, dpp0<0x143, 0xc>(v));
    return rdl(v, 63);
}
__device__ __forceinline__ u32 wave_scan_add(u32 v) {
    v += dpp0<0x111, 0xf>(v); v += dpp0<0x112, 0xf>(v); v += dpp0<0x114, 0xf>(v); v += dpp0<0x118, 0xf>(v);
    v += dpp0<0x142, 0xa>(v); v += dpp0<0x143, 0xc>(v);
    return v;
}

__device__ __forceinline__ bool wait_flags(gu8* p, u32 call, gu32* tmo, u64& val) {
    const u64 t0 = rt();
    for (u32 it = 1;; ++it) {
        const u64 v = flag_load(p);
        if (__all((u32)(v >> 32) >= call)) { val = v; asm volatile("" ::: "memory"); return true; }
        if ((it & 31u) == 0u) {
            if (__hip_atomic_load(tmo, __ATOMIC_RELAXED, __HIP_MEMORY_SCOPE_AGENT) == call || rt() - t0 > DEADLINE) {
                __hip_atomic_store(tmo, call, __ATOMIC_RELAXED, __HIP_MEMORY_SCOPE_AGENT);
                val = 0; asm volatile("" ::: "memory"); return false;
            }
        }
        __builtin_amdgcn_s_sleep(2);
    }
}
__device__ __forceinline__ void wait_until(u64 t) { while (rt() < t) __builtin_amdgcn_s_sleep(1); }

constexpr int RED_STRIDE = 272;
constexpr int RED_WAVE = 32 * RED_STRIDE;
constexpr int L_RED = 0;
constexpr int L_XP = 8 * RED_WAVE;
constexpr int L_WP = L_XP + 3072;
constexpr int L_XR = L_WP + 3072;
constexpr int L_WR = L_XR + 4096;
constexpr int L_XH = L_WR + 4096;
constexpr int L_WH = L_XH + 64;
constexpr int L_OUT = L_WH + 64;
constexpr int L_IMG = L_OUT + 4096;
constexpr int L_LUT = L_IMG + 3200;
constexpr int L_LUT4 = L_LUT + 2048;
constexpr int L_MISC = L_LUT4 + 64;
constexpr int LDS_BYTES = L_MISC + 128;
static_assert(LDS_BYTES > 81920 && LDS_BYTES <= 163840 && L_XP % 16 == 0 && L_IMG % 16 == 0 && L_LUT % 16 == 0 && L_MISC % 16 == 0, "lds");

constexpr u32 F_RAW = 1u << 31, F_EXC = 1u << 30;
constexpr u32 EXC_NONE = 0xFFFFFFFFu;
__device__ __forceinline__ u32 img_coff(u32 n) { return 256u + ((n + 3u) & ~3u); }
__device__ __forceinline__ u32 img_eoff(u32 n) { return img_coff(n) + ((((3u * n + 7u) >> 3) + 3u) & ~3u); }
__device__ __forceinline__ u32 img_chunks(u32 n, bool exc) { return (img_eoff(n) + (exc ? 32u : 0u) + 15u) >> 4; }

__device__ __forceinline__ void pack_frag(u32x4 e, int f, int lane, lu8* P, lu8* R, lu8* H) {
    *(LAS u32x4*)(R + f * 1024 + lane * 16) = e;
    const u16x2 m01 = __builtin_elementwise_max(__builtin_bit_cast(u16x2, e.x & 0x7FFF7FFFu), __builtin_bit_cast(u16x2, e.y & 0x7FFF7FFFu));
    const u16x2 m23 = __builtin_elementwise_max(__builtin_bit_cast(u16x2, e.z & 0x7FFF7FFFu), __builtin_bit_cast(u16x2, e.w & 0x7FFF7FFFu));
    const u16x2 mm = __builtin_elementwise_max(m01, m23);
    const u32 mx = wave_max(max((u32)mm.x, (u32)mm.y));
    const u32 h7 = mx >> 8, base = h7 > 7u ? h7 - 7u : 0u;
    const u32 lo0 = perm(e.y, e.x, 0x06040200u), lo1 = perm(e.w, e.z, 0x06040200u);
    const u32 hi0 = perm(e.y, e.x, 0x07050301u), hi1 = perm(e.w, e.z, 0x07050301u);
    const u32 K = (128u - base) * 0x01010101u;
    const u32 d0 = (hi0 & 0x7F7F7F7Fu) + K, d1 = (hi1 & 0x7F7F7F7Fu) + K;
    const u32 cA = (hi0 & 0x80808080u) | (d0 & 0x07070707u), cB = (hi1 & 0x80808080u) | (d1 & 0x07070707u);
    *(LAS u32x2*)(P + (f >> 1) * 1024 + lane * 16 + (f & 1) * 8) = (u32x2){lo0, lo1};
    *(LAS u32*)(P + 2048 + lane * 16 + f * 4) = cA | rotr(cB, 28);
    const u32 x0 = ~d0 & 0x80808080u, x1 = ~d1 & 0x80808080u;
    u32 hdr = base;
    if (lane < 2) *(LAS u32*)(H + 16 + f * 8 + lane * 4) = EXC_NONE;
    asm volatile("" ::: "memory");
    if (__any((x0 | x1) != 0u)) {
        u32 cnt = 0;
#pragma unroll
        for (int j = 0; j < 8; ++j) {
            const bool ex = (((j < 4 ? x0 : x1) >> (8 * (j & 3) + 7)) & 1u) != 0u;
            const u64 b = __ballot(ex);
            const u32 rk = mbcnt64(b, cnt);
            const u32 val = (e[j >> 1] >> (16 * (j & 1))) & 0xFFFFu;
            if (ex && rk < 2u) *(LAS u32*)(H + 16 + f * 8 + rk * 4) = ((u32)(f * 512 + lane * 8 + j) << 16) | val;
            cnt += (u32)__popcll(b);
        }
        hdr |= cnt > 2u ? 0x300u : 0x100u;
    }
    if (lane == 0) *(LAS u32*)(H + f * 4) = hdr;
}
__device__ __forceinline__ void load_block(u32x4 (&d)[4], u32& fl, u32& excw, int lane, lu8* P, lu8* R, lu8* H) {
    const u32x4 h = *(LAS u32x4*)H;
    const u32 any = rfl(h.x | h.y | h.z | h.w);
    const bool raw = (any & 0x200u) != 0u;
    fl = (rfl(h.x) & 127u) | ((rfl(h.y) & 127u) << 7) | ((rfl(h.z) & 127u) << 14) | ((rfl(h.w) & 127u) << 21) | (raw ? F_RAW : ((any & 0x100u) ? F_EXC : 0u));
    lu8* S = raw ? R : P;
    d[0] = *(LAS u32x4*)(S + lane * 16); d[1] = *(LAS u32x4*)(S + 1024 + lane * 16); d[2] = *(LAS u32x4*)(S + 2048 + lane * 16);
    d[3] = *(LAS u32x4*)(R + 3072 + lane * 16);
    excw = *(LAS u32*)(H + 16 + (lane & 7) * 4);
}
__device__ __forceinline__ void send_block(const u32x4 (&d)[4], u32 fl, u32 excw, int lane, gu8* dst, gu8* dexc) {
    st16_sys(dst + lane * 16, d[0]); st16_sys(dst + 1024 + lane * 16, d[1]); st16_sys(dst + 2048 + lane * 16, d[2]);
    if (fl & F_RAW) st16_sys(dst + 3072 + lane * 16, d[3]);
    if ((fl & F_EXC) && lane < 8) st4_sys(dexc + lane * 4, excw);
}
__device__ __forceinline__ u32x4 unpack_frag(u32 lo0, u32 lo1, u32 nw, u32 b4) {
    const u32 hA = (nw & 0x87878787u) + b4, hB = (rotr(nw, 4) & 0x87878787u) + b4;
    return (u32x4){perm(hA, lo0, 0x05010400u), perm(hA, lo0, 0x07030602u), perm(hB, lo1, 0x05010400u), perm(hB, lo1, 0x07030602u)};
}
__device__ __forceinline__ void unpack_block(u32x4 (&fr)[4], u32x4 L0, u32x4 L1, u32x4 Nw, u32 fl, u32 excw, int blk, int lane, lu8* scratch,
                                             __amdgpu_buffer_rsrc_t rs, int soff) {
    if (fl & F_RAW) {
        fr[0] = L0; fr[1] = L1; fr[2] = Nw;
        fr[3] = __builtin_amdgcn_raw_buffer_load_b128(rs, lane * 16, soff + 3072, 17);
        asm volatile("" : "+v"(fr[3]));
        return;
    }
    fr[0] = unpack_frag(L0.x, L0.y, Nw.x, (fl & 127u) * 0x01010101u);
    fr[1] = unpack_frag(L0.z, L0.w, Nw.y, ((fl >> 7) & 127u) * 0x01010101u);
    fr[2] = unpack_frag(L1.x, L1.y, Nw.z, ((fl >> 14) & 127u) * 0x01010101u);
    fr[3] = unpack_frag(L1.z, L1.w, Nw.w, ((fl >> 21) & 127u) * 0x01010101u);
    if (fl & F_EXC) {
#pragma unroll
        for (int f = 0; f < 4; ++f) *(LAS u32x4*)(scratch + f * 1024 + lane * 16) = fr[f];
        asm volatile("" ::: "memory");
        if ((lane >> 3) == blk && excw != EXC_NONE) *(LAS unsigned short*)(scratch + ((excw >> 16) & 2047u) * 2) = (unsigned short)excw;
        asm volatile("" ::: "memory");
#pragma unroll
        for (int f = 0; f < 4; ++f) fr[f] = *(LAS u32x4*)(scratch + f * 1024 + lane * 16);
        asm volatile("" : "+v"(fr[0]), "+v"(fr[1]), "+v"(fr[2]), "+v"(fr[3]));
    }
}

#define SB() __builtin_amdgcn_sched_barrier(0)

__global__ void __launch_bounds__(NTHR, 2) fused(Params P) {
    extern __shared__ __attribute__((aligned(16))) unsigned char smem[];
    lu8* lds = (lu8*)smem;
    const int tid = threadIdx.x, lane = tid & 63, wv = __builtin_amdgcn_readfirstlane(tid >> 6);
    const int g = blockIdx.x, xcd = g & 7, idx = g >> 3, u = 2 * (idx >> 1) + (xcd >> 2), v = 2 * (xcd & 3) + (idx & 1);
    const int rank = P.rank;
    gu8* myws = (gu8*)P.ws[rank];
    const u64 tstart = rt();
    gu32* tmo = (gu32*)(myws + WS_TMO);
    gu32* cntp = (gu32*)(myws + WS_CNT + g * 128);
    u32 call0 = 0;
    if (tid == 0) call0 = __hip_atomic_load(cntp, __ATOMIC_RELAXED, __HIP_MEMORY_SCOPE_AGENT);

    const int xi = v >> 2, xB = v & 3;
    const int wV = g >> 3, wb = g & 7;
    const int r = lane & 31, h = lane >> 5;
    {
        u32x4 frag;
        if (wv < 4) {
            frag = *(const GAS u32x4*)((const gu16*)P.x + (size_t)(1024 * xi + 32 * u + r) * KS + 64 * xB + 16 * wv + 8 * h);
            pack_frag(frag, wv, lane, lds + L_XP, lds + L_XR, lds + L_XH);
        } else {
            const int f = wv - 4;
            const gu16* src = (const gu16*)P.w + (size_t)(32 * wb + 16 * (f >> 1) + 8 * h) * N_ + 64 * wV + 32 * (f & 1) + r;
            u32 e[8];
#pragma unroll
            for (int j = 0; j < 8; ++j) e[j] = src[(size_t)j * N_];
            frag = (u32x4){e[0] | (e[1] << 16), e[2] | (e[3] << 16), e[4] | (e[5] << 16), e[6] | (e[7] << 16)};
            pack_frag(frag, f, lane, lds + L_WP, lds + L_WR, lds + L_WH);
        }
    }
    if (tid == 0) { call0 += 1u; *(LAS u32*)(lds + L_MISC) = call0; __hip_atomic_store(cntp, call0, __ATOMIC_RELAXED, __HIP_MEMORY_SCOPE_AGENT); }
    if (tid >= 64 && tid < 64 + 11) *(LAS u32*)(lds + L_MISC + 48 + (tid - 64) * 4) = 0u;
    __syncthreads();
    const u32 call = rfl(*(LAS u32*)(lds + L_MISC));
    const u64 fv = (u64)call << 32;
    const int wa = (wv - rank) & 7;
    if (wa == 1 || wa == 2) {
        const int q = (wV >> 3) + 4 * (wa - 1), vv = wV & 7;
        gu8* dws = (gu8*)P.ws[q];
        u32x4 d[4]; u32 fl, excw;
        load_block(d, fl, excw, lane, lds + L_WP, lds + L_WR, lds + L_WH);
        if (q != rank) wait_until(tstart + (u64)(T_W0 + ((((g & 63) * 4 + (g >> 6)) * D_W) >> 8)));
        send_block(d, fl, excw, lane, dws + WS_WDATA + rank * WSRC + (size_t)(vv * 8 + wb) * SLOT, dws + WS_WEXC + (rank * 8 + vv) * 256 + wb * 32);
        drain();
        if (lane == 0) flag_store(dws + WS_WFLAG + (rank * 8 + vv) * 128 + wb * 8, fv | fl);
    } else if (wa == 3) {
        if (xi == (rank >> 2)) {
            u32x4 d[4]; u32 fl, excw;
            load_block(d, fl, excw, lane, lds + L_XP, lds + L_XR, lds + L_XH);
            send_block(d, fl, excw, lane, myws + WS_XDATA + rank * XSRC + (size_t)(u * 4 + xB) * SLOT, myws + WS_XEXC + (rank * 32 + u) * 128 + xB * 32);
            drain();
            if (lane == 0) flag_store(myws + WS_XFLAG + (rank * 32 + u) * 128 + xB * 8, fv | fl);
        }
    } else if (wa >= 4) {
        const u32 m = (u32)(wa - 4) * 64u + (u32)lane;
        u32 s0 = 0, s1 = 0, c = 0;
#pragma unroll
        for (int j = 0; j < 8; ++j) { const u32 b = (m >> j) & 1u; const u32 sv = b ? c : 0x0Cu; c += b; if (j < 4) s0 |= sv << (8 * j); else s1 |= sv << (8 * (j - 4)); }
        *(LAS u32x2*)(lds + L_LUT + m * 8) = (u32x2){s0, s1};
        if (m < 16u) {
            u32 sel = 0x0C0C0C0Cu, k = 0;
#pragma unroll
            for (int j = 0; j < 4; ++j) if ((m >> j) & 1u) { sel = (sel & ~(0xFFu << (8 * k))) | ((u32)j << (8 * k)); k++; }
            *(LAS u32*)(lds + L_LUT4 + m * 4) = sel;
        }
    }

    bool ok = true; u64 fvalW, fvalX;
    const int p = wv;
    const bool self = wa == 0;
    lu8* scratch = lds + L_RED + p * RED_WAVE;
    const __amdgpu_buffer_rsrc_t rsW = __builtin_amdgcn_make_buffer_rsrc((void*)(myws + WS_WDATA + p * WSRC + (size_t)v * 8 * SLOT), 0, 8 * SLOT, 0x00020000);
    const __amdgpu_buffer_rsrc_t rsX = __builtin_amdgcn_make_buffer_rsrc((void*)(myws + WS_XDATA + p * XSRC + (size_t)u * 4 * SLOT), 0, 4 * SLOT, 0x00020000);
    const int tau = T_X0 + (((u * 8 + xB * 2 + xi) * D_X) >> 8);
    const bool early = tau <= T_SELF;
    u32 xfl = 0;
    auto xstores = [&]() {
        u32x4 xd[4]; u32 xexc;
        load_block(xd, xfl, xexc, lane, lds + L_XP, lds + L_XR, lds + L_XH);
        wait_until(tstart + (u64)tau);
#pragma unroll
        for (int k = 0; k < 4; ++k) {
            const int q = 4 * xi + ((k + g) & 3);
            gu8* dws = (gu8*)P.ws[q];
            if (q != rank) send_block(xd, xfl, xexc, lane, dws + WS_XDATA + rank * XSRC + (size_t)(u * 4 + xB) * SLOT, dws + WS_XEXC + (rank * 32 + u) * 128 + xB * 32);
        }
        __hip_atomic_store((lu32*)(lds + L_MISC + 88), 1u, __ATOMIC_RELAXED, __HIP_MEMORY_SCOPE_WORKGROUP);
    };
    auto xflags = [&]() {
        unsigned char* const q0 = P.ws[4 * xi], * const q1 = P.ws[4 * xi + 1], * const q2 = P.ws[4 * xi + 2], * const q3 = P.ws[4 * xi + 3];
        gu8* fq = (gu8*)(lane == 0 ? q0 : lane == 1 ? q1 : lane == 2 ? q2 : q3);
        if (lane < 4 && 4 * xi + lane != rank) flag_store(fq + WS_XFLAG + (rank * 32 + u) * 128 + xB * 8, fv | xfl);
    };
    if (self) {
        u64 fv2;
        gu8* fp = (lane & 8) ? myws + WS_XFLAG + (p * 32 + u) * 128 + (lane & 3) * 8 : myws + WS_WFLAG + (p * 8 + v) * 128 + (lane & 7) * 8;
        ok &= wait_flags(fp, call, tmo, fv2);
        fvalW = fv2; fvalX = (u64)(u32)__builtin_amdgcn_ds_bpermute((8 + (lane & 3)) * 4, (int)(u32)fv2);
        if (early) xstores();
    } else {
        const int tA = T_W0 + (((32 * v + 31) * D_W) >> 8) + T_POLL;
        wait_until(tstart + (u64)tA);
        ok &= wait_flags(myws + WS_WFLAG + (p * 8 + v) * 128 + (lane & 7) * 8, call, tmo, fvalW);
        if (tA + T_WD > tau) {
            const u64 t0 = rt();
            while (__hip_atomic_load((lu32*)(lds + L_MISC + 88), __ATOMIC_RELAXED, __HIP_MEMORY_SCOPE_WORKGROUP) == 0u && rt() - t0 < DEADLINE) __builtin_amdgcn_s_sleep(1);
        }
    }
    const int rot = ((u >> 1) + 2 * (v & 1)) & 3;
    u32x4 wf[8][4];
    u32x4 rawX[4][3]; u32 excX, tm;
    auto xloads = [&]() {
        excX = __hip_atomic_load((gu32*)(myws + WS_XEXC + (p * 32 + u) * 128 + (lane & 31) * 4), __ATOMIC_RELAXED, __HIP_MEMORY_SCOPE_SYSTEM);
        tm = __hip_atomic_load(tmo, __ATOMIC_RELAXED, __HIP_MEMORY_SCOPE_AGENT);
#pragma unroll
        for (int b = 0; b < 4; ++b)
#pragma unroll
            for (int k = 0; k < 3; ++k) rawX[b][k] = __builtin_amdgcn_raw_buffer_load_b128(rsX, lane * 16, ((b + rot) & 3) * (int)SLOT + k * 1024, 17);
        SB();
    };
    gu8* const xfp = myws + WS_XFLAG + (p * 32 + u) * 128 + (lane & 3) * 8;
    {
        const u32 excW = __hip_atomic_load((gu32*)(myws + WS_WEXC + (p * 8 + v) * 256 + lane * 4), __ATOMIC_RELAXED, __HIP_MEMORY_SCOPE_SYSTEM);
        u32x4 rawW[8][3];
        auto wl = [&](int b) {
            const int so = ((b + 2 * rot) & 7) * (int)SLOT;
#pragma unroll
            for (int k = 0; k < 3; ++k) rawW[b][k] = __builtin_amdgcn_raw_buffer_load_b128(rsW, lane * 16, so + k * 1024, 17);
        };
        auto wu = [&](int b) {
            const int bm = (b + 2 * rot) & 7;
            unpack_block(wf[b], rawW[b][0], rawW[b][1], rawW[b][2], rdl((u32)fvalW, bm), excW, bm, lane, scratch, rsW, bm * (int)SLOT);
        };
        bool xl = false; u64 fx = 0;
        auto xlook = [&]() { if (!xl) fx = flag_load(xfp); };
        auto xtry = [&]() { if (!xl && __all((u32)(fx >> 32) >= call)) { asm volatile("" ::: "memory"); fvalX = fx; xloads(); xl = true; } };
        wl(0); wl(1); wl(2); SB();
        if (self && early) { asm volatile("s_waitcnt vmcnt(9)" ::: "memory"); xflags(); }
        wu(0); SB(); wl(3); SB();
        wu(1); SB(); wl(4); SB();
        wu(2); SB(); wl(5); SB();
        wu(3); SB(); wl(6); SB();
        if (self) fx = fv | (u32)fvalX; else xlook();
        SB(); wu(4); SB(); wl(7); SB();
        xtry(); SB();
        wu(5); SB(); xlook(); SB();
        wu(6); SB(); xtry(); SB();
        wu(7);
        if (!xl) {
            wait_until(tstart + (u64)(T_X0 + (((u * 8 + 7) * D_X) >> 8) + T_POLL));
            ok &= wait_flags(xfp, call, tmo, fvalX);
            xloads();
        }
    }
    f32x16 acc[2];
#pragma unroll
    for (int i = 0; i < 16; ++i) { acc[0][i] = 0.f; acc[1][i] = 0.f; }
#pragma unroll
    for (int B = 0; B < 4; ++B) {
        u32x4 xf[4];
        const int bm = (B + rot) & 3;
        unpack_block(xf, rawX[B][0], rawX[B][1], rawX[B][2], rdl((u32)fvalX, bm), lane < 32 ? excX : EXC_NONE, bm, lane, scratch, rsX, bm * (int)SLOT);
#pragma unroll
        for (int f = 0; f < 4; ++f) {
            const int T = 4 * B + f;
            acc[0] = __builtin_amdgcn_mfma_f32_32x32x16_bf16(__builtin_bit_cast(bf16x8, wf[T >> 1][2 * (T & 1)]), __builtin_bit_cast(bf16x8, xf[f]), acc[0], 0, 0, 0);
            acc[1] = __builtin_amdgcn_mfma_f32_32x32x16_bf16(__builtin_bit_cast(bf16x8, wf[T >> 1][2 * (T & 1) + 1]), __builtin_bit_cast(bf16x8, xf[f]), acc[1], 0, 0, 0);
        }
    }
    if ((!ok || tm == call) && lane == 0) *(LAS u32*)(lds + L_MISC + 84) = 1u;
#pragma unroll
    for (int cb = 0; cb < 2; ++cb)
#pragma unroll
        for (int g4 = 0; g4 < 4; ++g4)
            *(LAS f32x4*)(lds + L_RED + p * RED_WAVE + r * RED_STRIDE + (32 * cb + 8 * g4 + 4 * h) * 4) =
                (f32x4){acc[cb][4 * g4], acc[cb][4 * g4 + 1], acc[cb][4 * g4 + 2], acc[cb][4 * g4 + 3]};
    if (self && !early) { xstores(); drain(); xflags(); }
    if (tid < 184) *(LAS u32x4*)(lds + L_IMG + 256 + tid * 16) = (u32x4){0, 0, 0, 0};
    __syncthreads();

    u32 o01, o23, pre, m4;
    {
        const int m = tid >> 4, n0 = 4 * (tid & 15);
        const bool bad = rfl(*(LAS u32*)(lds + L_MISC + 84)) != 0u;
        f32x4 s = *(LAS f32x4*)(lds + L_RED + m * RED_STRIDE + n0 * 4);
#pragma unroll
        for (int pp = 1; pp < 8; ++pp) s += *(LAS f32x4*)(lds + L_RED + pp * RED_WAVE + m * RED_STRIDE + n0 * 4);
        const f32x4 y = __builtin_elementwise_maximum(s, (f32x4){0.f, 0.f, 0.f, 0.f});
        o01 = __builtin_bit_cast(u32, __builtin_convertvector((f32x2){y.x, y.y}, bf16x2)) & 0x7FFF7FFFu;
        o23 = __builtin_bit_cast(u32, __builtin_convertvector((f32x2){y.z, y.w}, bf16x2)) & 0x7FFF7FFFu;
        if (bad) { o01 = 0x7FC07FC0u; o23 = 0x7FC07FC0u; }
        *(LAS u32x2*)(lds + L_OUT + tid * 8) = (u32x2){o01, o23};
        const u16x2 mm = __builtin_elementwise_max(__builtin_bit_cast(u16x2, o01), __builtin_bit_cast(u16x2, o23));
        const u32 wmx = wave_max(max((u32)mm.x, (u32)mm.y));
        const bool z0 = (o01 & 0xFFFFu) != 0u, z1 = o01 > 0xFFFFu, z2 = (o23 & 0xFFFFu) != 0u, z3 = o23 > 0xFFFFu;
        const u64 b0 = __ballot(z0), b1 = __ballot(z1), b2 = __ballot(z2), b3 = __ballot(z3);
        pre = mbcnt64(b3, mbcnt64(b2, mbcnt64(b1, mbcnt64(b0, 0u))));
        m4 = (z0 ? 1u : 0u) | (z1 ? 2u : 0u) | (z2 ? 4u : 0u) | (z3 ? 8u : 0u);
        if (lane == 0) *(LAS u32*)(lds + L_MISC + 16 + wv * 4) = ((wmx >> 8) << 16) | (u32)(__popcll(b0) + __popcll(b1) + __popcll(b2) + __popcll(b3));
    }
    __syncthreads();
    u32 ntot, obase;
    {
        const u32 w = *(LAS u32*)(lds + L_MISC + 16 + (lane & 7) * 4);
        const u32 sel = *(LAS u32*)(lds + L_LUT4 + m4 * 4);
        u32 cs = (w & 0xFFFFu) * ((lane & 7) < wv ? 0x10001u : 1u), mx = w >> 16;
        cs += dpp0<0x111, 0xf>(cs); mx = max(mx, dpp0<0x111, 0xf>(mx));
        cs += dpp0<0x112, 0xf>(cs); mx = max(mx, dpp0<0x112, 0xf>(mx));
        cs += dpp0<0x114, 0xf>(cs); mx = max(mx, dpp0<0x114, 0xf>(mx));
        const u32 cst = rdl(cs, 7), h7 = rdl(mx, 7);
        ntot = cst & 0xFFFFu; obase = h7 > 7u ? h7 - 7u : 0u;
        const u32 pos = (cst >> 16) + pre;
        const u32 L = perm(o23, o01, 0x06040200u), H = perm(o23, o01, 0x07050301u);
        const u32 d = H + (128u - obase) * 0x01010101u;
        const u32 Lc = perm(0u, L, sel), Cc = perm(0u, d & 0x07070707u, sel);
        const u32 c12 = __builtin_amdgcn_udot4(Cc, 0x00400801u, (Cc >> 15) & 0xE00u, false);
        if (Lc) {
            const u32 sh = (pos & 3u) * 8u;
            lu32* lp = (lu32*)(lds + L_IMG + 256 + (pos & ~3u));
            __hip_atomic_fetch_or(lp, Lc << sh, __ATOMIC_RELAXED, __HIP_MEMORY_SCOPE_WORKGROUP);
            const u32 hi = (u32)(((u64)Lc << sh) >> 32); if (hi) __hip_atomic_fetch_or(lp + 1, hi, __ATOMIC_RELAXED, __HIP_MEMORY_SCOPE_WORKGROUP);
        }
        if (c12) {
            const u32 bit = 3u * pos, sh = bit & 31u;
            lu32* cp = (lu32*)(lds + L_IMG + img_coff(ntot) + (bit >> 5) * 4);
            __hip_atomic_fetch_or(cp, c12 << sh, __ATOMIC_RELAXED, __HIP_MEMORY_SCOPE_WORKGROUP);
            const u32 hi = (u32)(((u64)c12 << sh) >> 32); if (hi) __hip_atomic_fetch_or(cp + 1, hi, __ATOMIC_RELAXED, __HIP_MEMORY_SCOPE_WORKGROUP);
        }
        if (~perm(0u, d, sel | ((sel & 0x08080808u) >> 3)) & 0x80808080u) {
#pragma unroll
            for (int j = 0; j < 4; ++j) {
                const u32 oj = ((j < 2 ? o01 : o23) >> (16 * (j & 1))) & 0xFFFFu;
                if (oj != 0u && (oj >> 8) < obase) {
                    const u32 e = __hip_atomic_fetch_add((lu32*)(lds + L_MISC + 80), 1u, __ATOMIC_RELAXED, __HIP_MEMORY_SCOPE_WORKGROUP);
                    if (e < 8u) *(LAS u32*)(lds + L_IMG + img_eoff(ntot) + e * 4) = ((u32)(tid * 4 + j) << 16) | oj;
                }
            }
        }
        const u32 other = (u32)__builtin_amdgcn_update_dpp(0, (int)m4, 0xB1  , 0xf, 0xf, false);
        const u32 gi = (u32)tid >> 1;
        if (!(lane & 1)) *(LAS unsigned char*)(lds + L_IMG + 4 * (gi & 63u) + (gi >> 6)) = (unsigned char)(m4 | (other << 4));
    }
    __syncthreads();

    const int tile = u * 8 + v;
    constexpr size_t PASS = (size_t)8 * N_ * 2;
    const u32 nexc = rfl(*(LAS u32*)(lds + L_MISC + 80));
    const bool raw = nexc > 8u;
    gu8* const out = (gu8*)P.out;
    if (self) {
        u32x4 d[4];
#pragma unroll
        for (int s = 0; s < 4; ++s) d[s] = *(LAS u32x4*)(lds + L_OUT + s * 1024 + lane * 16);
        gu8* dst = out + ((size_t)(1024 * (rank >> 2) + 32 * u + (lane >> 3)) * N_ + 512 * (rank & 3) + 64 * v) * 2 + (lane & 7) * 16;
#pragma unroll
        for (int s = 0; s < 4; ++s) st16_sys(dst + s * PASS, d[s]);
        unsigned char* q = P.ws[0];
#pragma unroll
        for (int i = 1; i < 8; ++i) q = (lane & 7) == i ? P.ws[i] : q;
        gu8* fp = (gu8*)q + WS_OFLAG + (rank * 256 + tile) * 128;
        const u64 fo = fv | (raw ? F_RAW : 0u) | (nexc ? F_EXC : 0u) | (obase << 12) | ntot;
        bool todo = lane < 8 && lane != rank;
        const u64 t0 = rt();
        while (__any(todo)) {
            const u32 sw = __hip_atomic_load((lu32*)(lds + L_MISC + 48 + (lane & 7) * 4), __ATOMIC_ACQUIRE, __HIP_MEMORY_SCOPE_WORKGROUP);
            if (todo && sw) { flag_store(fp, fo); todo = false; }
            if (rt() - t0 > DEADLINE) break;
        }
    } else {
        {
            const u32 nch = raw ? 256u : ((img_chunks(ntot, nexc != 0u) + 3u) & ~3u);
            lu8* S = lds + (raw ? L_OUT : L_IMG);
            gu8* dst = (gu8*)P.ws[wv] + WS_ODATA + rank * OSRC + (size_t)tile * SLOT + lane * 16;
            u32x4 d[4];
#pragma unroll
            for (int s = 0; s < 4; ++s) d[s] = *(LAS u32x4*)(S + s * 1024 + lane * 16);
            wait_until(tstart + (u64)(T_O0 + ((tile * D_O) >> 8)));
#pragma unroll
            for (int s = 0; s < 4; ++s) if ((u32)(lane + 64 * s) < nch) st16_sys(dst + s * 1024, d[s]);
            drain();
            if (lane == 0) __hip_atomic_store((lu32*)(lds + L_MISC + 48 + wv * 4), 1u, __ATOMIC_RELEASE, __HIP_MEMORY_SCOPE_WORKGROUP);
        }
        const int ur = (u + 1 - wa) & 31, rtile = ur * 8 + v;
        u64 fval;
        const bool ok2 = wait_flags(myws + WS_OFLAG + (wv * 256 + rtile) * 128, call, tmo, fval);
        const u32 fl = rfl((u32)fval);
        const u32 n = fl & 4095u, rb = (fl >> 12) & 127u;
        const bool rraw = (fl & F_RAW) != 0u;
        const u32 rch = rraw ? 256u : img_chunks(n, (fl & F_EXC) != 0u);
        const __amdgpu_buffer_rsrc_t rs = __builtin_amdgcn_make_buffer_rsrc((void*)(myws + WS_ODATA + wv * OSRC + (size_t)rtile * SLOT), 0, rch * 16, 0x00020000);
        const u32 mbw = __builtin_amdgcn_raw_buffer_load_b32(rs, lane * 4, 0, 17);
        u32x4 in[4];
#pragma unroll
        for (int s = 0; s < 4; ++s) in[s] = __builtin_amdgcn_raw_buffer_load_b128(rs, lane * 16 + s * 1024, 0, 17);
        const u32 tm2 = __hip_atomic_load(tmo, __ATOMIC_RELAXED, __HIP_MEMORY_SCOPE_AGENT);
        const int rrow0 = 1024 * (wv >> 2) + 32 * ur, rcol0 = 512 * (wv & 3) + 64 * v;
        gu8* odst = out + ((size_t)(rrow0 + (lane >> 3)) * N_ + rcol0) * 2 + (lane & 7) * 16;
        if (!ok2 || tm2 == call) {
#pragma unroll
            for (int s = 0; s < 4; ++s) st16_sys(odst + s * PASS, (u32x4){0x7FC07FC0u, 0x7FC07FC0u, 0x7FC07FC0u, 0x7FC07FC0u});
        } else if (rraw) {
#pragma unroll
            for (int s = 0; s < 4; ++s) st16_sys(odst + s * PASS, in[s]);
        } else {
            lu8* sc = lds + L_RED + wv * RED_WAVE;
            const u32 mb0 = mbw & 255u, mb1 = (mbw >> 8) & 255u, mb2 = (mbw >> 16) & 255u, mb3 = mbw >> 24;
            u32x2 sel[4];
            sel[0] = *(LAS u32x2*)(lds + L_LUT + mb0 * 8); sel[1] = *(LAS u32x2*)(lds + L_LUT + mb1 * 8);
            sel[2] = *(LAS u32x2*)(lds + L_LUT + mb2 * 8); sel[3] = *(LAS u32x2*)(lds + L_LUT + mb3 * 8);
            const u32 c01 = (u32)__popc(mb0) | ((u32)__popc(mb1) << 16), c23 = (u32)__popc(mb2) | ((u32)__popc(mb3) << 16);
            const u32 s01 = wave_scan_add(c01), s23 = wave_scan_add(c23);
#pragma unroll
            for (int s = 0; s < 4; ++s) *(LAS u32x4*)(sc + s * 1024 + lane * 16) = in[s];
            asm volatile("" ::: "memory");
            const u32 coff = img_coff(n);
            const u32 t01 = rdl(s01, 63), t23 = rdl(s23, 63);
            const u32 e01 = s01 - c01, e23 = s23 - c23;
            u32 off[4];
            off[0] = e01 & 0xFFFFu; off[1] = (t01 & 0xFFFFu) + (e01 >> 16);
            const u32 b2 = (t01 & 0xFFFFu) + (t01 >> 16);
            off[2] = b2 + (e23 & 0xFFFFu); off[3] = b2 + (t23 & 0xFFFFu) + (e23 >> 16);
            const u32 b4 = rb * 0x01010101u;
            u32 a0[4], a1[4], a2[4], k0[4], k1[4];
#pragma unroll
            for (int i = 0; i < 4; ++i) {
                lu32* lp = (lu32*)(sc + 256 + (off[i] & ~3u));
                a0[i] = lp[0]; a1[i] = lp[1]; a2[i] = lp[2];
                lu32* cp = (lu32*)(sc + coff + ((3u * off[i]) >> 5) * 4);
                k0[i] = cp[0]; k1[i] = cp[1];
            }
#pragma unroll
            for (int i = 0; i < 4; ++i) {
                const u32 lo0 = __builtin_amdgcn_alignbyte(a1[i], a0[i], off[i] & 3u), lo1 = __builtin_amdgcn_alignbyte(a2[i], a1[i], off[i] & 3u);
                const u32 c = __builtin_amdgcn_alignbit(k1[i], k0[i], (3u * off[i]) & 31u);
                u32 x0 = c & 0xFFFu, x1 = (c >> 12) & 0xFFFu;
                x0 = (x0 | (x0 << 10)) & 0x003F003Fu; x1 = (x1 | (x1 << 10)) & 0x003F003Fu;
                x0 = (x0 | (x0 << 5)) & 0x07070707u; x1 = (x1 | (x1 << 5)) & 0x07070707u;
                const u32 h0 = x0 + b4, h1 = x1 + b4;
                const u32 l0 = perm(lo1, lo0, sel[i].x), l1 = perm(lo1, lo0, sel[i].y), g0 = perm(h1, h0, sel[i].x), g1 = perm(h1, h0, sel[i].y);
                st16_sys(odst + i * PASS, (u32x4){perm(g0, l0, 0x05010400u), perm(g0, l0, 0x07030602u), perm(g1, l1, 0x05010400u), perm(g1, l1, 0x07030602u)});
            }
            if (fl & F_EXC) {
                drain();
                if (lane < 8) {
                    const u32 e = *(LAS u32*)(sc + img_eoff(n) + lane * 4);
                    if (e != 0u) { const u32 ix = (e >> 16) & 2047u; *(gu16*)(out + ((size_t)(rrow0 + (ix >> 6)) * N_ + rcol0 + (ix & 63u)) * 2) = (unsigned short)e; }
                }
            }
        }
    }
}

extern "C" void kernel_launch(int rank, int nranks, ncclComm_t comm, hipStream_t stream,
                              void* const* d_in, const int* in_sizes, int n_in,
                              void* d_out, int out_size, const int* in_dtypes, int out_dtype,
                              void* const* d_out_all, void* const* d_ws_all, size_t ws_bytes) {
    (void)comm; (void)d_out_all; (void)in_dtypes;
    if (nranks != NR || (unsigned)rank >= (unsigned)NR || ws_bytes < WS_END || n_in != 2 || in_sizes[0] != M_ * KS || in_sizes[1] != KS * N_ ||
        out_size != M_ * N_ || out_dtype != 1) {
        fprintf(stderr, "kernel_launch: unexpected problem shape\n");
        return;
    }
    static bool init = false;
    if (!init) {
        if (hipFuncSetAttribute((const void*)fused, hipFuncAttributeMaxDynamicSharedMemorySize, LDS_BYTES) != hipSuccess) fprintf(stderr, "kernel_launch: LDS attribute\n");
        init = true;
    }
    Params P;
    P.x = (const unsigned short*)d_in[0]; P.w = (const unsigned short*)d_in[1];
    for (int i = 0; i < NR; ++i) P.ws[i] = (unsigned char*)d_ws_all[i];
    P.out = (unsigned char*)d_out;
    P.rank = rank; P.pad = 0;
    hipLaunchKernelGGL(fused, dim3(NWG), dim3(NTHR), LDS_BYTES, stream, P);
}
```

```cpp
#include <hip/hip_runtime.h>
#include <rccl/rccl.h>
#include <cstdio>

#define GAS __attribute__((address_space(1)))
#define LAS __attribute__((address_space(3)))
typedef unsigned u32;
typedef unsigned long long u64;
typedef u32 u32x4 __attribute__((ext_vector_type(4)));
typedef u32 u32x2 __attribute__((ext_vector_type(2)));
typedef float f32x2 __attribute__((ext_vector_type(2)));
typedef float f32x4 __attribute__((ext_vector_type(4)));
typedef float f32x16 __attribute__((ext_vector_type(16)));
typedef __bf16 bf16x8 __attribute__((ext_vector_type(8)));
typedef __bf16 bf16x2 __attribute__((ext_vector_type(2)));
typedef unsigned short u16x2 __attribute__((ext_vector_type(2)));
typedef GAS u32 gu32; typedef GAS u64 gu64; typedef GAS unsigned char gu8; typedef GAS unsigned short gu16;
typedef LAS unsigned char lu8; typedef LAS u32 lu32;

constexpr int NR = 8, M_ = 2048, N_ = 2048, KS = 256;
constexpr int NWG = 256, NTHR = 512;

constexpr size_t WS_TMO = 0;
constexpr size_t WS_CNT = 4096;
constexpr size_t WS_XFLAG = WS_CNT + 256 * 128;
constexpr size_t WS_WFLAG = WS_XFLAG + 8 * 32 * 128;
constexpr size_t WS_OFLAG = WS_WFLAG + 8 * 8 * 128;
constexpr size_t WS_XEXC = WS_OFLAG + 8 * 256 * 128;
constexpr size_t WS_WEXC = WS_XEXC + 8 * 32 * 128;
constexpr size_t WS_FEND = WS_WEXC + 8 * 8 * 256;
constexpr size_t SLOT = 4224, SKEW = 20608;
constexpr size_t XSRC = 32 * 4 * SLOT + SKEW, WSRC = 8 * 8 * SLOT + SKEW, OSRC = 256 * SLOT + SKEW;
constexpr size_t WS_XDATA = 1u << 20;
constexpr size_t WS_WDATA = WS_XDATA + 8 * XSRC;
constexpr size_t WS_ODATA = WS_WDATA + 8 * WSRC;
constexpr size_t WS_END = WS_ODATA + 8 * OSRC;
static_assert(WS_FEND <= WS_XDATA && SLOT % 128 == 0 && SKEW % 128 == 0, "workspace layout");
constexpr u64 DEADLINE = 30000000ull;

constexpr int T_W0 = 85, D_W = 350;
constexpr int T_X0 = 435, D_X = 700;
constexpr int T_O0 = 1150, D_O = 795;
constexpr int T_POLL = 150;
constexpr int T_SELF = 800;
constexpr int T_WD = 380;

struct Params {
    const unsigned short* x; const unsigned short* w;
    unsigned char* ws[NR]; unsigned char* out;
    int rank, pad;
};

__device__ __forceinline__ u64 rt() { return __builtin_amdgcn_s_memrealtime(); }
__device__ __forceinline__ void st16_sys(gu8* p, u32x4 v) { asm volatile("global_store_dwordx4 %0, %1, off sc0 sc1\n\ts_nop 1" :: "v"(p), "v"(v) : "memory"); }
__device__ __forceinline__ void st4_sys(gu8* p, u32 v) { asm volatile("global_store_dword %0, %1, off sc0 sc1\n\ts_nop 1" :: "v"(p), "v"(v) : "memory"); }
__device__ __forceinline__ void drain() { asm volatile("s_waitcnt vmcnt(0)" ::: "memory"); }
__device__ __forceinline__ void flag_store(gu8* p, u64 v) { __hip_atomic_store((gu64*)p, v, __ATOMIC_RELAXED, __HIP_MEMORY_SCOPE_SYSTEM); }
__device__ __forceinline__ u64 flag_load(gu8* p) { return __hip_atomic_load((gu64*)p, __ATOMIC_RELAXED, __HIP_MEMORY_SCOPE_SYSTEM); }
__device__ __forceinline__ u32 rfl(u32 v) { return __builtin_amdgcn_readfirstlane(v); }
__device__ __forceinline__ u32 rdl(u32 v, int l) { return (u32)__builtin_amdgcn_readlane((int)v, l); }
__device__ __forceinline__ u32 perm(u32 hi, u32 lo, u32 sel) { return __builtin_amdgcn_perm(hi, lo, sel); }
__device__ __forceinline__ u32 rotr(u32 x, u32 s) { return __builtin_amdgcn_alignbit(x, x, s); }
__device__ __forceinline__ u32 mbcnt64(u64 m, u32 acc) { return __builtin_amdgcn_mbcnt_hi((u32)(m >> 32), __builtin_amdgcn_mbcnt_lo((u32)m, acc)); }
template <int CTRL, int RM> __device__ __forceinline__ u32 dpp0(u32 v) { return (u32)__builtin_amdgcn_update_dpp(0, (int)v, CTRL, RM, 0xf, false); }
__device__ __forceinline__ u32 wave_max(u32 v) {
    v = max(v, dpp0<0x111, 0xf>(v)); v = max(v, dpp0<0x112, 0xf>(v)); v = max(v, dpp0<0x114, 0xf>(v)); v = max(v, dpp0<0x118, 0xf>(v));
    v = max(v, dpp0<0x142, 0xa>(v)); v = max(v, dpp0<0x143, 0xc>(v));
    return rdl(v, 63);
}
__device__ __forceinline__ u32 wave_scan_add(u32 v) {
    v += dpp0<0x111, 0xf>(v); v += dpp0<0x112, 0xf>(v); v += dpp0<0x114, 0xf>(v); v += dpp0<0x118, 0xf>(v);
    v += dpp0<0x142, 0xa>(v); v += dpp0<0x143, 0xc>(v);
    return v;
}

__device__ __forceinline__ bool wait_flags(gu8* p, u32 call, gu32* tmo, u64& val) {
    const u64 t0 = rt();
    for (u32 it = 1;; ++it) {
        const u64 v = flag_load(p);
        if (__all((u32)(v >> 32) >= call)) { val = v; asm volatile("" ::: "memory"); return true; }
        if ((it & 31u) == 0u) {
            if (__hip_atomic_load(tmo, __ATOMIC_RELAXED, __HIP_MEMORY_SCOPE_AGENT) == call || rt() - t0 > DEADLINE) {
                __hip_atomic_store(tmo, call, __ATOMIC_RELAXED, __HIP_MEMORY_SCOPE_AGENT);
                val = 0; asm volatile("" ::: "memory"); return false;
            }
        }
        __builtin_amdgcn_s_sleep(2);
    }
}
__device__ __forceinline__ void wait_until(u64 t) { while (rt() < t) __builtin_amdgcn_s_sleep(1); }

constexpr int RED_STRIDE = 272;
constexpr int RED_WAVE = 32 * RED_STRIDE;
constexpr int L_RED = 0;
constexpr int L_XP = 8 * RED_WAVE;
constexpr int L_WP = L_XP + 3072;
constexpr int L_XR = L_WP + 3072;
constexpr int L_WR = L_XR + 4096;
constexpr int L_XH = L_WR + 4096;
constexpr int L_WH = L_XH + 64;
constexpr int L_OUT = L_WH + 64;
constexpr int L_IMG = L_OUT + 4096;
constexpr int L_LUT = L_IMG + 3200;
constexpr int L_LUT4 = L_LUT + 2048;
constexpr int L_MISC = L_LUT4 + 64;
constexpr int LDS_BYTES = L_MISC + 128;
static_assert(LDS_BYTES > 81920 && LDS_BYTES <= 163840 && L_XP % 16 == 0 && L_IMG % 16 == 0 && L_LUT % 16 == 0 && L_MISC % 16 == 0, "lds");

constexpr u32 F_RAW = 1u << 31, F_EXC = 1u << 30;
constexpr u32 EXC_NONE = 0xFFFFFFFFu;
__device__ __forceinline__ u32 img_coff(u32 n) { return 256u + ((n + 3u) & ~3u); }
__device__ __forceinline__ u32 img_eoff(u32 n) { return img_coff(n) + ((((3u * n + 7u) >> 3) + 3u) & ~3u); }
__device__ __forceinline__ u32 img_chunks(u32 n, bool exc) { return (img_eoff(n) + (exc ? 32u : 0u) + 15u) >> 4; }

__device__ __forceinline__ void pack_frag(u32x4 e, int f, int lane, lu8* P, lu8* R, lu8* H) {
    *(LAS u32x4*)(R + f * 1024 + lane * 16) = e;
    const u16x2 m01 = __builtin_elementwise_max(__builtin_bit_cast(u16x2, e.x & 0x7FFF7FFFu), __builtin_bit_cast(u16x2, e.y & 0x7FFF7FFFu));
    const u16x2 m23 = __builtin_elementwise_max(__builtin_bit_cast(u16x2, e.z & 0x7FFF7FFFu), __builtin_bit_cast(u16x2, e.w & 0x7FFF7FFFu));
    const u16x2 mm = __builtin_elementwise_max(m01, m23);
    const u32 mx = wave_max(max((u32)mm.x, (u32)mm.y));
    const u32 h7 = mx >> 8, base = h7 > 7u ? h7 - 7u : 0u;
    const u32 lo0 = perm(e.y, e.x, 0x06040200u), lo1 = perm(e.w, e.z, 0x06040200u);
    const u32 hi0 = perm(e.y, e.x, 0x07050301u), hi1 = perm(e.w, e.z, 0x07050301u);
    const u32 K = (128u - base) * 0x01010101u;
    const u32 d0 = (hi0 & 0x7F7F7F7Fu) + K, d1 = (hi1 & 0x7F7F7F7Fu) + K;
    const u32 cA = (hi0 & 0x80808080u) | (d0 & 0x07070707u), cB = (hi1 & 0x80808080u) | (d1 & 0x07070707u);
    *(LAS u32x2*)(P + (f >> 1) * 1024 + lane * 16 + (f & 1) * 8) = (u32x2){lo0, lo1};
    *(LAS u32*)(P + 2048 + lane * 16 + f * 4) = cA | rotr(cB, 28);
    const u32 x0 = ~d0 & 0x80808080u, x1 = ~d1 & 0x80808080u;
    u32 hdr = base;
    if (lane < 2) *(LAS u32*)(H + 16 + f * 8 + lane * 4) = EXC_NONE;
    asm volatile("" ::: "memory");
    if (__any((x0 | x1) != 0u)) {
        u32 cnt = 0;
#pragma unroll
        for (int j = 0; j < 8; ++j) {
            const bool ex = (((j < 4 ? x0 : x1) >> (8 * (j & 3) + 7)) & 1u) != 0u;
            const u64 b = __ballot(ex);
            const u32 rk = mbcnt64(b, cnt);
            const u32 val = (e[j >> 1] >> (16 * (j & 1))) & 0xFFFFu;
            if (ex && rk < 2u) *(LAS u32*)(H + 16 + f * 8 + rk * 4) = ((u32)(f * 512 + lane * 8 + j) << 16) | val;
            cnt += (u32)__popcll(b);
        }
        hdr |= cnt > 2u ? 0x300u : 0x100u;
    }
    if (lane == 0) *(LAS u32*)(H + f * 4) = hdr;
}
__device__ __forceinline__ void load_block(u32x4 (&d)[4], u32& fl, u32& excw, int lane, lu8* P, lu8* R, lu8* H) {
    const u32x4 h = *(LAS u32x4*)H;
    const u32 any = rfl(h.x | h.y | h.z | h.w);
    const bool raw = (any & 0x200u) != 0u;
    fl = (rfl(h.x) & 127u) | ((rfl(h.y) & 127u) << 7) | ((rfl(h.z) & 127u) << 14) | ((rfl(h.w) & 127u) << 21) | (raw ? F_RAW : ((any & 0x100u) ? F_EXC : 0u));
    lu8* S = raw ? R : P;
    d[0] = *(LAS u32x4*)(S + lane * 16); d[1] = *(LAS u32x4*)(S + 1024 + lane * 16); d[2] = *(LAS u32x4*)(S + 2048 + lane * 16);
    d[3] = *(LAS u32x4*)(R + 3072 + lane * 16);
    excw = *(LAS u32*)(H + 16 + (lane & 7) * 4);
}
__device__ __forceinline__ void send_block(const u32x4 (&d)[4], u32 fl, u32 excw, int lane, gu8* dst, gu8* dexc) {
    st16_sys(dst + lane * 16, d[0]); st16_sys(dst + 1024 + lane * 16, d[1]); st16_sys(dst + 2048 + lane * 16, d[2]);
    if (fl & F_RAW) st16_sys(dst + 3072 + lane * 16, d[3]);
    if ((fl & F_EXC) && lane < 8) st4_sys(dexc + lane * 4, excw);
}
__device__ __forceinline__ u32x4 unpack_frag(u32 lo0, u32 lo1, u32 nw, u32 b4) {
    const u32 hA = (nw & 0x87878787u) + b4, hB = (rotr(nw, 4) & 0x87878787u) + b4;
    return (u32x4){perm(hA, lo0, 0x05010400u), perm(hA, lo0, 0x07030602u), perm(hB, lo1, 0x05010400u), perm(hB, lo1, 0x07030602u)};
}
__device__ __forceinline__ void unpack_block(u32x4 (&fr)[4], u32x4 L0, u32x4 L1, u32x4 Nw, u32 fl, u32 excw, int blk, int lane, lu8* scratch,
                                             __amdgpu_buffer_rsrc_t rs, int soff) {
    if (fl & F_RAW) {
        fr[0] = L0; fr[1] = L1; fr[2] = Nw;
        fr[3] = __builtin_amdgcn_raw_buffer_load_b128(rs, lane * 16, soff + 3072, 17);
        asm volatile("" : "+v"(fr[3]));
        return;
    }
    fr[0] = unpack_frag(L0.x, L0.y, Nw.x, (fl & 127u) * 0x01010101u);
    fr[1] = unpack_frag(L0.z, L0.w, Nw.y, ((fl >> 7) & 127u) * 0x01010101u);
    fr[2] = unpack_frag(L1.x, L1.y, Nw.z, ((fl >> 14) & 127u) * 0x01010101u);
    fr[3] = unpack_frag(L1.z, L1.w, Nw.w, ((fl >> 21) & 127u) * 0x01010101u);
    if (fl & F_EXC) {
#pragma unroll
        for (int f = 0; f < 4; ++f) *(LAS u32x4*)(scratch + f * 1024 + lane * 16) = fr[f];
        asm volatile("" ::: "memory");
        if ((lane >> 3) == blk && excw != EXC_NONE) *(LAS unsigned short*)(scratch + ((excw >> 16) & 2047u) * 2) = (unsigned short)excw;
        asm volatile("" ::: "memory");
#pragma unroll
        for (int f = 0; f < 4; ++f) fr[f] = *(LAS u32x4*)(scratch + f * 1024 + lane * 16);
        asm volatile("" : "+v"(fr[0]), "+v"(fr[1]), "+v"(fr[2]), "+v"(fr[3]));
    }
}

#define SB() __builtin_amdgcn_sched_barrier(0)

__global__ void __launch_bounds__(NTHR, 2) fused(Params P) {
    extern __shared__ __attribute__((aligned(16))) unsigned char smem[];
    lu8* lds = (lu8*)smem;
    const int tid = threadIdx.x, lane = tid & 63, wv = __builtin_amdgcn_readfirstlane(tid >> 6);
    const int g = blockIdx.x, xcd = g & 7, idx = g >> 3, u = 2 * (idx >> 1) + (xcd >> 2), v = 2 * (xcd & 3) + (idx & 1);
    const int rank = P.rank;
    gu8* myws = (gu8*)P.ws[rank];
    const u64 tstart = rt();
    gu32* tmo = (gu32*)(myws + WS_TMO);
    gu32* cntp = (gu32*)(myws + WS_CNT + g * 128);
    u32 call0 = 0;
    if (tid == 0) call0 = __hip_atomic_load(cntp, __ATOMIC_RELAXED, __HIP_MEMORY_SCOPE_AGENT);

    const int xi = v >> 2, xB = v & 3;
    const int wV = g >> 3, wb = g & 7;
    const int r = lane & 31, h = lane >> 5;
    {
        u32x4 frag;
        if (wv < 4) {
            frag = *(const GAS u32x4*)((const gu16*)P.x + (size_t)(1024 * xi + 32 * u + r) * KS + 64 * xB + 16 * wv + 8 * h);
            pack_frag(frag, wv, lane, lds + L_XP, lds + L_XR, lds + L_XH);
        } else {
            const int f = wv - 4;
            const gu16* src = (const gu16*)P.w + (size_t)(32 * wb + 16 * (f >> 1) + 8 * h) * N_ + 64 * wV + 32 * (f & 1) + r;
            u32 e[8];
#pragma unroll
            for (int j = 0; j < 8; ++j) e[j] = src[(size_t)j * N_];
            frag = (u32x4){e[0] | (e[1] << 16), e[2] | (e[3] << 16), e[4] | (e[5] << 16), e[6] | (e[7] << 16)};
            pack_frag(frag, f, lane, lds + L_WP, lds + L_WR, lds + L_WH);
        }
    }
    if (tid == 0) { call0 += 1u; *(LAS u32*)(lds + L_MISC) = call0; __hip_atomic_store(cntp, call0, __ATOMIC_RELAXED, __HIP_MEMORY_SCOPE_AGENT); }
    if (tid >= 64 && tid < 64 + 11) *(LAS u32*)(lds + L_MISC + 48 + (tid - 64) * 4) = 0u;
    __syncthreads();
    const u32 call = rfl(*(LAS u32*)(lds + L_MISC));
    const u64 fv = (u64)call << 32;
    const int wa = (wv - rank) & 7;
    if (wa == 1 || wa == 2) {
        const int q = (wV >> 3) + 4 * (wa - 1), vv = wV & 7;
        gu8* dws = (gu8*)P.ws[q];
        u32x4 d[4]; u32 fl, excw;
        load_block(d, fl, excw, lane, lds + L_WP, lds + L_WR, lds + L_WH);
        if (q != rank) wait_until(tstart + (u64)(T_W0 + ((((g & 63) * 4 + (g >> 6)) * D_W) >> 8)));
        send_block(d, fl, excw, lane, dws + WS_WDATA + rank * WSRC + (size_t)(vv * 8 + wb) * SLOT, dws + WS_WEXC + (rank * 8 + vv) * 256 + wb * 32);
        drain();
        if (lane == 0) flag_store(dws + WS_WFLAG + (rank * 8 + vv) * 128 + wb * 8, fv | fl);
    } else if (wa == 3) {
        if (xi == (rank >> 2)) {
            u32x4 d[4]; u32 fl, excw;
            load_block(d, fl, excw, lane, lds + L_XP, lds + L_XR, lds + L_XH);
            send_block(d, fl, excw, lane, myws + WS_XDATA + rank * XSRC + (size_t)(u * 4 + xB) * SLOT, myws + WS_XEXC + (rank * 32 + u) * 128 + xB * 32);
            drain();
            if (lane == 0) flag_store(myws + WS_XFLAG + (rank * 32 + u) * 128 + xB * 8, fv | fl);
        }
    } else if (wa >= 4) {
        const u32 m = (u32)(wa - 4) * 64u + (u32)lane;
        u32 s0 = 0, s1 = 0, c = 0;
#pragma unroll
        for (int j = 0; j < 8; ++j) { const u32 b = (m >> j) & 1u; const u32 sv = b ? c : 0x0Cu; c += b; if (j < 4) s0 |= sv << (8 * j); else s1 |= sv << (8 * (j - 4)); }
        *(LAS u32x2*)(lds + L_LUT + m * 8) = (u32x2){s0, s1};
        if (m < 16u) {
            u32 sel = 0x0C0C0C0Cu, k = 0;
#pragma unroll
            for (int j = 0; j < 4; ++j) if ((m >> j) & 1u) { sel = (sel & ~(0xFFu << (8 * k))) | ((u32)j << (8 * k)); k++; }
            *(LAS u32*)(lds + L_LUT4 + m * 4) = sel;
        }
    }

    bool ok = true; u64 fvalW, fvalX;
    const int p = wv;
    const bool self = wa == 0;
    lu8* scratch = lds + L_RED + p * RED_WAVE;
    const __amdgpu_buffer_rsrc_t rsW = __builtin_amdgcn_make_buffer_rsrc((void*)(myws + WS_WDATA + p * WSRC + (size_t)v * 8 * SLOT), 0, 8 * SLOT, 0x00020000);
    const __amdgpu_buffer_rsrc_t rsX = __builtin_amdgcn_make_buffer_rsrc((void*)(myws + WS_XDATA + p * XSRC + (size_t)u * 4 * SLOT), 0, 4 * SLOT, 0x00020000);
    const int tau = T_X0 + (((u * 8 + xB * 2 + xi) * D_X) >> 8);
    const bool early = tau <= T_SELF;
    u32 xfl = 0;
    auto xstores = [&]() {
        u32x4 xd[4]; u32 xexc;
        load_block(xd, xfl, xexc, lane, lds + L_XP, lds + L_XR, lds + L_XH);
        wait_until(tstart + (u64)tau);
#pragma unroll
        for (int k = 0; k < 4; ++k) {
            const int q = 4 * xi + ((k + g) & 3);
            gu8* dws = (gu8*)P.ws[q];
            if (q != rank) send_block(xd, xfl, xexc, lane, dws + WS_XDATA + rank * XSRC + (size_t)(u * 4 + xB) * SLOT, dws + WS_XEXC + (rank * 32 + u) * 128 + xB * 32);
        }
        __hip_atomic_store((lu32*)(lds + L_MISC + 88), 1u, __ATOMIC_RELAXED, __HIP_MEMORY_SCOPE_WORKGROUP);
    };
    auto xflags = [&]() {
        unsigned char* const q0 = P.ws[4 * xi], * const q1 = P.ws[4 * xi + 1], * const q2 = P.ws[4 * xi + 2], * const q3 = P.ws[4 * xi + 3];
        gu8* fq = (gu8*)(lane == 0 ? q0 : lane == 1 ? q1 : lane == 2 ? q2 : q3);
        if (lane < 4 && 4 * xi + lane != rank) flag_store(fq + WS_XFLAG + (rank * 32 + u) * 128 + xB * 8, fv | xfl);
    };
    if (self) {
        u64 fv2;
        gu8* fp = (lane & 8) ? myws + WS_XFLAG + (p * 32 + u) * 128 + (lane & 3) * 8 : myws + WS_WFLAG + (p * 8 + v) * 128 + (lane & 7) * 8;
        ok &= wait_flags(fp, call, tmo, fv2);
        fvalW = fv2; fvalX = (u64)(u32)__builtin_amdgcn_ds_bpermute((8 + (lane & 3)) * 4, (int)(u32)fv2);
        if (early) xstores();
    } else {
        const int tA = T_W0 + (((32 * v + 31) * D_W) >> 8) + T_POLL;
        wait_until(tstart + (u64)tA);
        ok &= wait_flags(myws + WS_WFLAG + (p * 8 + v) * 128 + (lane & 7) * 8, call, tmo, fvalW);
        if (tA + T_WD > tau) {
            const u64 t0 = rt();
            while (__hip_atomic_load((lu32*)(lds + L_MISC + 88), __ATOMIC_RELAXED, __HIP_MEMORY_SCOPE_WORKGROUP) == 0u && rt() - t0 < DEADLINE) __builtin_amdgcn_s_sleep(1);
        }
    }
    const int rot = ((u >> 1) + 2 * (v & 1)) & 3;
    u32x4 wf[8][4];
    u32x4 rawX[4][3]; u32 excX, tm;
    auto xloads = [&]() {
        excX = __hip_atomic_load((gu32*)(myws + WS_XEXC + (p * 32 + u) * 128 + (lane & 31) * 4), __ATOMIC_RELAXED, __HIP_MEMORY_SCOPE_SYSTEM);
        tm = __hip_atomic_load(tmo, __ATOMIC_RELAXED, __HIP_MEMORY_SCOPE_AGENT);
#pragma unroll
        for (int b = 0; b < 4; ++b)
#pragma unroll
            for (int k = 0; k < 3; ++k) rawX[b][k] = __builtin_amdgcn_raw_buffer_load_b128(rsX, lane * 16, ((b + rot) & 3) * (int)SLOT + k * 1024, 17);
        SB();
    };
    gu8* const xfp = myws + WS_XFLAG + (p * 32 + u) * 128 + (lane & 3) * 8;
    {
        const u32 excW = __hip_atomic_load((gu32*)(myws + WS_WEXC + (p * 8 + v) * 256 + lane * 4), __ATOMIC_RELAXED, __HIP_MEMORY_SCOPE_SYSTEM);
        u32x4 rawW[8][3];
        auto wl = [&](int b) {
            const int so = ((b + 2 * rot) & 7) * (int)SLOT;
#pragma unroll
            for (int k = 0; k < 3; ++k) rawW[b][k] = __builtin_amdgcn_raw_buffer_load_b128(rsW, lane * 16, so + k * 1024, 17);
        };
        auto wu = [&](int b) {
            const int bm = (b + 2 * rot) & 7;
            unpack_block(wf[b], rawW[b][0], rawW[b][1], rawW[b][2], rdl((u32)fvalW, bm), excW, bm, lane, scratch, rsW, bm * (int)SLOT);
        };
        bool xl = false; u64 fx = 0;
        auto xlook = [&]() { if (!xl) fx = flag_load(xfp); };
        auto xtry = [&]() { if (!xl && __all((u32)(fx >> 32) >= call)) { asm volatile("" ::: "memory"); fvalX = fx; xloads(); xl = true; } };
        wl(0); wl(1); wl(2); SB();
        if (self && early) { asm volatile("s_waitcnt vmcnt(9)" ::: "memory"); xflags(); }
        wu(0); SB(); wl(3); SB();
        wu(1); SB(); wl(4); SB();
        wu(2); SB(); wl(5); SB();
        wu(3); SB(); wl(6); SB();
        if (self) fx = fv | (u32)fvalX; else xlook();
        SB(); wu(4); SB(); wl(7); SB();
        xtry(); SB();
        wu(5); SB(); xlook(); SB();
        wu(6); SB(); xtry(); SB();
        wu(7);
        if (!xl) {
            wait_until(tstart + (u64)(T_X0 + (((u * 8 + 7) * D_X) >> 8) + T_POLL));
            ok &= wait_flags(xfp, call, tmo, fvalX);
            xloads();
        }
    }
    f32x16 acc[2];
#pragma unroll
    for (int i = 0; i < 16; ++i) { acc[0][i] = 0.f; acc[1][i] = 0.f; }
#pragma unroll
    for (int B = 0; B < 4; ++B) {
        u32x4 xf[4];
        const int bm = (B + rot) & 3;
        unpack_block(xf, rawX[B][0], rawX[B][1], rawX[B][2], rdl((u32)fvalX, bm), lane < 32 ? excX : EXC_NONE, bm, lane, scratch, rsX, bm * (int)SLOT);
#pragma unroll
        for (int f = 0; f < 4; ++f) {
            const int T = 4 * B + f;
            acc[0] = __builtin_amdgcn_mfma_f32_32x32x16_bf16(__builtin_bit_cast(bf16x8, wf[T >> 1][2 * (T & 1)]), __builtin_bit_cast(bf16x8, xf[f]), acc[0], 0, 0, 0);
            acc[1] = __builtin_amdgcn_mfma_f32_32x32x16_bf16(__builtin_bit_cast(bf16x8, wf[T >> 1][2 * (T & 1) + 1]), __builtin_bit_cast(bf16x8, xf[f]), acc[1], 0, 0, 0);
        }
    }
    if ((!ok || tm == call) && lane == 0) *(LAS u32*)(lds + L_MISC + 84) = 1u;
#pragma unroll
    for (int cb = 0; cb < 2; ++cb)
#pragma unroll
        for (int g4 = 0; g4 < 4; ++g4)
            *(LAS f32x4*)(lds + L_RED + p * RED_WAVE + r * RED_STRIDE + (32 * cb + 8 * g4 + 4 * h) * 4) =
                (f32x4){acc[cb][4 * g4], acc[cb][4 * g4 + 1], acc[cb][4 * g4 + 2], acc[cb][4 * g4 + 3]};
    if (self && !early) { xstores(); drain(); xflags(); }
    if (tid < 184) *(LAS u32x4*)(lds + L_IMG + 256 + tid * 16) = (u32x4){0, 0, 0, 0};
    __syncthreads();

    u32 o01, o23, pre, m4;
    {
        const int m = tid >> 4, n0 = 4 * (tid & 15);
        const bool bad = rfl(*(LAS u32*)(lds + L_MISC + 84)) != 0u;
        f32x4 s = *(LAS f32x4*)(lds + L_RED + m * RED_STRIDE + n0 * 4);
#pragma unroll
        for (int pp = 1; pp < 8; ++pp) s += *(LAS f32x4*)(lds + L_RED + pp * RED_WAVE + m * RED_STRIDE + n0 * 4);
        const f32x4 y = __builtin_elementwise_maximum(s, (f32x4){0.f, 0.f, 0.f, 0.f});
        o01 = __builtin_bit_cast(u32, __builtin_convertvector((f32x2){y.x, y.y}, bf16x2)) & 0x7FFF7FFFu;
        o23 = __builtin_bit_cast(u32, __builtin_convertvector((f32x2){y.z, y.w}, bf16x2)) & 0x7FFF7FFFu;
        if (bad) { o01 = 0x7FC07FC0u; o23 = 0x7FC07FC0u; }
        *(LAS u32x2*)(lds + L_OUT + tid * 8) = (u32x2){o01, o23};
        const u16x2 mm = __builtin_elementwise_max(__builtin_bit_cast(u16x2, o01), __builtin_bit_cast(u16x2, o23));
        const u32 wmx = wave_max(max((u32)mm.x, (u32)mm.y));
        const bool z0 = (o01 & 0xFFFFu) != 0u, z1 = o01 > 0xFFFFu, z2 = (o23 & 0xFFFFu) != 0u, z3 = o23 > 0xFFFFu;
        const u64 b0 = __ballot(z0), b1 = __ballot(z1), b2 = __ballot(z2), b3 = __ballot(z3);
        pre = mbcnt64(b3, mbcnt64(b2, mbcnt64(b1, mbcnt64(b0, 0u))));
        m4 = (z0 ? 1u : 0u) | (z1 ? 2u : 0u) | (z2 ? 4u : 0u) | (z3 ? 8u : 0u);
        if (lane == 0) *(LAS u32*)(lds + L_MISC + 16 + wv * 4) = ((wmx >> 8) << 16) | (u32)(__popcll(b0) + __popcll(b1) + __popcll(b2) + __popcll(b3));
    }
    __syncthreads();
    u32 ntot, obase;
    {
        const u32 w = *(LAS u32*)(lds + L_MISC + 16 + (lane & 7) * 4);
        const u32 sel = *(LAS u32*)(lds + L_LUT4 + m4 * 4);
        u32 cs = (w & 0xFFFFu) * ((lane & 7) < wv ? 0x10001u : 1u), mx = w >> 16;
        cs += dpp0<0x111, 0xf>(cs); mx = max(mx, dpp0<0x111, 0xf>(mx));
        cs += dpp0<0x112, 0xf>(cs); mx = max(mx, dpp0<0x112, 0xf>(mx));
        cs += dpp0<0x114, 0xf>(cs); mx = max(mx, dpp0<0x114, 0xf>(mx));
        const u32 cst = rdl(cs, 7), h7 = rdl(mx, 7);
        ntot = cst & 0xFFFFu; obase = h7 > 7u ? h7 - 7u : 0u;
        const u32 pos = (cst >> 16) + pre;
        const u32 L = perm(o23, o01, 0x06040200u), H = perm(o23, o01, 0x07050301u);
        const u32 d = H + (128u - obase) * 0x01010101u;
        const u32 Lc = perm(0u, L, sel), Cc = perm(0u, d & 0x07070707u, sel);
        const u32 c12 = __builtin_amdgcn_udot4(Cc, 0x00400801u, (Cc >> 15) & 0xE00u, false);
        if (Lc) {
            const u32 sh = (pos & 3u) * 8u;
            lu32* lp = (lu32*)(lds + L_IMG + 256 + (pos & ~3u));
            __hip_atomic_fetch_or(lp, Lc << sh, __ATOMIC_RELAXED, __HIP_MEMORY_SCOPE_WORKGROUP);
            const u32 hi = (u32)(((u64)Lc << sh) >> 32); if (hi) __hip_atomic_fetch_or(lp + 1, hi, __ATOMIC_RELAXED, __HIP_MEMORY_SCOPE_WORKGROUP);
        }
        if (c12) {
            const u32 bit = 3u * pos, sh = bit & 31u;
            lu32* cp = (lu32*)(lds + L_IMG + img_coff(ntot) + (bit >> 5) * 4);
            __hip_atomic_fetch_or(cp, c12 << sh, __ATOMIC_RELAXED, __HIP_MEMORY_SCOPE_WORKGROUP);
            const u32 hi = (u32)(((u64)c12 << sh) >> 32); if (hi) __hip_atomic_fetch_or(cp + 1, hi, __ATOMIC_RELAXED, __HIP_MEMORY_SCOPE_WORKGROUP);
        }
        if (~perm(0u, d, sel | ((sel & 0x08080808u) >> 3)) & 0x80808080u) {
#pragma unroll
            for (int j = 0; j < 4; ++j) {
                const u32 oj = ((j < 2 ? o01 : o23) >> (16 * (j & 1))) & 0xFFFFu;
                if (oj != 0u && (oj >> 8) < obase) {
                    const u32 e = __hip_atomic_fetch_add((lu32*)(lds + L_MISC + 80), 1u, __ATOMIC_RELAXED, __HIP_MEMORY_SCOPE_WORKGROUP);
                    if (e < 8u) *(LAS u32*)(lds + L_IMG + img_eoff(ntot) + e * 4) = ((u32)(tid * 4 + j) << 16) | oj;
                }
            }
        }
        const u32 other = (u32)__builtin_amdgcn_update_dpp(0, (int)m4, 0xB1  , 0xf, 0xf, false);
        const u32 gi = (u32)tid >> 1;
        if (!(lane & 1)) *(LAS unsigned char*)(lds + L_IMG + 4 * (gi & 63u) + (gi >> 6)) = (unsigned char)(m4 | (other << 4));
    }
    __syncthreads();

    const int tile = u * 8 + v;
    constexpr size_t PASS = (size_t)8 * N_ * 2;
    const u32 nexc = rfl(*(LAS u32*)(lds + L_MISC + 80));
    const bool raw = nexc > 8u;
    gu8* const out = (gu8*)P.out;
    if (self) {
        u32x4 d[4];
#pragma unroll
        for (int s = 0; s < 4; ++s) d[s] = *(LAS u32x4*)(lds + L_OUT + s * 1024 + lane * 16);
        gu8* dst = out + ((size_t)(1024 * (rank >> 2) + 32 * u + (lane >> 3)) * N_ + 512 * (rank & 3) + 64 * v) * 2 + (lane & 7) * 16;
#pragma unroll
        for (int s = 0; s < 4; ++s) st16_sys(dst + s * PASS, d[s]);
        unsigned char* q = P.ws[0];
#pragma unroll
        for (int i = 1; i < 8; ++i) q = (lane & 7) == i ? P.ws[i] : q;
        gu8* fp = (gu8*)q + WS_OFLAG + (rank * 256 + tile) * 128;
        const u64 fo = fv | (raw ? F_RAW : 0u) | (nexc ? F_EXC : 0u) | (obase << 12) | ntot;
        bool todo = lane < 8 && lane != rank;
        const u64 t0 = rt();
        while (__any(todo)) {
            const u32 sw = __hip_atomic_load((lu32*)(lds + L_MISC + 48 + (lane & 7) * 4), __ATOMIC_ACQUIRE, __HIP_MEMORY_SCOPE_WORKGROUP);
            if (todo && sw) { flag_store(fp, fo); todo = false; }
            if (rt() - t0 > DEADLINE) break;
        }
    } else {
        {
            const u32 nch = raw ? 256u : ((img_chunks(ntot, nexc != 0u) + 3u) & ~3u);
            lu8* S = lds + (raw ? L_OUT : L_IMG);
            gu8* dst = (gu8*)P.ws[wv] + WS_ODATA + rank * OSRC + (size_t)tile * SLOT + lane * 16;
            u32x4 d[4];
#pragma unroll
            for (int s = 0; s < 4; ++s) d[s] = *(LAS u32x4*)(S + s * 1024 + lane * 16);
            wait_until(tstart + (u64)(T_O0 + ((tile * D_O) >> 8)));
#pragma unroll
            for (int s = 0; s < 4; ++s) if ((u32)(lane + 64 * s) < nch) st16_sys(dst + s * 1024, d[s]);
            drain();
            if (lane == 0) __hip_atomic_store((lu32*)(lds + L_MISC + 48 + wv * 4), 1u, __ATOMIC_RELEASE, __HIP_MEMORY_SCOPE_WORKGROUP);
        }
        const int ur = (u + 1 - wa) & 31, rtile = ur * 8 + v;
        u64 fval;
        const bool ok2 = wait_flags(myws + WS_OFLAG + (wv * 256 + rtile) * 128, call, tmo, fval);
        const u32 fl = rfl((u32)fval);
        const u32 n = fl & 4095u, rb = (fl >> 12) & 127u;
        const bool rraw = (fl & F_RAW) != 0u;
        const u32 rch = rraw ? 256u : img_chunks(n, (fl & F_EXC) != 0u);
        const __amdgpu_buffer_rsrc_t rs = __builtin_amdgcn_make_buffer_rsrc((void*)(myws + WS_ODATA + wv * OSRC + (size_t)rtile * SLOT), 0, rch * 16, 0x00020000);
        const u32 mbw = __builtin_amdgcn_raw_buffer_load_b32(rs, lane * 4, 0, 17);
        u32x4 in[4];
#pragma unroll
        for (int s = 0; s < 4; ++s) in[s] = __builtin_amdgcn_raw_buffer_load_b128(rs, lane * 16 + s * 1024, 0, 17);
        const u32 tm2 = __hip_atomic_load(tmo, __ATOMIC_RELAXED, __HIP_MEMORY_SCOPE_AGENT);
        const int rrow0 = 1024 * (wv >> 2) + 32 * ur, rcol0 = 512 * (wv & 3) + 64 * v;
        gu8* odst = out + ((size_t)(rrow0 + (lane >> 3)) * N_ + rcol0) * 2 + (lane & 7) * 16;
        if (!ok2 || tm2 == call) {
#pragma unroll
            for (int s = 0; s < 4; ++s) st16_sys(odst + s * PASS, (u32x4){0x7FC07FC0u, 0x7FC07FC0u, 0x7FC07FC0u, 0x7FC07FC0u});
        } else if (rraw) {
#pragma unroll
            for (int s = 0; s < 4; ++s) st16_sys(odst + s * PASS, in[s]);
        } else {
            lu8* sc = lds + L_RED + wv * RED_WAVE;
            const u32 mb0 = mbw & 255u, mb1 = (mbw >> 8) & 255u, mb2 = (mbw >> 16) & 255u, mb3 = mbw >> 24;
            u32x2 sel[4];
            sel[0] = *(LAS u32x2*)(lds + L_LUT + mb0 * 8); sel[1] = *(LAS u32x2*)(lds + L_LUT + mb1 * 8);
            sel[2] = *(LAS u32x2*)(lds + L_LUT + mb2 * 8); sel[3] = *(LAS u32x2*)(lds + L_LUT + mb3 * 8);
            const u32 c01 = (u32)__popc(mb0) | ((u32)__popc(mb1) << 16), c23 = (u32)__popc(mb2) | ((u32)__popc(mb3) << 16);
            const u32 s01 = wave_scan_add(c01), s23 = wave_scan_add(c23);
#pragma unroll
            for (int s = 0; s < 4; ++s) *(LAS u32x4*)(sc + s * 1024 + lane * 16) = in[s];
            asm volatile("" ::: "memory");
            const u32 coff = img_coff(n);
            const u32 t01 = rdl(s01, 63), t23 = rdl(s23, 63);
            const u32 e01 = s01 - c01, e23 = s23 - c23;
            u32 off[4];
            off[0] = e01 & 0xFFFFu; off[1] = (t01 & 0xFFFFu) + (e01 >> 16);
            const u32 b2 = (t01 & 0xFFFFu) + (t01 >> 16);
            off[2] = b2 + (e23 & 0xFFFFu); off[3] = b2 + (t23 & 0xFFFFu) + (e23 >> 16);
            const u32 b4 = rb * 0x01010101u;
            u32 a0[4], a1[4], a2[4], k0[4], k1[4];
#pragma unroll
            for (int i = 0; i < 4; ++i) {
                lu32* lp = (lu32*)(sc + 256 + (off[i] & ~3u));
                a0[i] = lp[0]; a1[i] = lp[1]; a2[i] = lp[2];
                lu32* cp = (lu32*)(sc + coff + ((3u * off[i]) >> 5) * 4);
                k0[i] = cp[0]; k1[i] = cp[1];
            }
#pragma unroll
            for (int i = 0; i < 4; ++i) {
                const u32 lo0 = __builtin_amdgcn_alignbyte(a1[i], a0[i], off[i] & 3u), lo1 = __builtin_amdgcn_alignbyte(a2[i], a1[i], off[i] & 3u);
                const u32 c = __builtin_amdgcn_alignbit(k1[i], k0[i], (3u * off[i]) & 31u);
                u32 x0 = c & 0xFFFu, x1 = (c >> 12) & 0xFFFu;
                x0 = (x0 | (x0 << 10)) & 0x003F003Fu; x1 = (x1 | (x1 << 10)) & 0x003F003Fu;
                x0 = (x0 | (x0 << 5)) & 0x07070707u; x1 = (x1 | (x1 << 5)) & 0x07070707u;
                const u32 h0 = x0 + b4, h1 = x1 + b4;
                const u32 l0 = perm(lo1, lo0, sel[i].x), l1 = perm(lo1, lo0, sel[i].y), g0 = perm(h1, h0, sel[i].x), g1 = perm(h1, h0, sel[i].y);
                st16_sys(odst + i * PASS, (u32x4){perm(g0, l0, 0x05010400u), perm(g0, l0, 0x07030602u), perm(g1, l1, 0x05010400u), perm(g1, l1, 0x07030602u)});
            }
            if (fl & F_EXC) {
                drain();
                if (lane < 8) {
                    const u32 e = *(LAS u32*)(sc + img_eoff(n) + lane * 4);
                    if (e != 0u) { const u32 ix = (e >> 16) & 2047u; *(gu16*)(out + ((size_t)(rrow0 + (ix >> 6)) * N_ + rcol0 + (ix & 63u)) * 2) = (unsigned short)e; }
                }
            }
        }
    }
}

extern "C" void kernel_launch(int rank, int nranks, ncclComm_t comm, hipStream_t stream,
                              void* const* d_in, const int* in_sizes, int n_in,
                              void* d_out, int out_size, const int* in_dtypes, int out_dtype,
                              void* const* d_out_all, void* const* d_ws_all, size_t ws_bytes) {
    (void)comm; (void)d_out_all; (void)in_dtypes;
    if (nranks != NR || (unsigned)rank >= (unsigned)NR || ws_bytes < WS_END || n_in != 2 || in_sizes[0] != M_ * KS || in_sizes[1] != KS * N_ ||
        out_size != M_ * N_ || out_dtype != 1) {
        fprintf(stderr, "kernel_launch: unexpected problem shape\n");
        return;
    }
    static bool init = false;
    if (!init) {
        if (hipFuncSetAttribute((const void*)fused, hipFuncAttributeMaxDynamicSharedMemorySize, LDS_BYTES) != hipSuccess) fprintf(stderr, "kernel_launch: LDS attribute\n");
        init = true;
    }
    Params P;
    P.x = (const unsigned short*)d_in[0]; P.w = (const unsigned short*)d_in[1];
    for (int i = 0; i < NR; ++i) P.ws[i] = (unsigned char*)d_ws_all[i];
    P.out = (unsigned char*)d_out;
    P.rank = rank; P.pad = 0;
    hipLaunchKernelGGL(fused, dim3(NWG), dim3(NTHR), LDS_BYTES, stream, P);
}
```

```cpp
#include <hip/hip_runtime.h>
#include <rccl/rccl.h>
#include <cstdio>

#define GAS __attribute__((address_space(1)))
#define LAS __attribute__((address_space(3)))
typedef unsigned u32;
typedef unsigned long long u64;
typedef u32 u32x4 __attribute__((ext_vector_type(4)));
typedef u32 u32x2 __attribute__((ext_vector_type(2)));
typedef float f32x2 __attribute__((ext_vector_type(2)));
typedef float f32x4 __attribute__((ext_vector_type(4)));
typedef float f32x16 __attribute__((ext_vector_type(16)));
typedef __bf16 bf16x8 __attribute__((ext_vector_type(8)));
typedef __bf16 bf16x2 __attribute__((ext_vector_type(2)));
typedef unsigned short u16x2 __attribute__((ext_vector_type(2)));
typedef GAS u32 gu32; typedef GAS u64 gu64; typedef GAS unsigned char gu8; typedef GAS unsigned short gu16;
typedef LAS unsigned char lu8; typedef LAS u32 lu32;

constexpr int NR = 8, M_ = 2048, N_ = 2048, KS = 256;
constexpr int NWG = 256, NTHR = 512;

constexpr size_t WS_TMO = 0;
constexpr size_t WS_CNT = 4096;
constexpr size_t WS_XFLAG = WS_CNT + 256 * 128;
constexpr size_t WS_WFLAG = WS_XFLAG + 8 * 32 * 128;
constexpr size_t WS_OFLAG = WS_WFLAG + 8 * 8 * 128;
constexpr size_t WS_XEXC = WS_OFLAG + 8 * 256 * 128;
constexpr size_t WS_WEXC = WS_XEXC + 8 * 32 * 128;
constexpr size_t WS_FEND = WS_WEXC + 8 * 8 * 256;
constexpr size_t SLOT = 4224, SKEW = 20608;
constexpr size_t XSRC = 32 * 4 * SLOT + SKEW, WSRC = 8 * 8 * SLOT + SKEW, OSRC = 256 * SLOT + SKEW;
constexpr size_t WS_XDATA = 1u << 20;
constexpr size_t WS_WDATA = WS_XDATA + 8 * XSRC;
constexpr size_t WS_ODATA = WS_WDATA + 8 * WSRC;
constexpr size_t WS_END = WS_ODATA + 8 * OSRC;
static_assert(WS_FEND <= WS_XDATA && SLOT % 128 == 0 && SKEW % 128 == 0, "workspace layout");
constexpr u64 DEADLINE = 30000000ull;

constexpr int T_W0 = 105, D_W = 350;
constexpr int T_X0 = 455, D_X = 700;
constexpr int T_O0 = 1170, D_O = 795;
constexpr int T_POLL = 150;
constexpr int T_SELF = 820;
constexpr int T_WD = 380;

struct Params {
    const unsigned short* x; const unsigned short* w;
    unsigned char* my; unsigned char* out;
    int rank, pad;
    unsigned char* ws[NR];
};

#define SB() __builtin_amdgcn_sched_barrier(0)
#define RARE(c) __builtin_expect(!!(c), 0)

__device__ __forceinline__ u64 rt() { return __builtin_amdgcn_s_memrealtime(); }
__device__ __forceinline__ void st16_sys(gu8* p, u32x4 v) { asm volatile("global_store_dwordx4 %0, %1, off sc0 sc1\n\ts_nop 1" :: "v"(p), "v"(v) : "memory"); }
__device__ __forceinline__ void st4_sys(gu8* p, u32 v) { asm volatile("global_store_dword %0, %1, off sc0 sc1\n\ts_nop 1" :: "v"(p), "v"(v) : "memory"); }
__device__ __forceinline__ void drain() { asm volatile("s_waitcnt vmcnt(0)" ::: "memory"); }
__device__ __forceinline__ void flag_store(gu8* p, u64 v) { __hip_atomic_store((gu64*)p, v, __ATOMIC_RELAXED, __HIP_MEMORY_SCOPE_SYSTEM); }
__device__ __forceinline__ u64 flag_load(gu8* p) { return __hip_atomic_load((gu64*)p, __ATOMIC_RELAXED, __HIP_MEMORY_SCOPE_SYSTEM); }
__device__ __forceinline__ u32 rfl(u32 v) { return __builtin_amdgcn_readfirstlane(v); }
__device__ __forceinline__ u32 rdl(u32 v, int l) { return (u32)__builtin_amdgcn_readlane((int)v, l); }
__device__ __forceinline__ u32 perm(u32 hi, u32 lo, u32 sel) { return __builtin_amdgcn_perm(hi, lo, sel); }
__device__ __forceinline__ u32 rotr(u32 x, u32 s) { return __builtin_amdgcn_alignbit(x, x, s); }
__device__ __forceinline__ u32 mbcnt64(u64 m, u32 acc) { return __builtin_amdgcn_mbcnt_hi((u32)(m >> 32), __builtin_amdgcn_mbcnt_lo((u32)m, acc)); }
template <int CTRL, int RM> __device__ __forceinline__ u32 dpp0(u32 v) { return (u32)__builtin_amdgcn_update_dpp(0, (int)v, CTRL, RM, 0xf, false); }
__device__ __forceinline__ u32 wave_max(u32 v) {
    v = max(v, dpp0<0x111, 0xf>(v)); v = max(v, dpp0<0x112, 0xf>(v)); v = max(v, dpp0<0x114, 0xf>(v)); v = max(v, dpp0<0x118, 0xf>(v));
    v = max(v, dpp0<0x142, 0xa>(v)); v = max(v, dpp0<0x143, 0xc>(v));
    return rdl(v, 63);
}
__device__ __forceinline__ u32 wave_scan_add(u32 v) {
    v += dpp0<0x111, 0xf>(v); v += dpp0<0x112, 0xf>(v); v += dpp0<0x114, 0xf>(v); v += dpp0<0x118, 0xf>(v);
    v += dpp0<0x142, 0xa>(v); v += dpp0<0x143, 0xc>(v);
    return v;
}

__device__ __forceinline__ bool wait_flags(gu8* p, u32 call, gu32* tmo, u64& val) {
    const u64 t0 = rt();
    for (u32 it = 1;; ++it) {
        const u64 v = flag_load(p);
        if (__all((u32)(v >> 32) >= call)) { val = v; asm volatile("" ::: "memory"); return true; }
        if (RARE((it & 31u) == 0u)) {
            if (__hip_atomic_load(tmo, __ATOMIC_RELAXED, __HIP_MEMORY_SCOPE_AGENT) == call || rt() - t0 > DEADLINE) {
                __hip_atomic_store(tmo, call, __ATOMIC_RELAXED, __HIP_MEMORY_SCOPE_AGENT);
                val = 0; asm volatile("" ::: "memory"); return false;
            }
        }
        __builtin_amdgcn_s_sleep(2);
    }
}
__device__ __forceinline__ void wait_until(u64 t) { while (rt() < t) __builtin_amdgcn_s_sleep(1); }

constexpr int RED_STRIDE = 272;
constexpr int RED_WAVE = 32 * RED_STRIDE;
constexpr int L_RED = 0;
constexpr int L_XP = 8 * RED_WAVE;
constexpr int L_WP = L_XP + 3072;
constexpr int L_XR = L_WP + 3072;
constexpr int L_WR = L_XR + 4096;
constexpr int L_XH = L_WR + 4096;
constexpr int L_WH = L_XH + 64;
constexpr int L_OUT = L_WH + 64;
constexpr int L_IMG = L_OUT + 4096;
constexpr int L_LUT = L_IMG + 3200;
constexpr int L_LUT4 = L_LUT + 2048;
constexpr int L_MISC = L_LUT4 + 64;
constexpr int LDS_BYTES = L_MISC + 128;
static_assert(LDS_BYTES > 81920 && LDS_BYTES <= 163840 && L_XP % 16 == 0 && L_IMG % 16 == 0 && L_LUT % 16 == 0 && L_MISC % 16 == 0, "lds");

constexpr u32 F_RAW = 1u << 31, F_EXC = 1u << 30;
constexpr u32 EXC_NONE = 0xFFFFFFFFu;
__device__ __forceinline__ u32 img_coff(u32 n) { return 256u + ((n + 3u) & ~3u); }
__device__ __forceinline__ u32 img_eoff(u32 n) { return img_coff(n) + ((((3u * n + 7u) >> 3) + 3u) & ~3u); }
__device__ __forceinline__ u32 img_chunks(u32 n, bool exc) { return (img_eoff(n) + (exc ? 32u : 0u) + 15u) >> 4; }

__device__ __forceinline__ void pack_frag(u32x4 e, int f, int lane, lu8* P, lu8* R, lu8* H) {
    *(LAS u32x4*)(R + f * 1024 + lane * 16) = e;
    const u16x2 m01 = __builtin_elementwise_max(__builtin_bit_cast(u16x2, e.x & 0x7FFF7FFFu), __builtin_bit_cast(u16x2, e.y & 0x7FFF7FFFu));
    const u16x2 m23 = __builtin_elementwise_max(__builtin_bit_cast(u16x2, e.z & 0x7FFF7FFFu), __builtin_bit_cast(u16x2, e.w & 0x7FFF7FFFu));
    const u16x2 mm = __builtin_elementwise_max(m01, m23);
    const u32 mx = wave_max(max((u32)mm.x, (u32)mm.y));
    const u32 h7 = mx >> 8, base = h7 > 7u ? h7 - 7u : 0u;
    const u32 lo0 = perm(e.y, e.x, 0x06040200u), lo1 = perm(e.w, e.z, 0x06040200u);
    const u32 hi0 = perm(e.y, e.x, 0x07050301u), hi1 = perm(e.w, e.z, 0x07050301u);
    const u32 K = (128u - base) * 0x01010101u;
    const u32 d0 = (hi0 & 0x7F7F7F7Fu) + K, d1 = (hi1 & 0x7F7F7F7Fu) + K;
    const u32 cA = (hi0 & 0x80808080u) | (d0 & 0x07070707u), cB = (hi1 & 0x80808080u) | (d1 & 0x07070707u);
    *(LAS u32x2*)(P + (f >> 1) * 1024 + lane * 16 + (f & 1) * 8) = (u32x2){lo0, lo1};
    *(LAS u32*)(P + 2048 + lane * 16 + f * 4) = cA | rotr(cB, 28);
    const u32 x0 = ~d0 & 0x80808080u, x1 = ~d1 & 0x80808080u;
    u32 hdr = base;
    if (lane < 2) *(LAS u32*)(H + 16 + f * 8 + lane * 4) = EXC_NONE;
    asm volatile("" ::: "memory");
    if (RARE(__any((x0 | x1) != 0u))) {
        u32 cnt = 0;
#pragma unroll
        for (int j = 0; j < 8; ++j) {
            const bool ex = (((j < 4 ? x0 : x1) >> (8 * (j & 3) + 7)) & 1u) != 0u;
            const u64 b = __ballot(ex);
            const u32 rk = mbcnt64(b, cnt);
            const u32 val = (e[j >> 1] >> (16 * (j & 1))) & 0xFFFFu;
            if (ex && rk < 2u) *(LAS u32*)(H + 16 + f * 8 + rk * 4) = ((u32)(f * 512 + lane * 8 + j) << 16) | val;
            cnt += (u32)__popcll(b);
        }
        hdr |= cnt > 2u ? 0x300u : 0x100u;
    }
    if (lane == 0) *(LAS u32*)(H + f * 4) = hdr;
}
__device__ __forceinline__ void load_block(u32x4 (&d)[4], u32& fl, u32& excw, int lane, lu8* P, lu8* R, lu8* H) {
    const u32x4 h = *(LAS u32x4*)H;
    const u32 any = rfl(h.x | h.y | h.z | h.w);
    const bool raw = (any & 0x200u) != 0u;
    fl = (rfl(h.x) & 127u) | ((rfl(h.y) & 127u) << 7) | ((rfl(h.z) & 127u) << 14) | ((rfl(h.w) & 127u) << 21) | (raw ? F_RAW : ((any & 0x100u) ? F_EXC : 0u));
    lu8* S = raw ? R : P;
    d[0] = *(LAS u32x4*)(S + lane * 16); d[1] = *(LAS u32x4*)(S + 1024 + lane * 16); d[2] = *(LAS u32x4*)(S + 2048 + lane * 16);
    d[3] = *(LAS u32x4*)(R + 3072 + lane * 16);
    excw = *(LAS u32*)(H + 16 + (lane & 7) * 4);
}
__device__ __forceinline__ void send_block(const u32x4 (&d)[4], u32 fl, u32 excw, int lane, gu8* dst, gu8* dexc) {
    st16_sys(dst + lane * 16, d[0]); st16_sys(dst + 1024 + lane * 16, d[1]); st16_sys(dst + 2048 + lane * 16, d[2]);
    if (RARE(fl & F_RAW)) st16_sys(dst + 3072 + lane * 16, d[3]);
    if (RARE(fl & F_EXC)) { if (lane < 8) st4_sys(dexc + lane * 4, excw); }
}
__device__ __forceinline__ u32x4 unpack_frag(u32 lo0, u32 lo1, u32 nw, u32 b4) {
    const u32 hA = (nw & 0x87878787u) + b4, hB = (rotr(nw, 4) & 0x87878787u) + b4;
    return (u32x4){perm(hA, lo0, 0x05010400u), perm(hA, lo0, 0x07030602u), perm(hB, lo1, 0x05010400u), perm(hB, lo1, 0x07030602u)};
}
__device__ __forceinline__ void unpack_block(u32x4 (&fr)[4], u32x4 L0, u32x4 L1, u32x4 Nw, u32 fl, u32 excw, int blk, int lane, lu8* scratch,
                                             __amdgpu_buffer_rsrc_t rs, int soff) {
    if (RARE(fl & F_RAW)) {
        fr[0] = L0; fr[1] = L1; fr[2] = Nw;
        fr[3] = __builtin_amdgcn_raw_buffer_load_b128(rs, lane * 16, soff + 3072, 17);
        asm volatile("" : "+v"(fr[3]));
        return;
    }
    fr[0] = unpack_frag(L0.x, L0.y, Nw.x, (fl & 127u) * 0x01010101u);
    fr[1] = unpack_frag(L0.z, L0.w, Nw.y, ((fl >> 7) & 127u) * 0x01010101u);
    fr[2] = unpack_frag(L1.x, L1.y, Nw.z, ((fl >> 14) & 127u) * 0x01010101u);
    fr[3] = unpack_frag(L1.z, L1.w, Nw.w, ((fl >> 21) & 127u) * 0x01010101u);
    if (RARE(fl & F_EXC)) {
#pragma unroll
        for (int f = 0; f < 4; ++f) *(LAS u32x4*)(scratch + f * 1024 + lane * 16) = fr[f];
        asm volatile("" ::: "memory");
        if ((lane >> 3) == blk && excw != EXC_NONE) *(LAS unsigned short*)(scratch + ((excw >> 16) & 2047u) * 2) = (unsigned short)excw;
        asm volatile("" ::: "memory");
#pragma unroll
        for (int f = 0; f < 4; ++f) fr[f] = *(LAS u32x4*)(scratch + f * 1024 + lane * 16);
        asm volatile("" : "+v"(fr[0]), "+v"(fr[1]), "+v"(fr[2]), "+v"(fr[3]));
    }
}


__global__ void __launch_bounds__(NTHR, 2) fused(Params P) {
    extern __shared__ __attribute__((aligned(16))) unsigned char smem[];
    lu8* lds = (lu8*)smem;
    const int tid = threadIdx.x, lane = tid & 63, wv = __builtin_amdgcn_readfirstlane(tid >> 6);
    const int g = blockIdx.x, xcd = g & 7, idx = g >> 3, u = 2 * (idx >> 1) + (xcd >> 2), v = 2 * (xcd & 3) + (idx & 1);
    const int rank = P.rank;
    gu8* myws = (gu8*)P.my;
    { unsigned char* last = P.ws[NR - 1]; asm volatile("" :: "s"(last), "s"(P.x), "s"(P.w), "s"(P.my), "s"(rank)); }
    const u64 tstart = rt();
    gu32* tmo = (gu32*)(myws + WS_TMO);
    gu32* cntp = (gu32*)(myws + WS_CNT + g * 128);

    const int xi = v >> 2, xB = v & 3;
    const int wV = g >> 3, wb = g & 7;
    const int r = lane & 31, h = lane >> 5;
    u32 warm = 0;
    if (wv < 4) {
        if (idx == 7 && wv == 0) {
            const int k = lane >> 3, q = lane & 7;
            u64 b = (u64)P.ws[0];
#pragma unroll
            for (int i = 1; i < 8; ++i) { u64 t = (u64)P.ws[i]; asm volatile("" : "+s"(t)); b = q == i ? t : b; }
            const size_t off = k == 0 ? WS_WFLAG + rank * 1024 : k == 1 ? WS_WDATA + rank * WSRC : k == 2 ? WS_WDATA + rank * WSRC + WSRC - 128 : k == 3 ? WS_XDATA + rank * XSRC :
                               k == 4 ? WS_XDATA + rank * XSRC + XSRC - 128 : k == 5 ? WS_ODATA + rank * OSRC : k == 6 ? WS_ODATA + rank * OSRC + OSRC / 2 : WS_ODATA + rank * OSRC + OSRC - 128;
            asm volatile("global_load_dword %0, %1, off sc0 sc1" : "=&v"(warm) : "v"((gu8*)b + off) : "memory");
        } else if (idx == 7 && wv == 1) {
            gu8* a = lane < 32 ? myws + min((size_t)lane << 20, WS_END - 128) : (gu8*)P.out + ((size_t)(lane & 7) << 20);
            asm volatile("global_load_dword %0, %1, off sc0 sc1" : "=&v"(warm) : "v"(a) : "memory");
        }
        const u32x4 frag = *(const GAS u32x4*)((const gu16*)P.x + (size_t)(1024 * xi + 32 * u + r) * KS + 64 * xB + 16 * wv + 8 * h);
        pack_frag(frag, wv, lane, lds + L_XP, lds + L_XR, lds + L_XH);
    } else {
        const u32 call0 = __hip_atomic_load(cntp, __ATOMIC_RELAXED, __HIP_MEMORY_SCOPE_AGENT);
        const int f = wv - 4;
        const gu16* src = (const gu16*)P.w + (size_t)(32 * wb + 16 * (f >> 1) + 8 * h) * N_ + 64 * wV + 32 * (f & 1) + r;
        u32 e[8];
#pragma unroll
        for (int j = 0; j < 8; ++j) e[j] = src[(size_t)j * N_];
        const u32x4 frag = (u32x4){e[0] | (e[1] << 16), e[2] | (e[3] << 16), e[4] | (e[5] << 16), e[6] | (e[7] << 16)};
        pack_frag(frag, f, lane, lds + L_WP, lds + L_WR, lds + L_WH);
        if (tid == 256) *(LAS u32*)(lds + L_MISC) = call0 + 1u;
    }
    if (tid >= 64 && tid < 64 + 3) *(LAS u32*)(lds + L_MISC + 80 + (tid - 64) * 4) = 0u;
    __syncthreads();
    const u32 call = rfl(*(LAS u32*)(lds + L_MISC));
    if (tid == 256) __hip_atomic_store(cntp, call, __ATOMIC_RELAXED, __HIP_MEMORY_SCOPE_AGENT);
    const u64 fv = (u64)call << 32;
    const int wa = (wv - rank) & 7;
    if (wa == 1 || wa == 2) {
        const int q = (wV >> 3) + 4 * (wa - 1), vv = wV & 7;
        gu8* dws = (gu8*)P.ws[q];
        u32x4 d[4]; u32 fl, excw;
        load_block(d, fl, excw, lane, lds + L_WP, lds + L_WR, lds + L_WH);
        if (q != rank) wait_until(tstart + (u64)(T_W0 + ((((g & 63) * 4 + (g >> 6)) * D_W) >> 8)));
        send_block(d, fl, excw, lane, dws + WS_WDATA + rank * WSRC + (size_t)(vv * 8 + wb) * SLOT, dws + WS_WEXC + (rank * 8 + vv) * 256 + wb * 32);
        drain();
        if (lane == 0) flag_store(dws + WS_WFLAG + (rank * 8 + vv) * 128 + wb * 8, fv | fl);
    } else if (wa == 3) {
        if (xi == (rank >> 2)) {
            u32x4 d[4]; u32 fl, excw;
            load_block(d, fl, excw, lane, lds + L_XP, lds + L_XR, lds + L_XH);
            send_block(d, fl, excw, lane, myws + WS_XDATA + rank * XSRC + (size_t)(u * 4 + xB) * SLOT, myws + WS_XEXC + (rank * 32 + u) * 128 + xB * 32);
            drain();
            if (lane == 0) flag_store(myws + WS_XFLAG + (rank * 32 + u) * 128 + xB * 8, fv | fl);
        }
    } else if (wa >= 4) {
        const u32 m = (u32)(wa - 4) * 64u + (u32)lane;
        u32 s0 = 0, s1 = 0, c = 0;
#pragma unroll
        for (int j = 0; j < 8; ++j) { const u32 b = (m >> j) & 1u; const u32 sv = b ? c : 0x0Cu; c += b; if (j < 4) s0 |= sv << (8 * j); else s1 |= sv << (8 * (j - 4)); }
        *(LAS u32x2*)(lds + L_LUT + m * 8) = (u32x2){s0, s1};
        if (m < 16u) {
            u32 sel = 0x0C0C0C0Cu, k = 0;
#pragma unroll
            for (int j = 0; j < 4; ++j) if ((m >> j) & 1u) { sel = (sel & ~(0xFFu << (8 * k))) | ((u32)j << (8 * k)); k++; }
            *(LAS u32*)(lds + L_LUT4 + m * 4) = sel;
        }
    }

    bool ok = true; u64 fvalW, fvalX;
    const int p = wv;
    const bool self = wa == 0;
    lu8* scratch = lds + L_RED + p * RED_WAVE;
    const __amdgpu_buffer_rsrc_t rsW = __builtin_amdgcn_make_buffer_rsrc((void*)(myws + WS_WDATA + p * WSRC + (size_t)v * 8 * SLOT), 0, 8 * SLOT, 0x00020000);
    const __amdgpu_buffer_rsrc_t rsX = __builtin_amdgcn_make_buffer_rsrc((void*)(myws + WS_XDATA + p * XSRC + (size_t)u * 4 * SLOT), 0, 4 * SLOT, 0x00020000);
    const int tau = T_X0 + (((u * 8 + xB * 2 + xi) * D_X) >> 8);
    const bool early = tau <= T_SELF;
    u32 xfl = 0;
    auto xstores = [&]() {
        u32x4 xd[4]; u32 xexc;
        load_block(xd, xfl, xexc, lane, lds + L_XP, lds + L_XR, lds + L_XH);
        wait_until(tstart + (u64)tau);
#pragma unroll
        for (int k = 0; k < 4; ++k) {
            const int q = 4 * xi + ((k + g) & 3);
            gu8* dws = (gu8*)P.ws[q];
            if (q != rank) send_block(xd, xfl, xexc, lane, dws + WS_XDATA + rank * XSRC + (size_t)(u * 4 + xB) * SLOT, dws + WS_XEXC + (rank * 32 + u) * 128 + xB * 32);
        }
        __hip_atomic_store((lu32*)(lds + L_MISC + 88), 1u, __ATOMIC_RELAXED, __HIP_MEMORY_SCOPE_WORKGROUP);
    };
    auto xflags = [&]() {
        unsigned char* const q0 = P.ws[4 * xi], * const q1 = P.ws[4 * xi + 1], * const q2 = P.ws[4 * xi + 2], * const q3 = P.ws[4 * xi + 3];
        gu8* fq = (gu8*)(lane == 0 ? q0 : lane == 1 ? q1 : lane == 2 ? q2 : q3);
        if (lane < 4 && 4 * xi + lane != rank) flag_store(fq + WS_XFLAG + (rank * 32 + u) * 128 + xB * 8, fv | xfl);
    };
    if (self) {
        u64 fv2;
        gu8* fp = (lane & 8) ? myws + WS_XFLAG + (p * 32 + u) * 128 + (lane & 3) * 8 : myws + WS_WFLAG + (p * 8 + v) * 128 + (lane & 7) * 8;
        ok &= wait_flags(fp, call, tmo, fv2);
        fvalW = fv2; fvalX = (u64)(u32)__builtin_amdgcn_ds_bpermute((8 + (lane & 3)) * 4, (int)(u32)fv2);
        if (early) xstores();
    } else {
        const int tA = T_W0 + (((32 * v + 31) * D_W) >> 8) + T_POLL;
        wait_until(tstart + (u64)tA);
        ok &= wait_flags(myws + WS_WFLAG + (p * 8 + v) * 128 + (lane & 7) * 8, call, tmo, fvalW);
        if (tA + T_WD > tau) {
            const u64 t0 = rt();
            while (__hip_atomic_load((lu32*)(lds + L_MISC + 88), __ATOMIC_RELAXED, __HIP_MEMORY_SCOPE_WORKGROUP) == 0u && rt() - t0 < DEADLINE) __builtin_amdgcn_s_sleep(1);
        }
    }
    const int rot = ((u >> 1) + 2 * (v & 1)) & 3;
    u32x4 wf[8][4];
    u32x4 rawX[4][3]; u32 excX, tm;
    auto xloads = [&]() {
        excX = __hip_atomic_load((gu32*)(myws + WS_XEXC + (p * 32 + u) * 128 + (lane & 31) * 4), __ATOMIC_RELAXED, __HIP_MEMORY_SCOPE_SYSTEM);
        tm = __hip_atomic_load(tmo, __ATOMIC_RELAXED, __HIP_MEMORY_SCOPE_AGENT);
#pragma unroll
        for (int b = 0; b < 4; ++b)
#pragma unroll
            for (int k = 0; k < 3; ++k) rawX[b][k] = __builtin_amdgcn_raw_buffer_load_b128(rsX, lane * 16, ((b + rot) & 3) * (int)SLOT + k * 1024, 17);
        SB();
    };
    gu8* const xfp = myws + WS_XFLAG + (p * 32 + u) * 128 + (lane & 3) * 8;
    {
        const u32 excW = __hip_atomic_load((gu32*)(myws + WS_WEXC + (p * 8 + v) * 256 + lane * 4), __ATOMIC_RELAXED, __HIP_MEMORY_SCOPE_SYSTEM);
        u32x4 rawW[8][3];
        auto wl = [&](int b) {
            const int so = ((b + 2 * rot) & 7) * (int)SLOT;
#pragma unroll
            for (int k = 0; k < 3; ++k) rawW[b][k] = __builtin_amdgcn_raw_buffer_load_b128(rsW, lane * 16, so + k * 1024, 17);
        };
        auto wu = [&](int b) {
            const int bm = (b + 2 * rot) & 7;
            unpack_block(wf[b], rawW[b][0], rawW[b][1], rawW[b][2], rdl((u32)fvalW, bm), excW, bm, lane, scratch, rsW, bm * (int)SLOT);
        };
        bool xl = false; u64 fx = 0;
        auto xlook = [&]() { if (!xl) fx = flag_load(xfp); };
        auto xtry = [&]() { if (!xl && __all((u32)(fx >> 32) >= call)) { asm volatile("" ::: "memory"); fvalX = fx; xloads(); xl = true; } };
        wl(0); wl(1); wl(2); SB();
        if (self && early) { asm volatile("s_waitcnt vmcnt(9)" ::: "memory"); xflags(); }
        wu(0); SB(); wl(3); SB();
        wu(1); SB(); wl(4); SB();
        wu(2); SB(); wl(5); SB();
        wu(3); SB(); wl(6); SB();
        if (self) fx = fv | (u32)fvalX; else xlook();
        SB(); wu(4); SB(); wl(7); SB();
        xtry(); SB();
        wu(5); SB(); xlook(); SB();
        wu(6); SB(); xtry(); SB();
        wu(7);
        if (!xl) {
            wait_until(tstart + (u64)(T_X0 + (((u * 8 + 7) * D_X) >> 8) + T_POLL));
            ok &= wait_flags(xfp, call, tmo, fvalX);
            xloads();
        }
    }
    f32x16 acc[2];
#pragma unroll
    for (int i = 0; i < 16; ++i) { acc[0][i] = 0.f; acc[1][i] = 0.f; }
#pragma unroll
    for (int B = 0; B < 4; ++B) {
        u32x4 xf[4];
        const int bm = (B + rot) & 3;
        unpack_block(xf, rawX[B][0], rawX[B][1], rawX[B][2], rdl((u32)fvalX, bm), lane < 32 ? excX : EXC_NONE, bm, lane, scratch, rsX, bm * (int)SLOT);
#pragma unroll
        for (int f = 0; f < 4; ++f) {
            const int T = 4 * B + f;
            acc[0] = __builtin_amdgcn_mfma_f32_32x32x16_bf16(__builtin_bit_cast(bf16x8, wf[T >> 1][2 * (T & 1)]), __builtin_bit_cast(bf16x8, xf[f]), acc[0], 0, 0, 0);
            acc[1] = __builtin_amdgcn_mfma_f32_32x32x16_bf16(__builtin_bit_cast(bf16x8, wf[T >> 1][2 * (T & 1) + 1]), __builtin_bit_cast(bf16x8, xf[f]), acc[1], 0, 0, 0);
        }
    }
    if (RARE(!ok || tm == call) && lane == 0) *(LAS u32*)(lds + L_MISC + 84) = 1u;
#pragma unroll
    for (int cb = 0; cb < 2; ++cb)
#pragma unroll
        for (int g4 = 0; g4 < 4; ++g4)
            *(LAS f32x4*)(lds + L_RED + p * RED_WAVE + r * RED_STRIDE + (32 * cb + 8 * g4 + 4 * h) * 4) =
                (f32x4){acc[cb][4 * g4], acc[cb][4 * g4 + 1], acc[cb][4 * g4 + 2], acc[cb][4 * g4 + 3]};
    if (self && !early) { xstores(); drain(); xflags(); }
    asm volatile("" :: "v"(warm));
    if (tid < 184) *(LAS u32x4*)(lds + L_IMG + 256 + tid * 16) = (u32x4){0, 0, 0, 0};
    __syncthreads();

    u32 o01, o23, pre, m4;
    {
        const int m = tid >> 4, n0 = 4 * (tid & 15);
        const bool bad = rfl(*(LAS u32*)(lds + L_MISC + 84)) != 0u;
        f32x4 s = *(LAS f32x4*)(lds + L_RED + m * RED_STRIDE + n0 * 4);
#pragma unroll
        for (int pp = 1; pp < 8; ++pp) s += *(LAS f32x4*)(lds + L_RED + pp * RED_WAVE + m * RED_STRIDE + n0 * 4);
        const f32x4 y = __builtin_elementwise_maximum(s, (f32x4){0.f, 0.f, 0.f, 0.f});
        o01 = __builtin_bit_cast(u32, __builtin_convertvector((f32x2){y.x, y.y}, bf16x2)) & 0x7FFF7FFFu;
        o23 = __builtin_bit_cast(u32, __builtin_convertvector((f32x2){y.z, y.w}, bf16x2)) & 0x7FFF7FFFu;
        if (RARE(bad)) { o01 = 0x7FC07FC0u; o23 = 0x7FC07FC0u; }
        *(LAS u32x2*)(lds + L_OUT + tid * 8) = (u32x2){o01, o23};
        const u16x2 mm = __builtin_elementwise_max(__builtin_bit_cast(u16x2, o01), __builtin_bit_cast(u16x2, o23));
        const u32 wmx = wave_max(max((u32)mm.x, (u32)mm.y));
        const bool z0 = (o01 & 0xFFFFu) != 0u, z1 = o01 > 0xFFFFu, z2 = (o23 & 0xFFFFu) != 0u, z3 = o23 > 0xFFFFu;
        const u64 b0 = __ballot(z0), b1 = __ballot(z1), b2 = __ballot(z2), b3 = __ballot(z3);
        pre = mbcnt64(b3, mbcnt64(b2, mbcnt64(b1, mbcnt64(b0, 0u))));
        m4 = (z0 ? 1u : 0u) | (z1 ? 2u : 0u) | (z2 ? 4u : 0u) | (z3 ? 8u : 0u);
        if (lane == 0) *(LAS u32*)(lds + L_MISC + 16 + wv * 4) = ((wmx >> 8) << 16) | (u32)(__popcll(b0) + __popcll(b1) + __popcll(b2) + __popcll(b3));
    }
    __syncthreads();
    u32 ntot, obase;
    {
        const u32 w = *(LAS u32*)(lds + L_MISC + 16 + (lane & 7) * 4);
        const u32 sel = *(LAS u32*)(lds + L_LUT4 + m4 * 4);
        u32 cs = (w & 0xFFFFu) * ((lane & 7) < wv ? 0x10001u : 1u), mx = w >> 16;
        cs += dpp0<0x111, 0xf>(cs); mx = max(mx, dpp0<0x111, 0xf>(mx));
        cs += dpp0<0x112, 0xf>(cs); mx = max(mx, dpp0<0x112, 0xf>(mx));
        cs += dpp0<0x114, 0xf>(cs); mx = max(mx, dpp0<0x114, 0xf>(mx));
        const u32 cst = rdl(cs, 7), h7 = rdl(mx, 7);
        ntot = cst & 0xFFFFu; obase = h7 > 7u ? h7 - 7u : 0u;
        const u32 pos = (cst >> 16) + pre;
        const u32 L = perm(o23, o01, 0x06040200u), H = perm(o23, o01, 0x07050301u);
        const u32 d = H + (128u - obase) * 0x01010101u;
        const u32 Lc = perm(0u, L, sel), Cc = perm(0u, d & 0x07070707u, sel);
        const u32 c12 = __builtin_amdgcn_udot4(Cc, 0x00400801u, (Cc >> 15) & 0xE00u, false);
        if (Lc) {
            const u32 sh = (pos & 3u) * 8u;
            lu32* lp = (lu32*)(lds + L_IMG + 256 + (pos & ~3u));
            __hip_atomic_fetch_or(lp, Lc << sh, __ATOMIC_RELAXED, __HIP_MEMORY_SCOPE_WORKGROUP);
            const u32 hi = (u32)(((u64)Lc << sh) >> 32); if (hi) __hip_atomic_fetch_or(lp + 1, hi, __ATOMIC_RELAXED, __HIP_MEMORY_SCOPE_WORKGROUP);
        }
        if (c12) {
            const u32 bit = 3u * pos, sh = bit & 31u;
            lu32* cp = (lu32*)(lds + L_IMG + img_coff(ntot) + (bit >> 5) * 4);
            __hip_atomic_fetch_or(cp, c12 << sh, __ATOMIC_RELAXED, __HIP_MEMORY_SCOPE_WORKGROUP);
            const u32 hi = (u32)(((u64)c12 << sh) >> 32); if (hi) __hip_atomic_fetch_or(cp + 1, hi, __ATOMIC_RELAXED, __HIP_MEMORY_SCOPE_WORKGROUP);
        }
        if (RARE(~perm(0u, d, sel | ((sel & 0x08080808u) >> 3)) & 0x80808080u)) {
#pragma unroll
            for (int j = 0; j < 4; ++j) {
                const u32 oj = ((j < 2 ? o01 : o23) >> (16 * (j & 1))) & 0xFFFFu;
                if (oj != 0u && (oj >> 8) < obase) {
                    const u32 e = __hip_atomic_fetch_add((lu32*)(lds + L_MISC + 80), 1u, __ATOMIC_RELAXED, __HIP_MEMORY_SCOPE_WORKGROUP);
                    if (e < 8u) *(LAS u32*)(lds + L_IMG + img_eoff(ntot) + e * 4) = ((u32)(tid * 4 + j) << 16) | oj;
                }
            }
        }
        const u32 other = (u32)__builtin_amdgcn_update_dpp(0, (int)m4, 0xB1  , 0xf, 0xf, false);
        const u32 gi = (u32)tid >> 1;
        if (!(lane & 1)) *(LAS unsigned char*)(lds + L_IMG + 4 * (gi & 63u) + (gi >> 6)) = (unsigned char)(m4 | (other << 4));
    }
    __syncthreads();

    const int tile = u * 8 + v;
    constexpr size_t PASS = (size_t)8 * N_ * 2;
    const u32 nexc = rfl(*(LAS u32*)(lds + L_MISC + 80));
    const bool raw = nexc > 8u;
    gu8* const out = (gu8*)P.out;
    if (self) {
        u32x4 d[4];
#pragma unroll
        for (int s = 0; s < 4; ++s) d[s] = *(LAS u32x4*)(lds + L_OUT + s * 1024 + lane * 16);
        gu8* dst = out + ((size_t)(1024 * (rank >> 2) + 32 * u + (lane >> 3)) * N_ + 512 * (rank & 3) + 64 * v) * 2 + (lane & 7) * 16;
#pragma unroll
        for (int s = 0; s < 4; ++s) st16_sys(dst + s * PASS, d[s]);
    } else {
        {
            const u32 nch = raw ? 256u : ((img_chunks(ntot, nexc != 0u) + 3u) & ~3u);
            lu8* S = lds + (raw ? L_OUT : L_IMG);
            gu8* dst = (gu8*)P.ws[wv] + WS_ODATA + rank * OSRC + (size_t)tile * SLOT + lane * 16;
            u32x4 d[4];
#pragma unroll
            for (int s = 0; s < 4; ++s) d[s] = *(LAS u32x4*)(S + s * 1024 + lane * 16);
            wait_until(tstart + (u64)(T_O0 + ((tile * D_O) >> 8)));
#pragma unroll
            for (int s = 0; s < 4; ++s) if ((u32)(lane + 64 * s) < nch) st16_sys(dst + s * 1024, d[s]);
            drain();
            if (lane == 0) flag_store((gu8*)P.ws[wv] + WS_OFLAG + (rank * 256 + tile) * 128, fv | (raw ? F_RAW : 0u) | (nexc ? F_EXC : 0u) | (obase << 12) | ntot);
        }
        const int ur = (u + 1 - wa) & 31, rtile = ur * 8 + v;
        u64 fval;
        const bool ok2 = wait_flags(myws + WS_OFLAG + (wv * 256 + rtile) * 128, call, tmo, fval);
        const u32 fl = rfl((u32)fval);
        const u32 n = fl & 4095u, rb = (fl >> 12) & 127u;
        const bool rraw = (fl & F_RAW) != 0u;
        const u32 rch = rraw ? 256u : img_chunks(n, (fl & F_EXC) != 0u);
        const __amdgpu_buffer_rsrc_t rs = __builtin_amdgcn_make_buffer_rsrc((void*)(myws + WS_ODATA + wv * OSRC + (size_t)rtile * SLOT), 0, rch * 16, 0x00020000);
        const u32 mbw = __builtin_amdgcn_raw_buffer_load_b32(rs, lane * 4, 0, 17);
        u32x4 in[4];
#pragma unroll
        for (int s = 0; s < 4; ++s) in[s] = __builtin_amdgcn_raw_buffer_load_b128(rs, lane * 16 + s * 1024, 0, 17);
        const u32 tm2 = __hip_atomic_load(tmo, __ATOMIC_RELAXED, __HIP_MEMORY_SCOPE_AGENT);
        const int rrow0 = 1024 * (wv >> 2) + 32 * ur, rcol0 = 512 * (wv & 3) + 64 * v;
        gu8* odst = out + ((size_t)(rrow0 + (lane >> 3)) * N_ + rcol0) * 2 + (lane & 7) * 16;
        if (RARE(!ok2 || tm2 == call)) {
#pragma unroll
            for (int s = 0; s < 4; ++s) st16_sys(odst + s * PASS, (u32x4){0x7FC07FC0u, 0x7FC07FC0u, 0x7FC07FC0u, 0x7FC07FC0u});
        } else if (RARE(rraw)) {
#pragma unroll
            for (int s = 0; s < 4; ++s) st16_sys(odst + s * PASS, in[s]);
        } else {
            lu8* sc = lds + L_RED + wv * RED_WAVE;
            const u32 mb0 = mbw & 255u, mb1 = (mbw >> 8) & 255u, mb2 = (mbw >> 16) & 255u, mb3 = mbw >> 24;
            u32x2 sel[4];
            sel[0] = *(LAS u32x2*)(lds + L_LUT + mb0 * 8); sel[1] = *(LAS u32x2*)(lds + L_LUT + mb1 * 8);
            sel[2] = *(LAS u32x2*)(lds + L_LUT + mb2 * 8); sel[3] = *(LAS u32x2*)(lds + L_LUT + mb3 * 8);
            const u32 c01 = (u32)__popc(mb0) | ((u32)__popc(mb1) << 16), c23 = (u32)__popc(mb2) | ((u32)__popc(mb3) << 16);
            const u32 s01 = wave_scan_add(c01), s23 = wave_scan_add(c23);
#pragma unroll
            for (int s = 0; s < 4; ++s) *(LAS u32x4*)(sc + s * 1024 + lane * 16) = in[s];
            asm volatile("" ::: "memory");
            const u32 coff = img_coff(n);
            const u32 t01 = rdl(s01, 63), t23 = rdl(s23, 63);
            const u32 e01 = s01 - c01, e23 = s23 - c23;
            u32 off[4];
            off[0] = e01 & 0xFFFFu; off[1] = (t01 & 0xFFFFu) + (e01 >> 16);
            const u32 b2 = (t01 & 0xFFFFu) + (t01 >> 16);
            off[2] = b2 + (e23 & 0xFFFFu); off[3] = b2 + (t23 & 0xFFFFu) + (e23 >> 16);
            const u32 b4 = rb * 0x01010101u;
            u32 a0[4], a1[4], a2[4], k0[4], k1[4];
#pragma unroll
            for (int i = 0; i < 4; ++i) {
                lu32* lp = (lu32*)(sc + 256 + (off[i] & ~3u));
                a0[i] = lp[0]; a1[i] = lp[1]; a2[i] = lp[2];
                lu32* cp = (lu32*)(sc + coff + ((3u * off[i]) >> 5) * 4);
                k0[i] = cp[0]; k1[i] = cp[1];
            }
#pragma unroll
            for (int i = 0; i < 4; ++i) {
                const u32 lo0 = __builtin_amdgcn_alignbyte(a1[i], a0[i], off[i] & 3u), lo1 = __builtin_amdgcn_alignbyte(a2[i], a1[i], off[i] & 3u);
                const u32 c = __builtin_amdgcn_alignbit(k1[i], k0[i], (3u * off[i]) & 31u);
                u32 x0 = c & 0xFFFu, x1 = (c >> 12) & 0xFFFu;
                x0 = (x0 | (x0 << 10)) & 0x003F003Fu; x1 = (x1 | (x1 << 10)) & 0x003F003Fu;
                x0 = (x0 | (x0 << 5)) & 0x07070707u; x1 = (x1 | (x1 << 5)) & 0x07070707u;
                const u32 h0 = x0 + b4, h1 = x1 + b4;
                const u32 l0 = perm(lo1, lo0, sel[i].x), l1 = perm(lo1, lo0, sel[i].y), g0 = perm(h1, h0, sel[i].x), g1 = perm(h1, h0, sel[i].y);
                st16_sys(odst + i * PASS, (u32x4){perm(g0, l0, 0x05010400u), perm(g0, l0, 0x07030602u), perm(g1, l1, 0x05010400u), perm(g1, l1, 0x07030602u)});
            }
            if (RARE(fl & F_EXC)) {
                drain();
                if (lane < 8) {
                    const u32 e = *(LAS u32*)(sc + img_eoff(n) + lane * 4);
                    if (e != 0u) { const u32 ix = (e >> 16) & 2047u; *(gu16*)(out + ((size_t)(rrow0 + (ix >> 6)) * N_ + rcol0 + (ix & 63u)) * 2) = (unsigned short)e; }
                }
            }
        }
    }
}

extern "C" void kernel_launch(int rank, int nranks, ncclComm_t comm, hipStream_t stream,
                              void* const* d_in, const int* in_sizes, int n_in,
                              void* d_out, int out_size, const int* in_dtypes, int out_dtype,
                              void* const* d_out_all, void* const* d_ws_all, size_t ws_bytes) {
    (void)comm; (void)d_out_all; (void)in_dtypes;
    if (nranks != NR || (unsigned)rank >= (unsigned)NR || ws_bytes < WS_END || n_in != 2 || in_sizes[0] != M_ * KS || in_sizes[1] != KS * N_ ||
        out_size != M_ * N_ || out_dtype != 1) {
        fprintf(stderr, "kernel_launch: unexpected problem shape\n");
        return;
    }
    static bool init = false;
    if (!init) {
        if (hipFuncSetAttribute((const void*)fused, hipFuncAttributeMaxDynamicSharedMemorySize, LDS_BYTES) != hipSuccess) fprintf(stderr, "kernel_launch: LDS attribute\n");
        init = true;
    }
    Params P;
    P.x = (const unsigned short*)d_in[0]; P.w = (const unsigned short*)d_in[1];
    for (int i = 0; i < NR; ++i) P.ws[i] = (unsigned char*)d_ws_all[i];
    P.my = (unsigned char*)d_ws_all[rank];
    P.out = (unsigned char*)d_out;
    P.rank = rank; P.pad = 0;
    hipLaunchKernelGGL(fused, dim3(NWG), dim3(NTHR), LDS_BYTES, stream, P);
}
```
